```python
import numpy as np

kernel_name = "gpu_a2a_gemm_m4096_k4096_n4096_e4m3e4m3_dqrelu"


ABSMAX_RTOL = 0.03

COMPUTE_DTYPE = "float32"
EPILOGUE = "dequant_relu"
IN_QUANT = None
OUT_QUANT = None
IN_FP8 = "e4m3e4m3"
IN_DTYPES = {"x": "e4m3", "w_mat": "e4m3",
             "scale_x": "float32", "scale_w": "float32"}
WORLD_SIZE = 8

def _snap_fp8(x, mant_bits, max_abs, min_normal):
    a = np.abs(x.astype(np.float64))
    _m, _e = np.frexp(a)
    q = np.where(a >= min_normal, np.ldexp(1.0, _e - 1 - mant_bits),
                 min_normal * 2.0 ** -mant_bits)
    snapped = np.minimum(np.round(a / q) * q, max_abs)
    return (np.sign(x) * snapped).astype(np.float32)

def setup_inputs():

    x = _snap_fp8(np.random.randn(4096, 4096).astype(np.float32), 3, 448.0, 2.0 ** -6)
    w_mat = _snap_fp8(np.random.randn(4096, 4096).astype(np.float32), 3, 448.0, 2.0 ** -6)
    scale_x = (0.02 * (1.0 + np.random.rand(1))).astype(np.float32)
    scale_w = (0.02 * (1.0 + np.random.rand(1))).astype(np.float32)
    return {"x": x, "w_mat": w_mat, "scale_x": scale_x, "scale_w": scale_w}

def reference(x, w_mat, scale_x, scale_w):
    acc = (x @ w_mat).astype(np.float32)
    y = (acc * (scale_x * scale_w)).astype(np.float32)
    return np.maximum(y, 0.0).astype(np.float32)
```

```cpp
#include <hip/hip_runtime.h>
#include <rccl/rccl.h>
#include <cstdio>

#define HIP_CHECK(expr) do { hipError_t _e = (expr); if (_e != hipSuccess) { \
    fprintf(stderr, "HIP error %s at %s:%d\n", hipGetErrorString(_e), __FILE__, __LINE__); \
    return; } } while (0)

namespace {

#define GAS __attribute__((address_space(1)))
#define LAS __attribute__((address_space(3)))
typedef unsigned u32x4 __attribute__((ext_vector_type(4)));
typedef int v2i __attribute__((ext_vector_type(2)));
typedef int v4i __attribute__((ext_vector_type(4)));
typedef int v8i __attribute__((ext_vector_type(8)));
typedef float v4f __attribute__((ext_vector_type(4)));

constexpr int NR = 8;
constexpr int NTOT = 4096, KC = 512;
constexpr int BM = 64, BN = 128, BK = 128;
constexpr int NSTEP = 32;
constexpr int NWG = 256;
constexpr int PIECE = BM * BK;
constexpr int WSLOT = BK * BN;
#define K_DW 6
#define K_DA 7
constexpr int DW = K_DW, DA = K_DA;
constexpr int LDS_W = 0, LDS_A = DW * WSLOT, LDS_CTL = LDS_A + DA * PIECE;
constexpr int LDS_BYTES = LDS_CTL + 64;
static_assert(LDS_BYTES <= 160 * 1024 && LDS_BYTES > 80 * 1024, "one workgroup per CU");
static_assert(DW * 8 <= 63 && DA * 8 + 1 <= 63, "vmcnt has 6 bits");

constexpr size_t WS_FLAG = 0;
constexpr size_t WS_CNTP = 64 * 1024;
constexpr size_t WS_CNTA = 96 * 1024;
constexpr size_t WS_TMO = 128 * 1024;
constexpr size_t WS_DBG = 256 * 1024;
constexpr size_t WS_A = 1024 * 1024;
constexpr size_t WS_A_HALF = (size_t)NR * 8 * 4 * PIECE;
constexpr size_t WS_END = WS_A + 2 * WS_A_HALF;
constexpr unsigned long long DEADLINE_TICKS = 50000000ull;

struct Params {
    const unsigned char* x;
    const unsigned char* w;
    const float* sx;
    const float* sw;
    float* out;
    unsigned char* ws[NR];
    int rank;
    int pad;
};

#define STAMP(i) do {} while (0)

__device__ __forceinline__ u32x4 lds_rd128(unsigned addr) {
    u32x4 v;
    asm volatile("ds_read_b128 %0, %1\n\ts_waitcnt lgkmcnt(0)" : "=v"(v) : "v"(addr) : "memory");
    return v;
}
__device__ __forceinline__ void lds_wr32(unsigned addr, unsigned v) {
    asm volatile("ds_write_b32 %0, %1" :: "v"(addr), "v"(v) : "memory");
}
__device__ __forceinline__ unsigned umin_(unsigned a, unsigned b) { return a < b ? a : b; }
__device__ __forceinline__ unsigned done_min(unsigned ctl) {
    u32x4 d = lds_rd128(ctl + 16);
    return __builtin_amdgcn_readfirstlane(umin_(umin_(d[0], d[1]), umin_(d[2], d[3])));
}
template <int EXTRA>
__device__ __forceinline__ void wait_vm8(int n8) {
    switch (n8) {
        case 0: asm volatile("s_waitcnt vmcnt(%0)" :: "i"(EXTRA) : "memory"); break;
        case 1: asm volatile("s_waitcnt vmcnt(%0)" :: "i"(8 + EXTRA) : "memory"); break;
        case 2: asm volatile("s_waitcnt vmcnt(%0)" :: "i"(16 + EXTRA) : "memory"); break;
        case 3: asm volatile("s_waitcnt vmcnt(%0)" :: "i"(24 + EXTRA) : "memory"); break;
        case 4: asm volatile("s_waitcnt vmcnt(%0)" :: "i"(32 + EXTRA) : "memory"); break;
        case 5: asm volatile("s_waitcnt vmcnt(%0)" :: "i"(40 + EXTRA) : "memory"); break;
        default: asm volatile("s_waitcnt vmcnt(%0)" :: "i"(48 + EXTRA) : "memory"); break;
    }
}

__device__ __forceinline__ void step_src(int s, int rank, int& src, int& ks) {
    if (s < 4) { src = rank; ks = s; }
    else { int t = s - 4; ks = t / 7; src = (rank + 1 + (t - ks * 7)) & 7; }
}

__device__ __forceinline__ void glds_w(const unsigned char* sbase, unsigned dst, const unsigned (&v)[8]) {
    unsigned keep;
    asm volatile(
        "s_mov_b32 %0, m0\n\t"
        "s_mov_b32 m0, %2\n\ts_nop 0\n\tglobal_load_lds_dwordx4 %3, %1\n\t"
        "s_add_u32 m0, m0, 0x400\n\ts_nop 0\n\tglobal_load_lds_dwordx4 %4, %1\n\t"
        "s_add_u32 m0, m0, 0xc00\n\ts_nop 0\n\tglobal_load_lds_dwordx4 %5, %1\n\t"
        "s_add_u32 m0, m0, 0x400\n\ts_nop 0\n\tglobal_load_lds_dwordx4 %6, %1\n\t"
        "s_add_u32 m0, m0, 0xc00\n\ts_nop 0\n\tglobal_load_lds_dwordx4 %7, %1\n\t"
        "s_add_u32 m0, m0, 0x400\n\ts_nop 0\n\tglobal_load_lds_dwordx4 %8, %1\n\t"
        "s_add_u32 m0, m0, 0xc00\n\ts_nop 0\n\tglobal_load_lds_dwordx4 %9, %1\n\t"
        "s_add_u32 m0, m0, 0x400\n\ts_nop 0\n\tglobal_load_lds_dwordx4 %10, %1\n\t"
        "s_mov_b32 m0, %0"
        : "=&s"(keep)
        : "s"(sbase), "s"(dst), "v"(v[0]), "v"(v[1]), "v"(v[2]), "v"(v[3]), "v"(v[4]), "v"(v[5]), "v"(v[6]), "v"(v[7])
        : "memory", "scc");
}
__device__ __forceinline__ void glds_a_local(const unsigned char* sbase, unsigned dst, const unsigned (&v)[8]) {
    unsigned keep;
    asm volatile(
        "s_mov_b32 %0, m0\n\t"
        "s_mov_b32 m0, %2\n\ts_nop 0\n\tglobal_load_lds_dwordx4 %3, %1\n\t"
        "s_add_u32 m0, m0, 0x400\n\ts_nop 0\n\tglobal_load_lds_dwordx4 %4, %1\n\t"
        "s_add_u32 m0, m0, 0x400\n\ts_nop 0\n\tglobal_load_lds_dwordx4 %5, %1\n\t"
        "s_add_u32 m0, m0, 0x400\n\ts_nop 0\n\tglobal_load_lds_dwordx4 %6, %1\n\t"
        "s_add_u32 m0, m0, 0x400\n\ts_nop 0\n\tglobal_load_lds_dwordx4 %7, %1\n\t"
        "s_add_u32 m0, m0, 0x400\n\ts_nop 0\n\tglobal_load_lds_dwordx4 %8, %1\n\t"
        "s_add_u32 m0, m0, 0x400\n\ts_nop 0\n\tglobal_load_lds_dwordx4 %9, %1\n\t"
        "s_add_u32 m0, m0, 0x400\n\ts_nop 0\n\tglobal_load_lds_dwordx4 %10, %1\n\t"
        "s_mov_b32 m0, %0"
        : "=&s"(keep)
        : "s"(sbase), "s"(dst), "v"(v[0]), "v"(v[1]), "v"(v[2]), "v"(v[3]), "v"(v[4]), "v"(v[5]), "v"(v[6]), "v"(v[7])
        : "memory", "scc");
}
__device__ __forceinline__ void glds_a_remote(const unsigned char* sbase, unsigned dst, unsigned v0, unsigned v1) {
    unsigned keep;
    asm volatile(
        "s_mov_b32 %0, m0\n\t"
        "s_mov_b32 m0, %2\n\ts_nop 0\n\t"
        "global_load_lds_dwordx4 %3, %1 sc0 sc1\n\t"
        "global_load_lds_dwordx4 %3, %1 offset:1024 sc0 sc1\n\t"
        "global_load_lds_dwordx4 %3, %1 offset:2048 sc0 sc1\n\t"
        "global_load_lds_dwordx4 %3, %1 offset:3072 sc0 sc1\n\t"
        "s_add_u32 m0, m0, 0x1000\n\ts_nop 0\n\t"
        "global_load_lds_dwordx4 %4, %1 sc0 sc1\n\t"
        "global_load_lds_dwordx4 %4, %1 offset:1024 sc0 sc1\n\t"
        "global_load_lds_dwordx4 %4, %1 offset:2048 sc0 sc1\n\t"
        "global_load_lds_dwordx4 %4, %1 offset:3072 sc0 sc1\n\t"
        "s_mov_b32 m0, %0"
        : "=&s"(keep)
        : "s"(sbase), "s"(dst), "v"(v0), "v"(v1)
        : "memory", "scc");
}

struct Frags { v8i w[4]; v8i a[2]; };

__device__ __forceinline__ v2i tr8(unsigned addr) {
    return __builtin_amdgcn_ds_read_tr8_b64_v2i32((LAS v2i*)addr);
}
__device__ __forceinline__ void load_frags(Frags& f, const unsigned (&wb)[4], const unsigned (&ab)[2],
                                           unsigned woff, unsigned aoff) {
#pragma unroll
    for (int mf = 0; mf < 2; ++mf) {
        v4i lo = *(LAS v4i*)(ab[0] + aoff + mf * 2048);
        v4i hi = *(LAS v4i*)(ab[1] + aoff + mf * 2048);
        f.a[mf] = __builtin_shufflevector(lo, hi, 0, 1, 2, 3, 4, 5, 6, 7);
    }
#pragma unroll
    for (int nf = 0; nf < 4; ++nf) {
        unsigned b = wb[nf] + woff;
        v2i t0 = tr8(b), t1 = tr8(b + 1024), t2 = tr8(b + 8192), t3 = tr8(b + 9216);
        v4i lo = __builtin_shufflevector(t0, t1, 0, 1, 2, 3);
        v4i hi = __builtin_shufflevector(t2, t3, 0, 1, 2, 3);
        f.w[nf] = __builtin_shufflevector(lo, hi, 0, 1, 2, 3, 4, 5, 6, 7);
    }
}
__device__ __forceinline__ void mma(v4f (&acc)[4][2], const Frags& f) {
#pragma unroll
    for (int nf = 0; nf < 4; ++nf)
#pragma unroll
        for (int mf = 0; mf < 2; ++mf)
            acc[nf][mf] = __builtin_amdgcn_mfma_scale_f32_16x16x128_f8f6f4(f.w[nf], f.a[mf], acc[nf][mf], 0, 0, 0, 0, 0, 0);
}
__device__ __forceinline__ unsigned ready_min(unsigned ctl) {
    u32x4 d = lds_rd128(ctl);
    return __builtin_amdgcn_readfirstlane(umin_(umin_(d[0], d[1]), d[2]));
}

__global__ __launch_bounds__(512, 2) void a2a_gemm(Params p) {
    extern __shared__ __attribute__((aligned(16))) unsigned char smem[];
    const unsigned lds0 = (unsigned)(size_t)(LAS unsigned char*)smem;
    const unsigned ctl = lds0 + LDS_CTL;
    const int lane = threadIdx.x & 63;
    const int wave = __builtin_amdgcn_readfirstlane(threadIdx.x >> 6);
    const int b = blockIdx.x;
    const int rank = p.rank;
    const int rb = (b >> 3) & 7;
    const int cb = (b & 7) * 4 + (b >> 6);
    unsigned char* myws = p.ws[rank];
    const bool pusher_wg = b < 56;

    if (wave == 0) {
        STAMP(0);
        if (!pusher_wg) { __builtin_amdgcn_s_barrier(); return; }
        const int g = b / 7;
        const unsigned dest = (unsigned)(rank + 1 + (b - g * 7)) & 7u;
        const unsigned char* src = p.x + ((size_t)dest * 512 + g * 64) * KC;
        const unsigned r = lane >> 3, c = lane & 7;
        const unsigned vo0 = r * KC + ((c ^ ((r >> 1) & 3)) << 4);
        const unsigned vo1 = (r + 8) * KC + ((c ^ (((r >> 1) & 3) | 4)) << 4);
        unsigned* cnt = (unsigned*)(myws + WS_CNTP + (size_t)b * 128);
        unsigned call;
        asm volatile("global_load_dword %0, %1, %2 sc1" : "=v"(call) : "v"(0u), "s"(cnt) : "memory");
        u32x4 d[4][8];
#pragma unroll
        for (int ks = 0; ks < 4; ++ks)
#pragma unroll
            for (int u = 0; u < 8; ++u)
                asm volatile("global_load_dwordx4 %0, %1, %2 offset:%3"
                             : "=v"(d[ks][u]) : "v"((u & 1) ? vo1 : vo0), "s"(src + (u >> 1) * 16 * KC), "i"(ks * 128) : "memory");
        __builtin_amdgcn_s_barrier();
        asm volatile("s_waitcnt vmcnt(32)" : "+v"(call) :: "memory");
        call = __builtin_amdgcn_readfirstlane(call) + 1;
        unsigned char* dws = p.ws[dest];
        unsigned char* dst = dws + WS_A + (call & 1) * WS_A_HALF + (size_t)((rank * 8 + g) * 4) * PIECE;
        const unsigned so0 = lane * 16, so1 = so0 + 4096;
#pragma unroll
        for (int ks = 0; ks < 4; ++ks) {
            asm volatile("s_waitcnt vmcnt(24)"
                         : "+v"(d[ks][0]), "+v"(d[ks][1]), "+v"(d[ks][2]), "+v"(d[ks][3]),
                           "+v"(d[ks][4]), "+v"(d[ks][5]), "+v"(d[ks][6]), "+v"(d[ks][7]) :: "memory");
            if (ks == 0) STAMP(1);
#pragma unroll
            for (int u = 0; u < 8; ++u)
                asm volatile("global_store_dwordx4 %0, %1, %2 offset:%3 sc0 sc1\n\ts_nop 1"
                             :: "v"(u < 4 ? so0 : so1), "v"(d[ks][u]), "s"(dst + ks * PIECE), "i"((u & 3) * 1024) : "memory");
        }
        unsigned char* fl = dws + WS_FLAG + (size_t)((rank * 8 + g) * 4) * 128;
        const unsigned long long m1 = 1;
#define PUBLISH(N, KS) asm volatile("s_waitcnt vmcnt(" #N ")\n\t" \
                                    "s_mov_b64 exec, %3\n\t" \
                                    "global_store_dword %0, %1, %2 offset:" #KS "*128 sc0 sc1\n\t" \
                                    "s_mov_b64 exec, -1" \
                                    :: "v"(0u), "v"(call), "s"(fl), "s"(m1) : "memory")
        PUBLISH(24, 0); PUBLISH(17, 1); PUBLISH(10, 2); PUBLISH(3, 3);
#undef PUBLISH
        if (lane == 0) __hip_atomic_store(cnt, call, __ATOMIC_RELAXED, __HIP_MEMORY_SCOPE_AGENT);
        return;
    }

    if (wave == 1) {
        unsigned* cnt = (unsigned*)(myws + WS_CNTA + (size_t)b * 128);
        unsigned call;
        asm volatile("global_load_dword %0, %1, %2 sc1" : "=v"(call) : "v"(0u), "s"(cnt) : "memory");
        if (pusher_wg) __builtin_amdgcn_s_barrier();
        {
            const unsigned r = lane >> 3, c = lane & 7;
            unsigned v[8];
#pragma unroll
            for (int u = 0; u < 8; ++u)
                v[u] = (r + 8 * u) * KC + ((c ^ (((r >> 1) & 3) | ((u & 1) << 2))) << 4);
            const unsigned char* src = p.x + ((size_t)rank * 512 + rb * 64) * KC;
#pragma unroll
            for (int ks = 0; ks < 4; ++ks) glds_a_local(src + ks * 128, lds0 + LDS_A + ks * PIECE, v);
        }
        if (!pusher_wg) __builtin_amdgcn_s_barrier();
        asm volatile("s_waitcnt vmcnt(32)" : "+v"(call) :: "memory");
        call = __builtin_amdgcn_readfirstlane(call) + 1;
        unsigned foff;
        {
            int src, ks;
            step_src(lane < 28 ? lane + 4 : 4, rank, src, ks);
            foff = (unsigned)WS_FLAG + ((src * 8 + rb) * 4 + ks) * 128;
            if (lane >= 28) foff = (unsigned)WS_TMO;
        }
        const unsigned char* abase = myws + WS_A + (call & 1) * WS_A_HALF;
        const unsigned so0 = lane * 16, so1 = so0 + 4096;
        const unsigned long long t0 = __builtin_amdgcn_s_memrealtime();
        int issued = 4, pub = 0, known = 4;
        bool dead = false;
        while (pub < NSTEP) {
            if (issued < known) {
                const int lim = (int)done_min(ctl) + DA;
                while (issued < known && issued < lim) {
                    int src, ks;
                    step_src(issued, rank, src, ks);
                    glds_a_remote(abase + (size_t)((src * 8 + rb) * 4 + ks) * PIECE,
                                  lds0 + LDS_A + (issued % DA) * PIECE, so0, so1);
                    ++issued;
                }
            }
            if (known < NSTEP) {
                unsigned f;
                const unsigned long long pm = (0x1fffffffull >> (known - 4)) << (known - 4);
                asm volatile("s_mov_b64 exec, %3\n\t"
                             "global_load_dword %0, %1, %2 sc0 sc1\n\t"
                             "s_mov_b64 exec, -1"
                             : "=&v"(f) : "v"(foff), "s"(myws), "s"(pm) : "memory");
                while (pub < issued) {
                    wait_vm8<1>(issued - pub - 1);
                    ++pub;
                    lds_wr32(ctl + 8, pub);
                }
                asm volatile("s_waitcnt vmcnt(0)" : "+v"(f) :: "memory");
                const unsigned long long ok = __ballot(f >= call) & pm;
                const unsigned long long eq = __ballot(f == call) & pm;
                const unsigned long long notok = (~ok & 0xfffffffull) >> (known - 4);
                const int nk = notok ? known + __builtin_ctzll(notok) : NSTEP;
                if ((eq >> 28) & 1) { dead = true; break; }
                if (nk == known) {
                    if (__builtin_amdgcn_s_memrealtime() - t0 > DEADLINE_TICKS) {
                        if (lane == 0)
                            __hip_atomic_store((unsigned*)(myws + WS_TMO), call, __ATOMIC_RELAXED, __HIP_MEMORY_SCOPE_AGENT);
                        dead = true;
                        break;
                    }
                    __builtin_amdgcn_s_sleep(1);
                }
                known = nk;
            } else if (pub < issued) {
                wait_vm8<0>(issued - pub - 1);
                ++pub;
                lds_wr32(ctl + 8, pub);
            } else {
                __builtin_amdgcn_s_sleep(1);
            }
        }
        if (dead) {
            asm volatile("s_waitcnt vmcnt(0)" ::: "memory");
            lds_wr32(ctl + 12, 1);
            lds_wr32(ctl + 8, NSTEP);
        }
        if (lane == 0) __hip_atomic_store(cnt, call, __ATOMIC_RELAXED, __HIP_MEMORY_SCOPE_AGENT);
        STAMP(3);
        return;
    }

    if (wave < 4) {
        const int lw = wave - 2;
        unsigned v[8];
        {
            const unsigned r = lane >> 3, c = lane & 7;
#pragma unroll
            for (int j = 0; j < 8; ++j) {
                const unsigned u = (j & 1) + 4 * (j >> 1);
                v[j] = (r + 8 * (u + 2 * lw)) * NTOT + ((c ^ (((r >> 1) & 3) | (lw << 2))) << 4);
            }
        }
        const unsigned char* wb = p.w + cb * BN;
        const unsigned dst0 = lds0 + LDS_W + lw * 2048;
        if (pusher_wg) __builtin_amdgcn_s_barrier();
        int issued = 0, pub = 0;
        for (; issued < DW; ++issued) {
            int src, ks;
            step_src(issued, rank, src, ks);
            glds_w(wb + (size_t)(src * KC + ks * BK) * NTOT, dst0 + issued * WSLOT, v);
        }
        if (!pusher_wg) __builtin_amdgcn_s_barrier();
        while (pub < NSTEP) {
            bool did = false;
            if (issued < NSTEP) {
                const int lim = (int)done_min(ctl) + DW;
                while (issued < NSTEP && issued < lim) {
                    int src, ks;
                    step_src(issued, rank, src, ks);
                    glds_w(wb + (size_t)(src * KC + ks * BK) * NTOT, dst0 + (issued % DW) * WSLOT, v);
                    ++issued;
                }
            }
            if (pub < issued) {
                wait_vm8<0>(issued - pub - 1);
                ++pub;
                lds_wr32(ctl + 4 * lw, pub);
                did = true;
            }
            if (!did) __builtin_amdgcn_s_sleep(1);
        }
        if (lw == 0) STAMP(4);
        return;
    }

    const int cw = wave - 4, wm = cw & 1, wn = cw >> 1;
    if (cw == 0 && lane < 16) lds_wr32(ctl + lane * 4, 0);
    const float scale = p.sx[0] * p.sw[0];
    unsigned wbase[4], abase[2];
    {
        const unsigned q = lane >> 4, i = (lane & 15) >> 1, pp = lane & 1;
        const unsigned fw = ((i >> 1) & 3) | ((q & 1) << 2);
#pragma unroll
        for (int nf = 0; nf < 4; ++nf)
            wbase[nf] = lds0 + LDS_W + (16 * q + i) * 128 + 8 * pp + (((wn * 4 + nf) ^ fw) << 4);
        const unsigned row = wm * 32 + (lane & 15), fa = (row >> 1) & 7;
        abase[0] = lds0 + LDS_A + row * 128 + ((q ^ fa) << 4);
        abase[1] = lds0 + LDS_A + row * 128 + (((q + 4) ^ fa) << 4);
    }
    v4f acc[4][2];
#pragma unroll
    for (int nf = 0; nf < 4; ++nf)
#pragma unroll
        for (int mf = 0; mf < 2; ++mf) acc[nf][mf] = v4f{0.f, 0.f, 0.f, 0.f};
    asm volatile("s_waitcnt lgkmcnt(0)" ::: "memory");
    __builtin_amdgcn_s_barrier();

    Frags f0, f1;
    unsigned ready = 0;
    int sw = 0, sa = 0;
    while ((ready = ready_min(ctl)) < 1u) __builtin_amdgcn_s_sleep(1);
    if (cw == 0) STAMP(5);
    load_frags(f0, wbase, abase, 0, 0);
    sw = 1; sa = 1;
#define STEP(CUR, NXT, S)                                                        \
    {                                                                            \
        __builtin_amdgcn_s_waitcnt(0xC07F);                       \
        lds_wr32(ctl + 16 + 4 * cw, (S) + 1);                                    \
        const bool more = (S) + 1 < NSTEP;                                       \
        if (more && ready < (unsigned)(S) + 2) ready = ready_min(ctl);           \
        const bool early = more && ready >= (unsigned)(S) + 2;                   \
        if (early) load_frags(NXT, wbase, abase, sw * WSLOT, sa * PIECE);        \
        __builtin_amdgcn_sched_barrier(0);                                       \
        mma(acc, CUR);                                                           \
        __builtin_amdgcn_sched_barrier(0);                                       \
        if (more && !early) {                                                    \
            while ((ready = ready_min(ctl)) < (unsigned)(S) + 2) {}              \
            load_frags(NXT, wbase, abase, sw * WSLOT, sa * PIECE);               \
        }                                                                        \
        sw = sw + 1 == DW ? 0 : sw + 1;                                          \
        sa = sa + 1 == DA ? 0 : sa + 1;                                          \
    }
    for (int s = 0; s < NSTEP; s += 2) {
        STEP(f0, f1, s);
        STEP(f1, f0, s + 1);
    }
#undef STEP
    if (cw == 0) STAMP(6);
    const bool dead = __builtin_amdgcn_readfirstlane(lds_rd128(ctl)[3]) != 0;
    float* o = p.out + (size_t)(rb * BM + wm * 32 + (lane & 15)) * NTOT + cb * BN + wn * 64 + (lane >> 4) * 4;
#pragma unroll
    for (int mf = 0; mf < 2; ++mf)
#pragma unroll
        for (int nf = 0; nf < 4; ++nf) {
            v4f y;
#pragma unroll
            for (int j = 0; j < 4; ++j) y[j] = dead ? __builtin_nanf("") : fmaxf(acc[nf][mf][j] * scale, 0.f);
            *(v4f*)(o + (size_t)mf * 16 * NTOT + nf * 16) = y;
        }
}

}

extern "C" void kernel_launch(int rank, int nranks, ncclComm_t comm,
                              hipStream_t stream,
                              void* const* d_in, const int* in_sizes, int n_in,
                              void* d_out, int out_size,
                              const int* in_dtypes, int out_dtype,
                              void* const* d_out_all,
                              void* const* d_ws_all, size_t ws_bytes) {
    if (nranks != NR || (unsigned)rank >= (unsigned)NR || ws_bytes < WS_END || n_in != 4 ||
        in_sizes[0] != 4096 * KC || in_sizes[1] != 4096 * NTOT || out_size != 512 * NTOT) {
        fprintf(stderr, "kernel_launch: unexpected problem shape\n");
        return;
    }
    HIP_CHECK(hipFuncSetAttribute((const void*)a2a_gemm, hipFuncAttributeMaxDynamicSharedMemorySize, LDS_BYTES));
    Params p;
    p.x = (const unsigned char*)d_in[0];
    p.w = (const unsigned char*)d_in[1];
    p.sx = (const float*)d_in[2];
    p.sw = (const float*)d_in[3];
    p.out = (float*)d_out;
    for (int i = 0; i < NR; ++i) p.ws[i] = (unsigned char*)d_ws_all[i];
    p.rank = rank;
    p.pad = 0;
    a2a_gemm<<<NWG, 512, LDS_BYTES, stream>>>(p);
    HIP_CHECK(hipGetLastError());
}
```

```cpp
#include <hip/hip_runtime.h>
#include <rccl/rccl.h>
#include <cstdio>

#define HIP_CHECK(expr) do { hipError_t _e = (expr); if (_e != hipSuccess) { \
    fprintf(stderr, "HIP error %s at %s:%d\n", hipGetErrorString(_e), __FILE__, __LINE__); \
    return; } } while (0)

namespace {

typedef unsigned u32x4 __attribute__((ext_vector_type(4)));

#define K_TAU 16
#define K_PFT 560
#define K_LEAD 120

constexpr int NR = 8;
constexpr int NTOT = 4096, KC = 512;
constexpr int BN = 128, BK = 128;
constexpr int NWG = 256, NTHREADS = 448;
constexpr int SLOT = 32768, SLOT_W = 80 * BK;
constexpr int D = 4;
constexpr int LDS_ABORT = D * SLOT - 16, LDS_BYTES = D * SLOT;
static_assert(LDS_BYTES <= 160 * 1024 && LDS_BYTES > 80 * 1024, "one workgroup per CU");

constexpr size_t WS_SRC = 1 << 20;
constexpr size_t WS_KS = 1 << 18;
constexpr size_t WS_PARITY = 512 * BK;
constexpr size_t WS_FLAG = 2 * WS_PARITY;
constexpr size_t WS_CNT = WS_KS + 2 * WS_PARITY;
constexpr size_t WS_TMO = WS_FLAG + 16 * 1024;
constexpr size_t WS_END = NR * WS_SRC;
constexpr unsigned DEADLINE_TICKS = 50000000u;

struct Link {
    unsigned char* dst;
    const unsigned char* src;
};
struct Head {
    const unsigned char* x;
    const unsigned char* w;
    unsigned char* myws;
    float* out;
    const float* sx;
    const float* sw;
    int rank;
    int pad[3];
};
struct Params {
    Head h;
    Link link[8];
};

#define BARRIER() asm volatile("s_barrier" ::: "memory")

__device__ __forceinline__ void lds_wr32(unsigned addr, unsigned v) {
    asm volatile("ds_write_b32 %0, %1\n\ts_waitcnt lgkmcnt(0)" :: "v"(addr), "v"(v) : "memory");
}
__device__ __forceinline__ int lane_id() {
    int l;
    asm volatile("v_mbcnt_lo_u32_b32 %0, -1, 0\n\tv_mbcnt_hi_u32_b32 %0, -1, %0" : "=v"(l));
    __builtin_assume(l >= 0 && l < 64);
    return l;
}
constexpr unsigned kperm(unsigned c) { return c ^ (((c ^ (c >> 1)) & 1) << 2); }
__device__ __forceinline__ unsigned now32() { return (unsigned)__builtin_amdgcn_s_memrealtime(); }
__device__ __forceinline__ int rb_row0(int rb) { return 64 * rb + 16 * (rb < 4 ? rb : 4); }

#define VCALL "%[call]"
#define CONS5_ASM \
      \
    "s_barrier\n\t" \
    "ds_read_b64_tr_b8 v[112:113], %[w00] offset:0\n\t" \
    "ds_read_b64_tr_b8 v[120:121], %[w10] offset:0\n\t" \
    "ds_read_b64_tr_b8 v[114:115], %[w00] offset:1024\n\t" \
    "ds_read_b64_tr_b8 v[122:123], %[w10] offset:1024\n\t" \
    "ds_read_b64_tr_b8 v[116:117], %[w00] offset:8192\n\t" \
    "ds_read_b64_tr_b8 v[124:125], %[w10] offset:8192\n\t" \
    "ds_read_b64_tr_b8 v[118:119], %[w00] offset:9216\n\t" \
    "ds_read_b64_tr_b8 v[126:127], %[w10] offset:9216\n\t" \
    "ds_read_b128 v[56:59], %[a00] offset:0\n\t" \
    "ds_read_b128 v[60:63], %[a10] offset:0\n\t" \
    "ds_read_b128 v[64:67], %[a00] offset:2048\n\t" \
    "ds_read_b128 v[68:71], %[a10] offset:2048\n\t" \
    "ds_read_b128 v[72:75], %[a00] offset:4096\n\t" \
    "ds_read_b128 v[76:79], %[a10] offset:4096\n\t" \
    "ds_read_b128 v[80:83], %[a00] offset:6144\n\t" \
    "ds_read_b128 v[84:87], %[a10] offset:6144\n\t" \
    "ds_read_b128 v[88:91], %[a00] offset:8192\n\t" \
    "ds_read_b128 v[92:95], %[a10] offset:8192\n\t" \
    "s_waitcnt lgkmcnt(0)\n\t" \
    "s_barrier\n\t" \
    "v_mfma_f32_16x16x128_f8f6f4 v[16:19], v[56:63], v[112:119], v[16:19]\n\t" \
    "ds_read_b64_tr_b8 v[128:129], %[w00] offset:32768\n\t" \
    "ds_read_b128 v[96:99], %[a00] offset:38912\n\t" \
    "ds_read_b128 v[100:103], %[a10] offset:38912\n\t" \
    "v_mfma_f32_16x16x128_f8f6f4 v[20:23], v[56:63], v[120:127], v[20:23]\n\t" \
    "ds_read_b64_tr_b8 v[136:137], %[w10] offset:32768\n\t" \
    "ds_read_b128 v[56:59], %[a00] offset:32768\n\t" \
    "ds_read_b128 v[60:63], %[a10] offset:32768\n\t" \
    "v_mfma_f32_16x16x128_f8f6f4 v[24:27], v[64:71], v[112:119], v[24:27]\n\t" \
    "ds_read_b64_tr_b8 v[130:131], %[w00] offset:33792\n\t" \
    "ds_read_b128 v[104:107], %[a00] offset:40960\n\t" \
    "ds_read_b128 v[108:111], %[a10] offset:40960\n\t" \
    "v_mfma_f32_16x16x128_f8f6f4 v[28:31], v[64:71], v[120:127], v[28:31]\n\t" \
    "ds_read_b64_tr_b8 v[138:139], %[w10] offset:33792\n\t" \
    "ds_read_b128 v[64:67], %[a00] offset:34816\n\t" \
    "ds_read_b128 v[68:71], %[a10] offset:34816\n\t" \
    "v_mfma_f32_16x16x128_f8f6f4 v[32:35], v[72:79], v[112:119], v[32:35]\n\t" \
    "ds_read_b64_tr_b8 v[132:133], %[w00] offset:40960\n\t" \
    "ds_read_b64_tr_b8 v[140:141], %[w10] offset:40960\n\t" \
    "ds_read_b64_tr_b8 v[134:135], %[w00] offset:41984\n\t" \
    "v_mfma_f32_16x16x128_f8f6f4 v[36:39], v[72:79], v[120:127], v[36:39]\n\t" \
    "ds_read_b128 v[72:75], %[a00] offset:36864\n\t" \
    "ds_read_b128 v[76:79], %[a10] offset:36864\n\t" \
    "ds_read_b64_tr_b8 v[142:143], %[w10] offset:41984\n\t" \
    "v_mfma_f32_16x16x128_f8f6f4 v[40:43], v[80:87], v[112:119], v[40:43]\n\t" \
    "v_mfma_f32_16x16x128_f8f6f4 v[44:47], v[80:87], v[120:127], v[44:47]\n\t" \
    "v_mfma_f32_16x16x128_f8f6f4 v[48:51], v[88:95], v[112:119], v[48:51]\n\t" \
    "s_waitcnt lgkmcnt(0)\n\t" \
    "s_barrier\n\t" \
    "v_mfma_f32_16x16x128_f8f6f4 v[52:55], v[88:95], v[120:127], v[52:55]\n\t" \
    "v_mfma_f32_16x16x128_f8f6f4 v[16:19], v[56:63], v[128:135], v[16:19]\n\t" \
    "ds_read_b64_tr_b8 v[112:113], %[w01] offset:0\n\t" \
    "ds_read_b128 v[80:83], %[a01] offset:6144\n\t" \
    "ds_read_b128 v[84:87], %[a11] offset:6144\n\t" \
    "v_mfma_f32_16x16x128_f8f6f4 v[20:23], v[56:63], v[136:143], v[20:23]\n\t" \
    "ds_read_b64_tr_b8 v[120:121], %[w11] offset:0\n\t" \
    "ds_read_b128 v[56:59], %[a01] offset:0\n\t" \
    "ds_read_b128 v[60:63], %[a11] offset:0\n\t" \
    "v_mfma_f32_16x16x128_f8f6f4 v[24:27], v[64:71], v[128:135], v[24:27]\n\t" \
    "ds_read_b64_tr_b8 v[114:115], %[w01] offset:1024\n\t" \
    "ds_read_b128 v[88:91], %[a01] offset:8192\n\t" \
    "ds_read_b128 v[92:95], %[a11] offset:8192\n\t" \
    "v_mfma_f32_16x16x128_f8f6f4 v[28:31], v[64:71], v[136:143], v[28:31]\n\t" \
    "ds_read_b64_tr_b8 v[122:123], %[w11] offset:1024\n\t" \
    "ds_read_b128 v[64:67], %[a01] offset:2048\n\t" \
    "ds_read_b128 v[68:71], %[a11] offset:2048\n\t" \
    "v_mfma_f32_16x16x128_f8f6f4 v[32:35], v[72:79], v[128:135], v[32:35]\n\t" \
    "ds_read_b64_tr_b8 v[116:117], %[w01] offset:8192\n\t" \
    "ds_read_b64_tr_b8 v[124:125], %[w11] offset:8192\n\t" \
    "ds_read_b64_tr_b8 v[118:119], %[w01] offset:9216\n\t" \
    "v_mfma_f32_16x16x128_f8f6f4 v[36:39], v[72:79], v[136:143], v[36:39]\n\t" \
    "ds_read_b128 v[72:75], %[a01] offset:4096\n\t" \
    "ds_read_b128 v[76:79], %[a11] offset:4096\n\t" \
    "ds_read_b64_tr_b8 v[126:127], %[w11] offset:9216\n\t" \
    "v_mfma_f32_16x16x128_f8f6f4 v[40:43], v[96:103], v[128:135], v[40:43]\n\t" \
    "v_mfma_f32_16x16x128_f8f6f4 v[44:47], v[96:103], v[136:143], v[44:47]\n\t" \
    "v_mfma_f32_16x16x128_f8f6f4 v[48:51], v[104:111], v[128:135], v[48:51]\n\t" \
    "s_waitcnt lgkmcnt(0)\n\t" \
    "s_barrier\n\t" \
    "v_mfma_f32_16x16x128_f8f6f4 v[52:55], v[104:111], v[136:143], v[52:55]\n\t" \
    "v_mfma_f32_16x16x128_f8f6f4 v[16:19], v[56:63], v[112:119], v[16:19]\n\t" \
    "ds_read_b64_tr_b8 v[128:129], %[w01] offset:32768\n\t" \
    "ds_read_b128 v[96:99], %[a01] offset:38912\n\t" \
    "ds_read_b128 v[100:103], %[a11] offset:38912\n\t" \
    "v_mfma_f32_16x16x128_f8f6f4 v[20:23], v[56:63], v[120:127], v[20:23]\n\t" \
    "ds_read_b64_tr_b8 v[136:137], %[w11] offset:32768\n\t" \
    "ds_read_b128 v[56:59], %[a01] offset:32768\n\t" \
    "ds_read_b128 v[60:63], %[a11] offset:32768\n\t" \
    "v_mfma_f32_16x16x128_f8f6f4 v[24:27], v[64:71], v[112:119], v[24:27]\n\t" \
    "ds_read_b64_tr_b8 v[130:131], %[w01] offset:33792\n\t" \
    "ds_read_b128 v[104:107], %[a01] offset:40960\n\t" \
    "ds_read_b128 v[108:111], %[a11] offset:40960\n\t" \
    "v_mfma_f32_16x16x128_f8f6f4 v[28:31], v[64:71], v[120:127], v[28:31]\n\t" \
    "ds_read_b64_tr_b8 v[138:139], %[w11] offset:33792\n\t" \
    "ds_read_b128 v[64:67], %[a01] offset:34816\n\t" \
    "ds_read_b128 v[68:71], %[a11] offset:34816\n\t" \
    "v_mfma_f32_16x16x128_f8f6f4 v[32:35], v[72:79], v[112:119], v[32:35]\n\t" \
    "ds_read_b64_tr_b8 v[132:133], %[w01] offset:40960\n\t" \
    "ds_read_b64_tr_b8 v[140:141], %[w11] offset:40960\n\t" \
    "ds_read_b64_tr_b8 v[134:135], %[w01] offset:41984\n\t" \
    "v_mfma_f32_16x16x128_f8f6f4 v[36:39], v[72:79], v[120:127], v[36:39]\n\t" \
    "ds_read_b128 v[72:75], %[a01] offset:36864\n\t" \
    "ds_read_b128 v[76:79], %[a11] offset:36864\n\t" \
    "ds_read_b64_tr_b8 v[142:143], %[w11] offset:41984\n\t" \
    "v_mfma_f32_16x16x128_f8f6f4 v[40:43], v[80:87], v[112:119], v[40:43]\n\t" \
    "v_mfma_f32_16x16x128_f8f6f4 v[44:47], v[80:87], v[120:127], v[44:47]\n\t" \
    "v_mfma_f32_16x16x128_f8f6f4 v[48:51], v[88:95], v[112:119], v[48:51]\n\t" \
    "s_waitcnt lgkmcnt(0)\n\t" \
    "s_barrier\n\t" \
    "v_mfma_f32_16x16x128_f8f6f4 v[52:55], v[88:95], v[120:127], v[52:55]\n\t" \
      \
    "s_mov_b32 %[n], 0\n\t" \
    "s_lshl_b32 %[t], %[rb], 4\n\t" \
    "s_sub_u32 %[t], 96, %[t]\n\t" \
    "s_mov_b32 %[t0], 3\n\t" \
    "s_add_u32 %[n], %[n], 32768\n\t" \
    "s_bitset0_b32 %[n], 17\n\t" \
    "v_add_u32 v80, %[n], %[w00]\n\t" \
    "v_add_u32 v81, %[n], %[w10]\n\t" \
    "v_mfma_f32_16x16x128_f8f6f4 v[16:19], v[56:63], v[128:135], v[16:19]\n\t" \
    "ds_read_b64_tr_b8 v[112:113], %[w00] offset:0\n\t" \
    "ds_read_b64_tr_b8 v[114:115], %[w10] offset:0\n\t" \
    "v_mfma_f32_16x16x128_f8f6f4 v[20:23], v[56:63], v[136:143], v[20:23]\n\t" \
    "ds_read_b64_tr_b8 v[116:117], %[w00] offset:1024\n\t" \
    "ds_read_b64_tr_b8 v[118:119], %[w10] offset:1024\n\t" \
    "v_mfma_f32_16x16x128_f8f6f4 v[24:27], v[64:71], v[128:135], v[24:27]\n\t" \
    "ds_read_b64_tr_b8 v[120:121], %[w00] offset:8192\n\t" \
    "ds_read_b64_tr_b8 v[122:123], %[w10] offset:8192\n\t" \
    "v_mfma_f32_16x16x128_f8f6f4 v[28:31], v[64:71], v[136:143], v[28:31]\n\t" \
    "ds_read_b64_tr_b8 v[124:125], %[w00] offset:9216\n\t" \
    "ds_read_b64_tr_b8 v[126:127], %[w10] offset:9216\n\t" \
    "v_mfma_f32_16x16x128_f8f6f4 v[32:35], v[72:79], v[128:135], v[32:35]\n\t" \
    "v_mfma_f32_16x16x128_f8f6f4 v[36:39], v[72:79], v[136:143], v[36:39]\n\t" \
    "v_mfma_f32_16x16x128_f8f6f4 v[40:43], v[96:103], v[128:135], v[40:43]\n\t" \
    "v_mfma_f32_16x16x128_f8f6f4 v[44:47], v[96:103], v[136:143], v[44:47]\n\t" \
    "v_mfma_f32_16x16x128_f8f6f4 v[48:51], v[104:111], v[128:135], v[48:51]\n\t" \
    "s_waitcnt lgkmcnt(0)\n\t" \
    "s_barrier\n\t" \
    "v_mfma_f32_16x16x128_f8f6f4 v[52:55], v[104:111], v[136:143], v[52:55]\n\t" \
    "3:\n\t" \
    "ds_read_b64_tr_b8 v[128:129], v80 offset:0\n\t" \
    "ds_read_b64_tr_b8 v[130:131], v81 offset:0\n\t" \
    "ds_read_b64_tr_b8 v[132:133], v80 offset:1024\n\t" \
    "ds_read_b64_tr_b8 v[134:135], v81 offset:1024\n\t" \
    "ds_read_b64_tr_b8 v[136:137], v80 offset:8192\n\t" \
    "ds_read_b64_tr_b8 v[138:139], v81 offset:8192\n\t" \
    "ds_read_b64_tr_b8 v[140:141], v80 offset:9216\n\t" \
    "ds_read_b64_tr_b8 v[142:143], v81 offset:9216\n\t" \
    "s_set_gpr_idx_on %[t], gpr_idx(DST)\n\t" \
    "v_mov_b64 v[144:145], v[112:113]\n\t" \
    "v_mov_b64 v[146:147], v[114:115]\n\t" \
    "v_mov_b64 v[148:149], v[116:117]\n\t" \
    "v_mov_b64 v[150:151], v[118:119]\n\t" \
    "v_mov_b64 v[152:153], v[120:121]\n\t" \
    "v_mov_b64 v[154:155], v[122:123]\n\t" \
    "v_mov_b64 v[156:157], v[124:125]\n\t" \
    "v_mov_b64 v[158:159], v[126:127]\n\t" \
    "s_set_gpr_idx_off\n\t" \
    "s_add_u32 %[t], %[t], 16\n\t" \
    "s_cmp_eq_u32 %[t], 112\n\t" \
    "s_cselect_b32 %[t], 0, %[t]\n\t" \
    "s_add_u32 %[n], %[n], 32768\n\t" \
    "s_bitset0_b32 %[n], 17\n\t" \
    "v_add_u32 v80, %[n], %[w00]\n\t" \
    "v_add_u32 v81, %[n], %[w10]\n\t" \
    "s_waitcnt lgkmcnt(0)\n\t" \
    "s_barrier\n\t" \
    "ds_read_b64_tr_b8 v[112:113], v80 offset:0\n\t" \
    "ds_read_b64_tr_b8 v[114:115], v81 offset:0\n\t" \
    "ds_read_b64_tr_b8 v[116:117], v80 offset:1024\n\t" \
    "ds_read_b64_tr_b8 v[118:119], v81 offset:1024\n\t" \
    "ds_read_b64_tr_b8 v[120:121], v80 offset:8192\n\t" \
    "ds_read_b64_tr_b8 v[122:123], v81 offset:8192\n\t" \
    "ds_read_b64_tr_b8 v[124:125], v80 offset:9216\n\t" \
    "ds_read_b64_tr_b8 v[126:127], v81 offset:9216\n\t" \
    "s_set_gpr_idx_on %[t], gpr_idx(DST)\n\t" \
    "v_mov_b64 v[144:145], v[128:129]\n\t" \
    "v_mov_b64 v[146:147], v[130:131]\n\t" \
    "v_mov_b64 v[148:149], v[132:133]\n\t" \
    "v_mov_b64 v[150:151], v[134:135]\n\t" \
    "v_mov_b64 v[152:153], v[136:137]\n\t" \
    "v_mov_b64 v[154:155], v[138:139]\n\t" \
    "v_mov_b64 v[156:157], v[140:141]\n\t" \
    "v_mov_b64 v[158:159], v[142:143]\n\t" \
    "s_set_gpr_idx_off\n\t" \
    "s_add_u32 %[t], %[t], 16\n\t" \
    "s_cmp_eq_u32 %[t], 112\n\t" \
    "s_cselect_b32 %[t], 0, %[t]\n\t" \
    "s_add_u32 %[n], %[n], 32768\n\t" \
    "s_bitset0_b32 %[n], 17\n\t" \
    "v_add_u32 v80, %[n], %[w00]\n\t" \
    "v_add_u32 v81, %[n], %[w10]\n\t" \
    "s_sub_u32 %[t0], %[t0], 1\n\t" \
    "s_cmp_lg_u32 %[t0], 0\n\t" \
    "s_waitcnt lgkmcnt(0)\n\t" \
    "s_barrier\n\t" \
    "s_cbranch_scc1 3b\n\t" \
    "s_mov_b32 %[n], 7\n\t" \
      \
    "ds_read_b32 %[vd], %[z1] offset:59392\n\t" \
    "ds_read_b128 v[56:59], %[a01] offset:32768\n\t" \
    "ds_read_b128 v[60:63], %[a11] offset:32768\n\t" \
    "ds_read_b128 v[64:67], %[a01] offset:34816\n\t" \
    "ds_read_b128 v[68:71], %[a11] offset:34816\n\t" \
    "ds_read_b128 v[72:75], %[a01] offset:36864\n\t" \
    "ds_read_b128 v[76:79], %[a11] offset:36864\n\t" \
    "ds_read_b128 v[80:83], %[a01] offset:38912\n\t" \
    "ds_read_b128 v[84:87], %[a11] offset:38912\n\t" \
    "ds_read_b128 v[88:91], %[a01] offset:40960\n\t" \
    "ds_read_b128 v[92:95], %[a11] offset:40960\n\t" \
    "s_set_gpr_idx_on %[t], gpr_idx(DST)\n\t" \
    "v_mov_b64 v[144:145], v[112:113]\n\t" \
    "v_mov_b64 v[146:147], v[114:115]\n\t" \
    "v_mov_b64 v[148:149], v[116:117]\n\t" \
    "v_mov_b64 v[150:151], v[118:119]\n\t" \
    "v_mov_b64 v[152:153], v[120:121]\n\t" \
    "v_mov_b64 v[154:155], v[122:123]\n\t" \
    "v_mov_b64 v[156:157], v[124:125]\n\t" \
    "v_mov_b64 v[158:159], v[126:127]\n\t" \
    "s_set_gpr_idx_off\n\t" \
    "s_add_u32 %[t], %[t], 16\n\t" \
    "s_cmp_eq_u32 %[t], 112\n\t" \
    "s_cselect_b32 %[t], 0, %[t]\n\t" \
    "s_waitcnt lgkmcnt(10)\n\t" \
    "v_readfirstlane_b32 %[t], %[vd]\n\t" \
    "ds_read_b64_tr_b8 v[112:113], %[w01] offset:32768\n\t" \
    "ds_read_b64_tr_b8 v[120:121], %[w11] offset:32768\n\t" \
    "ds_read_b64_tr_b8 v[114:115], %[w01] offset:33792\n\t" \
    "ds_read_b64_tr_b8 v[122:123], %[w11] offset:33792\n\t" \
    "ds_read_b64_tr_b8 v[116:117], %[w01] offset:40960\n\t" \
    "ds_read_b64_tr_b8 v[124:125], %[w11] offset:40960\n\t" \
    "s_set_gpr_idx_on %[t], gpr_idx(SRC0)\n\t" \
    "v_mov_b64 v[118:119], v[144:145]\n\t" \
    "v_mov_b64 v[126:127], v[146:147]\n\t" \
    "s_set_gpr_idx_off\n\t" \
    "s_waitcnt lgkmcnt(0)\n\t" \
    "s_barrier\n\t" \
    "ds_read_b32 %[vd], %[z0] offset:26624\n\t" \
    "1:\n\t" \
    "v_mfma_f32_16x16x128_f8f6f4 v[16:19], v[56:63], v[112:119], v[16:19]\n\t" \
    "ds_read_b64_tr_b8 v[128:129], %[w00] offset:0\n\t" \
    "ds_read_b128 v[96:99], %[a00] offset:6144\n\t" \
    "ds_read_b128 v[100:103], %[a10] offset:6144\n\t" \
    "v_mfma_f32_16x16x128_f8f6f4 v[20:23], v[56:63], v[120:127], v[20:23]\n\t" \
    "ds_read_b64_tr_b8 v[136:137], %[w10] offset:0\n\t" \
    "ds_read_b128 v[56:59], %[a00] offset:0\n\t" \
    "ds_read_b128 v[60:63], %[a10] offset:0\n\t" \
    "v_mfma_f32_16x16x128_f8f6f4 v[24:27], v[64:71], v[112:119], v[24:27]\n\t" \
    "ds_read_b64_tr_b8 v[130:131], %[w00] offset:1024\n\t" \
    "ds_read_b128 v[104:107], %[a00] offset:8192\n\t" \
    "ds_read_b128 v[108:111], %[a10] offset:8192\n\t" \
    "v_mfma_f32_16x16x128_f8f6f4 v[28:31], v[64:71], v[120:127], v[28:31]\n\t" \
    "ds_read_b64_tr_b8 v[138:139], %[w10] offset:1024\n\t" \
    "ds_read_b128 v[64:67], %[a00] offset:2048\n\t" \
    "ds_read_b128 v[68:71], %[a10] offset:2048\n\t" \
    "v_mfma_f32_16x16x128_f8f6f4 v[32:35], v[72:79], v[112:119], v[32:35]\n\t" \
    "s_waitcnt lgkmcnt(12)\n\t" \
    "v_readfirstlane_b32 %[t], %[vd]\n\t" \
    "ds_read_b64_tr_b8 v[132:133], %[w00] offset:8192\n\t" \
    "ds_read_b64_tr_b8 v[140:141], %[w10] offset:8192\n\t" \
    "s_set_gpr_idx_on %[t], gpr_idx(SRC0)\n\t" \
    "v_mov_b64 v[134:135], v[144:145]\n\t" \
    "v_mov_b64 v[142:143], v[146:147]\n\t" \
    "s_set_gpr_idx_off\n\t" \
    "v_mfma_f32_16x16x128_f8f6f4 v[36:39], v[72:79], v[120:127], v[36:39]\n\t" \
    "ds_read_b128 v[72:75], %[a00] offset:4096\n\t" \
    "ds_read_b128 v[76:79], %[a10] offset:4096\n\t" \
    "v_mfma_f32_16x16x128_f8f6f4 v[40:43], v[80:87], v[112:119], v[40:43]\n\t" \
    "v_mfma_f32_16x16x128_f8f6f4 v[44:47], v[80:87], v[120:127], v[44:47]\n\t" \
    "v_mfma_f32_16x16x128_f8f6f4 v[48:51], v[88:95], v[112:119], v[48:51]\n\t" \
    "s_waitcnt lgkmcnt(0)\n\t" \
    "s_barrier\n\t" \
    "ds_read_b32 %[vd], %[z0] offset:59392\n\t" \
    "v_mfma_f32_16x16x128_f8f6f4 v[52:55], v[88:95], v[120:127], v[52:55]\n\t" \
    "v_mfma_f32_16x16x128_f8f6f4 v[16:19], v[56:63], v[128:135], v[16:19]\n\t" \
    "ds_read_b64_tr_b8 v[112:113], %[w00] offset:32768\n\t" \
    "ds_read_b128 v[80:83], %[a00] offset:38912\n\t" \
    "ds_read_b128 v[84:87], %[a10] offset:38912\n\t" \
    "v_mfma_f32_16x16x128_f8f6f4 v[20:23], v[56:63], v[136:143], v[20:23]\n\t" \
    "ds_read_b64_tr_b8 v[120:121], %[w10] offset:32768\n\t" \
    "ds_read_b128 v[56:59], %[a00] offset:32768\n\t" \
    "ds_read_b128 v[60:63], %[a10] offset:32768\n\t" \
    "v_mfma_f32_16x16x128_f8f6f4 v[24:27], v[64:71], v[128:135], v[24:27]\n\t" \
    "ds_read_b64_tr_b8 v[114:115], %[w00] offset:33792\n\t" \
    "ds_read_b128 v[88:91], %[a00] offset:40960\n\t" \
    "ds_read_b128 v[92:95], %[a10] offset:40960\n\t" \
    "v_mfma_f32_16x16x128_f8f6f4 v[28:31], v[64:71], v[136:143], v[28:31]\n\t" \
    "ds_read_b64_tr_b8 v[122:123], %[w10] offset:33792\n\t" \
    "ds_read_b128 v[64:67], %[a00] offset:34816\n\t" \
    "ds_read_b128 v[68:71], %[a10] offset:34816\n\t" \
    "v_mfma_f32_16x16x128_f8f6f4 v[32:35], v[72:79], v[128:135], v[32:35]\n\t" \
    "s_waitcnt lgkmcnt(12)\n\t" \
    "v_readfirstlane_b32 %[t], %[vd]\n\t" \
    "ds_read_b64_tr_b8 v[116:117], %[w00] offset:40960\n\t" \
    "ds_read_b64_tr_b8 v[124:125], %[w10] offset:40960\n\t" \
    "s_set_gpr_idx_on %[t], gpr_idx(SRC0)\n\t" \
    "v_mov_b64 v[118:119], v[144:145]\n\t" \
    "v_mov_b64 v[126:127], v[146:147]\n\t" \
    "s_set_gpr_idx_off\n\t" \
    "v_mfma_f32_16x16x128_f8f6f4 v[36:39], v[72:79], v[136:143], v[36:39]\n\t" \
    "ds_read_b128 v[72:75], %[a00] offset:36864\n\t" \
    "ds_read_b128 v[76:79], %[a10] offset:36864\n\t" \
    "v_mfma_f32_16x16x128_f8f6f4 v[40:43], v[96:103], v[128:135], v[40:43]\n\t" \
    "v_mfma_f32_16x16x128_f8f6f4 v[44:47], v[96:103], v[136:143], v[44:47]\n\t" \
    "v_mfma_f32_16x16x128_f8f6f4 v[48:51], v[104:111], v[128:135], v[48:51]\n\t" \
    "s_waitcnt lgkmcnt(0)\n\t" \
    "s_barrier\n\t" \
    "ds_read_b32 %[vd], %[z1] offset:26624\n\t" \
    "v_mfma_f32_16x16x128_f8f6f4 v[52:55], v[104:111], v[136:143], v[52:55]\n\t" \
    "v_mfma_f32_16x16x128_f8f6f4 v[16:19], v[56:63], v[112:119], v[16:19]\n\t" \
    "ds_read_b64_tr_b8 v[128:129], %[w01] offset:0\n\t" \
    "ds_read_b128 v[96:99], %[a01] offset:6144\n\t" \
    "ds_read_b128 v[100:103], %[a11] offset:6144\n\t" \
    "v_mfma_f32_16x16x128_f8f6f4 v[20:23], v[56:63], v[120:127], v[20:23]\n\t" \
    "ds_read_b64_tr_b8 v[136:137], %[w11] offset:0\n\t" \
    "ds_read_b128 v[56:59], %[a01] offset:0\n\t" \
    "ds_read_b128 v[60:63], %[a11] offset:0\n\t" \
    "v_mfma_f32_16x16x128_f8f6f4 v[24:27], v[64:71], v[112:119], v[24:27]\n\t" \
    "ds_read_b64_tr_b8 v[130:131], %[w01] offset:1024\n\t" \
    "ds_read_b128 v[104:107], %[a01] offset:8192\n\t" \
    "ds_read_b128 v[108:111], %[a11] offset:8192\n\t" \
    "v_mfma_f32_16x16x128_f8f6f4 v[28:31], v[64:71], v[120:127], v[28:31]\n\t" \
    "ds_read_b64_tr_b8 v[138:139], %[w11] offset:1024\n\t" \
    "ds_read_b128 v[64:67], %[a01] offset:2048\n\t" \
    "ds_read_b128 v[68:71], %[a11] offset:2048\n\t" \
    "v_mfma_f32_16x16x128_f8f6f4 v[32:35], v[72:79], v[112:119], v[32:35]\n\t" \
    "s_waitcnt lgkmcnt(12)\n\t" \
    "v_readfirstlane_b32 %[t], %[vd]\n\t" \
    "ds_read_b64_tr_b8 v[132:133], %[w01] offset:8192\n\t" \
    "ds_read_b64_tr_b8 v[140:141], %[w11] offset:8192\n\t" \
    "s_set_gpr_idx_on %[t], gpr_idx(SRC0)\n\t" \
    "v_mov_b64 v[134:135], v[144:145]\n\t" \
    "v_mov_b64 v[142:143], v[146:147]\n\t" \
    "s_set_gpr_idx_off\n\t" \
    "v_mfma_f32_16x16x128_f8f6f4 v[36:39], v[72:79], v[120:127], v[36:39]\n\t" \
    "ds_read_b128 v[72:75], %[a01] offset:4096\n\t" \
    "ds_read_b128 v[76:79], %[a11] offset:4096\n\t" \
    "v_mfma_f32_16x16x128_f8f6f4 v[40:43], v[80:87], v[112:119], v[40:43]\n\t" \
    "v_mfma_f32_16x16x128_f8f6f4 v[44:47], v[80:87], v[120:127], v[44:47]\n\t" \
    "v_mfma_f32_16x16x128_f8f6f4 v[48:51], v[88:95], v[112:119], v[48:51]\n\t" \
    "s_sub_u32 %[n], %[n], 1\n\t" \
    "s_cmp_eq_u32 %[n], 0\n\t" \
    "s_waitcnt lgkmcnt(0)\n\t" \
    "s_cbranch_scc1 2f\n\t" \
    "s_barrier\n\t" \
    "ds_read_b32 %[vd], %[z1] offset:59392\n\t" \
    "v_mfma_f32_16x16x128_f8f6f4 v[52:55], v[88:95], v[120:127], v[52:55]\n\t" \
    "v_mfma_f32_16x16x128_f8f6f4 v[16:19], v[56:63], v[128:135], v[16:19]\n\t" \
    "ds_read_b64_tr_b8 v[112:113], %[w01] offset:32768\n\t" \
    "ds_read_b128 v[80:83], %[a01] offset:38912\n\t" \
    "ds_read_b128 v[84:87], %[a11] offset:38912\n\t" \
    "v_mfma_f32_16x16x128_f8f6f4 v[20:23], v[56:63], v[136:143], v[20:23]\n\t" \
    "ds_read_b64_tr_b8 v[120:121], %[w11] offset:32768\n\t" \
    "ds_read_b128 v[56:59], %[a01] offset:32768\n\t" \
    "ds_read_b128 v[60:63], %[a11] offset:32768\n\t" \
    "v_mfma_f32_16x16x128_f8f6f4 v[24:27], v[64:71], v[128:135], v[24:27]\n\t" \
    "ds_read_b64_tr_b8 v[114:115], %[w01] offset:33792\n\t" \
    "ds_read_b128 v[88:91], %[a01] offset:40960\n\t" \
    "ds_read_b128 v[92:95], %[a11] offset:40960\n\t" \
    "v_mfma_f32_16x16x128_f8f6f4 v[28:31], v[64:71], v[136:143], v[28:31]\n\t" \
    "ds_read_b64_tr_b8 v[122:123], %[w11] offset:33792\n\t" \
    "ds_read_b128 v[64:67], %[a01] offset:34816\n\t" \
    "ds_read_b128 v[68:71], %[a11] offset:34816\n\t" \
    "v_mfma_f32_16x16x128_f8f6f4 v[32:35], v[72:79], v[128:135], v[32:35]\n\t" \
    "s_waitcnt lgkmcnt(12)\n\t" \
    "v_readfirstlane_b32 %[t], %[vd]\n\t" \
    "ds_read_b64_tr_b8 v[116:117], %[w01] offset:40960\n\t" \
    "ds_read_b64_tr_b8 v[124:125], %[w11] offset:40960\n\t" \
    "s_set_gpr_idx_on %[t], gpr_idx(SRC0)\n\t" \
    "v_mov_b64 v[118:119], v[144:145]\n\t" \
    "v_mov_b64 v[126:127], v[146:147]\n\t" \
    "s_set_gpr_idx_off\n\t" \
    "v_mfma_f32_16x16x128_f8f6f4 v[36:39], v[72:79], v[136:143], v[36:39]\n\t" \
    "ds_read_b128 v[72:75], %[a01] offset:36864\n\t" \
    "ds_read_b128 v[76:79], %[a11] offset:36864\n\t" \
    "v_mfma_f32_16x16x128_f8f6f4 v[40:43], v[96:103], v[128:135], v[40:43]\n\t" \
    "v_mfma_f32_16x16x128_f8f6f4 v[44:47], v[96:103], v[136:143], v[44:47]\n\t" \
    "v_mfma_f32_16x16x128_f8f6f4 v[48:51], v[104:111], v[128:135], v[48:51]\n\t" \
    "s_waitcnt lgkmcnt(0)\n\t" \
    "s_barrier\n\t" \
    "ds_read_b32 %[vd], %[z0] offset:26624\n\t" \
    "v_mfma_f32_16x16x128_f8f6f4 v[52:55], v[104:111], v[136:143], v[52:55]\n\t" \
    "s_branch 1b\n\t" \
      \
    "2:\n\t" \
    "ds_read_b32 %[vd], %[z1] offset:65520\n\t" \
    "v_mfma_f32_16x16x128_f8f6f4 v[52:55], v[88:95], v[120:127], v[52:55]\n\t" \
    "v_mfma_f32_16x16x128_f8f6f4 v[16:19], v[56:63], v[128:135], v[16:19]\n\t" \
    "v_mfma_f32_16x16x128_f8f6f4 v[20:23], v[56:63], v[136:143], v[20:23]\n\t" \
    "s_waitcnt lgkmcnt(0)\n\t" \
    "v_cmp_ne_u32 vcc, 0, %[vd]\n\t" \
    "v_mov_b32 %[vd], 0x7fc00000\n\t" \
    "s_nop 0\n\t" \
    "v_cndmask_b32 %[sc], %[sc], %[vd], vcc\n\t" \
    "v_mfma_f32_16x16x128_f8f6f4 v[24:27], v[64:71], v[128:135], v[24:27]\n\t" \
    "v_mfma_f32_16x16x128_f8f6f4 v[28:31], v[64:71], v[136:143], v[28:31]\n\t" \
    "s_nop 4\n\t" \
    "v_mul_f32 v16, v16, %[sc]\n\t" \
    "v_mul_f32 v17, v17, %[sc]\n\t" \
    "v_mul_f32 v18, v18, %[sc]\n\t" \
    "v_mul_f32 v19, v19, %[sc]\n\t" \
    "v_mul_f32 v20, v20, %[sc]\n\t" \
    "v_mul_f32 v21, v21, %[sc]\n\t" \
    "v_mul_f32 v22, v22, %[sc]\n\t" \
    "v_mul_f32 v23, v23, %[sc]\n\t" \
    "v_maximum3_f32 v16, v16, 0, 0\n\t" \
    "v_maximum3_f32 v20, v20, 0, 0\n\t" \
    "v_maximum3_f32 v17, v17, 0, 0\n\t" \
    "v_maximum3_f32 v21, v21, 0, 0\n\t" \
    "v_maximum3_f32 v18, v18, 0, 0\n\t" \
    "v_maximum3_f32 v22, v22, 0, 0\n\t" \
    "v_maximum3_f32 v19, v19, 0, 0\n\t" \
    "v_maximum3_f32 v23, v23, 0, 0\n\t" \
    "v_permlane16_swap_b32 v16, v20\n\t" \
    "v_permlane16_swap_b32 v17, v21\n\t" \
    "v_permlane16_swap_b32 v18, v22\n\t" \
    "v_permlane16_swap_b32 v19, v23\n\t" \
    "buffer_store_dword v16, %[vo], %[ro], 0 offen sc0 sc1\n\t" \
    "buffer_store_dword v17, %[vo], %[ro], s70 offen sc0 sc1\n\t" \
    "buffer_store_dword v18, %[vo], %[ro], s71 offen sc0 sc1\n\t" \
    "buffer_store_dword v19, %[vo], %[ro], s72 offen sc0 sc1\n\t" \
    "buffer_store_dword v20, %[vo], %[ro], s73 offen sc0 sc1\n\t" \
    "buffer_store_dword v21, %[vo], %[ro], s74 offen sc0 sc1\n\t" \
    "buffer_store_dword v22, %[vo], %[ro], s75 offen sc0 sc1\n\t" \
    "buffer_store_dword v23, %[vo], %[ro], s76 offen sc0 sc1\n\t" \
    "v_add_u32 %[vo], s77, %[vo]\n\t" \
    "v_mfma_f32_16x16x128_f8f6f4 v[32:35], v[72:79], v[128:135], v[32:35]\n\t" \
    "v_mfma_f32_16x16x128_f8f6f4 v[36:39], v[72:79], v[136:143], v[36:39]\n\t" \
    "v_mul_f32 v24, v24, %[sc]\n\t" \
    "v_mul_f32 v25, v25, %[sc]\n\t" \
    "v_mul_f32 v26, v26, %[sc]\n\t" \
    "v_mul_f32 v27, v27, %[sc]\n\t" \
    "v_mul_f32 v28, v28, %[sc]\n\t" \
    "v_mul_f32 v29, v29, %[sc]\n\t" \
    "v_mul_f32 v30, v30, %[sc]\n\t" \
    "v_mul_f32 v31, v31, %[sc]\n\t" \
    "v_maximum3_f32 v24, v24, 0, 0\n\t" \
    "v_maximum3_f32 v28, v28, 0, 0\n\t" \
    "v_maximum3_f32 v25, v25, 0, 0\n\t" \
    "v_maximum3_f32 v29, v29, 0, 0\n\t" \
    "v_maximum3_f32 v26, v26, 0, 0\n\t" \
    "v_maximum3_f32 v30, v30, 0, 0\n\t" \
    "v_maximum3_f32 v27, v27, 0, 0\n\t" \
    "v_maximum3_f32 v31, v31, 0, 0\n\t" \
    "v_permlane16_swap_b32 v24, v28\n\t" \
    "v_permlane16_swap_b32 v25, v29\n\t" \
    "v_permlane16_swap_b32 v26, v30\n\t" \
    "v_permlane16_swap_b32 v27, v31\n\t" \
    "buffer_store_dword v24, %[vo], %[ro], 0 offen sc0 sc1\n\t" \
    "buffer_store_dword v25, %[vo], %[ro], s70 offen sc0 sc1\n\t" \
    "buffer_store_dword v26, %[vo], %[ro], s71 offen sc0 sc1\n\t" \
    "buffer_store_dword v27, %[vo], %[ro], s72 offen sc0 sc1\n\t" \
    "buffer_store_dword v28, %[vo], %[ro], s73 offen sc0 sc1\n\t" \
    "buffer_store_dword v29, %[vo], %[ro], s74 offen sc0 sc1\n\t" \
    "buffer_store_dword v30, %[vo], %[ro], s75 offen sc0 sc1\n\t" \
    "buffer_store_dword v31, %[vo], %[ro], s76 offen sc0 sc1\n\t" \
    "v_add_u32 %[vo], s77, %[vo]\n\t" \
    "v_mfma_f32_16x16x128_f8f6f4 v[40:43], v[96:103], v[128:135], v[40:43]\n\t" \
    "v_mfma_f32_16x16x128_f8f6f4 v[44:47], v[96:103], v[136:143], v[44:47]\n\t" \
    "v_mul_f32 v32, v32, %[sc]\n\t" \
    "v_mul_f32 v33, v33, %[sc]\n\t" \
    "v_mul_f32 v34, v34, %[sc]\n\t" \
    "v_mul_f32 v35, v35, %[sc]\n\t" \
    "v_mul_f32 v36, v36, %[sc]\n\t" \
    "v_mul_f32 v37, v37, %[sc]\n\t" \
    "v_mul_f32 v38, v38, %[sc]\n\t" \
    "v_mul_f32 v39, v39, %[sc]\n\t" \
    "v_maximum3_f32 v32, v32, 0, 0\n\t" \
    "v_maximum3_f32 v36, v36, 0, 0\n\t" \
    "v_maximum3_f32 v33, v33, 0, 0\n\t" \
    "v_maximum3_f32 v37, v37, 0, 0\n\t" \
    "v_maximum3_f32 v34, v34, 0, 0\n\t" \
    "v_maximum3_f32 v38, v38, 0, 0\n\t" \
    "v_maximum3_f32 v35, v35, 0, 0\n\t" \
    "v_maximum3_f32 v39, v39, 0, 0\n\t" \
    "v_permlane16_swap_b32 v32, v36\n\t" \
    "v_permlane16_swap_b32 v33, v37\n\t" \
    "v_permlane16_swap_b32 v34, v38\n\t" \
    "v_permlane16_swap_b32 v35, v39\n\t" \
    "buffer_store_dword v32, %[vo], %[ro], 0 offen sc0 sc1\n\t" \
    "buffer_store_dword v33, %[vo], %[ro], s70 offen sc0 sc1\n\t" \
    "buffer_store_dword v34, %[vo], %[ro], s71 offen sc0 sc1\n\t" \
    "buffer_store_dword v35, %[vo], %[ro], s72 offen sc0 sc1\n\t" \
    "buffer_store_dword v36, %[vo], %[ro], s73 offen sc0 sc1\n\t" \
    "buffer_store_dword v37, %[vo], %[ro], s74 offen sc0 sc1\n\t" \
    "buffer_store_dword v38, %[vo], %[ro], s75 offen sc0 sc1\n\t" \
    "buffer_store_dword v39, %[vo], %[ro], s76 offen sc0 sc1\n\t" \
    "v_add_u32 %[vo], s77, %[vo]\n\t" \
    "v_mfma_f32_16x16x128_f8f6f4 v[48:51], v[104:111], v[128:135], v[48:51]\n\t" \
    "v_mfma_f32_16x16x128_f8f6f4 v[52:55], v[104:111], v[136:143], v[52:55]\n\t" \
    "v_mul_f32 v40, v40, %[sc]\n\t" \
    "v_mul_f32 v41, v41, %[sc]\n\t" \
    "v_mul_f32 v42, v42, %[sc]\n\t" \
    "v_mul_f32 v43, v43, %[sc]\n\t" \
    "v_mul_f32 v44, v44, %[sc]\n\t" \
    "v_mul_f32 v45, v45, %[sc]\n\t" \
    "v_mul_f32 v46, v46, %[sc]\n\t" \
    "v_mul_f32 v47, v47, %[sc]\n\t" \
    "v_maximum3_f32 v40, v40, 0, 0\n\t" \
    "v_maximum3_f32 v44, v44, 0, 0\n\t" \
    "v_maximum3_f32 v41, v41, 0, 0\n\t" \
    "v_maximum3_f32 v45, v45, 0, 0\n\t" \
    "v_maximum3_f32 v42, v42, 0, 0\n\t" \
    "v_maximum3_f32 v46, v46, 0, 0\n\t" \
    "v_maximum3_f32 v43, v43, 0, 0\n\t" \
    "v_maximum3_f32 v47, v47, 0, 0\n\t" \
    "v_permlane16_swap_b32 v40, v44\n\t" \
    "v_permlane16_swap_b32 v41, v45\n\t" \
    "v_permlane16_swap_b32 v42, v46\n\t" \
    "v_permlane16_swap_b32 v43, v47\n\t" \
    "buffer_store_dword v40, %[vo], %[ro], 0 offen sc0 sc1\n\t" \
    "buffer_store_dword v41, %[vo], %[ro], s70 offen sc0 sc1\n\t" \
    "buffer_store_dword v42, %[vo], %[ro], s71 offen sc0 sc1\n\t" \
    "buffer_store_dword v43, %[vo], %[ro], s72 offen sc0 sc1\n\t" \
    "buffer_store_dword v44, %[vo], %[ro], s73 offen sc0 sc1\n\t" \
    "buffer_store_dword v45, %[vo], %[ro], s74 offen sc0 sc1\n\t" \
    "buffer_store_dword v46, %[vo], %[ro], s75 offen sc0 sc1\n\t" \
    "buffer_store_dword v47, %[vo], %[ro], s76 offen sc0 sc1\n\t" \
    "v_add_u32 %[vo], s77, %[vo]\n\t" \
    "v_mul_f32 v48, v48, %[sc]\n\t" \
    "v_mul_f32 v49, v49, %[sc]\n\t" \
    "v_mul_f32 v50, v50, %[sc]\n\t" \
    "v_mul_f32 v51, v51, %[sc]\n\t" \
    "v_mul_f32 v52, v52, %[sc]\n\t" \
    "v_mul_f32 v53, v53, %[sc]\n\t" \
    "v_mul_f32 v54, v54, %[sc]\n\t" \
    "v_mul_f32 v55, v55, %[sc]\n\t" \
    "v_maximum3_f32 v48, v48, 0, 0\n\t" \
    "v_maximum3_f32 v52, v52, 0, 0\n\t" \
    "v_maximum3_f32 v49, v49, 0, 0\n\t" \
    "v_maximum3_f32 v53, v53, 0, 0\n\t" \
    "v_maximum3_f32 v50, v50, 0, 0\n\t" \
    "v_maximum3_f32 v54, v54, 0, 0\n\t" \
    "v_maximum3_f32 v51, v51, 0, 0\n\t" \
    "v_maximum3_f32 v55, v55, 0, 0\n\t" \
    "v_permlane16_swap_b32 v48, v52\n\t" \
    "v_permlane16_swap_b32 v49, v53\n\t" \
    "v_permlane16_swap_b32 v50, v54\n\t" \
    "v_permlane16_swap_b32 v51, v55\n\t" \
    "buffer_store_dword v48, %[vo], %[ro], 0 offen sc0 sc1\n\t" \
    "buffer_store_dword v49, %[vo], %[ro], s70 offen sc0 sc1\n\t" \
    "buffer_store_dword v50, %[vo], %[ro], s71 offen sc0 sc1\n\t" \
    "buffer_store_dword v51, %[vo], %[ro], s72 offen sc0 sc1\n\t" \
    "buffer_store_dword v52, %[vo], %[ro], s73 offen sc0 sc1\n\t" \
    "buffer_store_dword v53, %[vo], %[ro], s74 offen sc0 sc1\n\t" \
    "buffer_store_dword v54, %[vo], %[ro], s75 offen sc0 sc1\n\t" \
    "buffer_store_dword v55, %[vo], %[ro], s76 offen sc0 sc1\n\t"

#define CONS4_ASM \
      \
    "s_barrier\n\t" \
    "ds_read_b64_tr_b8 v[112:113], %[w00] offset:0\n\t" \
    "ds_read_b64_tr_b8 v[120:121], %[w10] offset:0\n\t" \
    "ds_read_b64_tr_b8 v[114:115], %[w00] offset:1024\n\t" \
    "ds_read_b64_tr_b8 v[122:123], %[w10] offset:1024\n\t" \
    "ds_read_b64_tr_b8 v[116:117], %[w00] offset:8192\n\t" \
    "ds_read_b64_tr_b8 v[124:125], %[w10] offset:8192\n\t" \
    "ds_read_b64_tr_b8 v[118:119], %[w00] offset:9216\n\t" \
    "ds_read_b64_tr_b8 v[126:127], %[w10] offset:9216\n\t" \
    "ds_read_b128 v[56:59], %[a00] offset:0\n\t" \
    "ds_read_b128 v[60:63], %[a10] offset:0\n\t" \
    "ds_read_b128 v[64:67], %[a00] offset:2048\n\t" \
    "ds_read_b128 v[68:71], %[a10] offset:2048\n\t" \
    "ds_read_b128 v[80:83], %[a00] offset:4096\n\t" \
    "ds_read_b128 v[84:87], %[a10] offset:4096\n\t" \
    "ds_read_b128 v[88:91], %[a00] offset:6144\n\t" \
    "ds_read_b128 v[92:95], %[a10] offset:6144\n\t" \
    "s_waitcnt lgkmcnt(0)\n\t" \
    "s_barrier\n\t" \
    "v_mfma_f32_16x16x128_f8f6f4 v[16:19], v[56:63], v[112:119], v[16:19]\n\t" \
    "ds_read_b64_tr_b8 v[128:129], %[w00] offset:32768\n\t" \
    "ds_read_b128 v[96:99], %[a00] offset:36864\n\t" \
    "ds_read_b128 v[100:103], %[a10] offset:36864\n\t" \
    "v_mfma_f32_16x16x128_f8f6f4 v[20:23], v[56:63], v[120:127], v[20:23]\n\t" \
    "ds_read_b64_tr_b8 v[136:137], %[w10] offset:32768\n\t" \
    "ds_read_b128 v[56:59], %[a00] offset:32768\n\t" \
    "ds_read_b128 v[60:63], %[a10] offset:32768\n\t" \
    "v_mfma_f32_16x16x128_f8f6f4 v[24:27], v[64:71], v[112:119], v[24:27]\n\t" \
    "ds_read_b64_tr_b8 v[130:131], %[w00] offset:33792\n\t" \
    "ds_read_b128 v[104:107], %[a00] offset:38912\n\t" \
    "ds_read_b128 v[108:111], %[a10] offset:38912\n\t" \
    "v_mfma_f32_16x16x128_f8f6f4 v[28:31], v[64:71], v[120:127], v[28:31]\n\t" \
    "ds_read_b64_tr_b8 v[138:139], %[w10] offset:33792\n\t" \
    "ds_read_b128 v[64:67], %[a00] offset:34816\n\t" \
    "ds_read_b128 v[68:71], %[a10] offset:34816\n\t" \
    "v_mfma_f32_16x16x128_f8f6f4 v[32:35], v[80:87], v[112:119], v[32:35]\n\t" \
    "ds_read_b64_tr_b8 v[132:133], %[w00] offset:40960\n\t" \
    "ds_read_b64_tr_b8 v[140:141], %[w10] offset:40960\n\t" \
    "ds_read_b64_tr_b8 v[134:135], %[w00] offset:41984\n\t" \
    "v_mfma_f32_16x16x128_f8f6f4 v[36:39], v[80:87], v[120:127], v[36:39]\n\t" \
    "ds_read_b64_tr_b8 v[142:143], %[w10] offset:41984\n\t" \
    "v_mfma_f32_16x16x128_f8f6f4 v[40:43], v[88:95], v[112:119], v[40:43]\n\t" \
    "s_waitcnt lgkmcnt(0)\n\t" \
    "s_barrier\n\t" \
    "v_mfma_f32_16x16x128_f8f6f4 v[44:47], v[88:95], v[120:127], v[44:47]\n\t" \
    "v_mfma_f32_16x16x128_f8f6f4 v[16:19], v[56:63], v[128:135], v[16:19]\n\t" \
    "ds_read_b64_tr_b8 v[112:113], %[w01] offset:0\n\t" \
    "ds_read_b128 v[80:83], %[a01] offset:4096\n\t" \
    "ds_read_b128 v[84:87], %[a11] offset:4096\n\t" \
    "v_mfma_f32_16x16x128_f8f6f4 v[20:23], v[56:63], v[136:143], v[20:23]\n\t" \
    "ds_read_b64_tr_b8 v[120:121], %[w11] offset:0\n\t" \
    "ds_read_b128 v[56:59], %[a01] offset:0\n\t" \
    "ds_read_b128 v[60:63], %[a11] offset:0\n\t" \
    "v_mfma_f32_16x16x128_f8f6f4 v[24:27], v[64:71], v[128:135], v[24:27]\n\t" \
    "ds_read_b64_tr_b8 v[114:115], %[w01] offset:1024\n\t" \
    "ds_read_b128 v[88:91], %[a01] offset:6144\n\t" \
    "ds_read_b128 v[92:95], %[a11] offset:6144\n\t" \
    "v_mfma_f32_16x16x128_f8f6f4 v[28:31], v[64:71], v[136:143], v[28:31]\n\t" \
    "ds_read_b64_tr_b8 v[122:123], %[w11] offset:1024\n\t" \
    "ds_read_b128 v[64:67], %[a01] offset:2048\n\t" \
    "ds_read_b128 v[68:71], %[a11] offset:2048\n\t" \
    "v_mfma_f32_16x16x128_f8f6f4 v[32:35], v[96:103], v[128:135], v[32:35]\n\t" \
    "ds_read_b64_tr_b8 v[116:117], %[w01] offset:8192\n\t" \
    "ds_read_b64_tr_b8 v[124:125], %[w11] offset:8192\n\t" \
    "ds_read_b64_tr_b8 v[118:119], %[w01] offset:9216\n\t" \
    "v_mfma_f32_16x16x128_f8f6f4 v[36:39], v[96:103], v[136:143], v[36:39]\n\t" \
    "ds_read_b64_tr_b8 v[126:127], %[w11] offset:9216\n\t" \
    "v_mfma_f32_16x16x128_f8f6f4 v[40:43], v[104:111], v[128:135], v[40:43]\n\t" \
    "s_waitcnt lgkmcnt(0)\n\t" \
    "s_barrier\n\t" \
    "v_mfma_f32_16x16x128_f8f6f4 v[44:47], v[104:111], v[136:143], v[44:47]\n\t" \
    "v_mfma_f32_16x16x128_f8f6f4 v[16:19], v[56:63], v[112:119], v[16:19]\n\t" \
    "ds_read_b64_tr_b8 v[128:129], %[w01] offset:32768\n\t" \
    "ds_read_b128 v[96:99], %[a01] offset:36864\n\t" \
    "ds_read_b128 v[100:103], %[a11] offset:36864\n\t" \
    "v_mfma_f32_16x16x128_f8f6f4 v[20:23], v[56:63], v[120:127], v[20:23]\n\t" \
    "ds_read_b64_tr_b8 v[136:137], %[w11] offset:32768\n\t" \
    "ds_read_b128 v[56:59], %[a01] offset:32768\n\t" \
    "ds_read_b128 v[60:63], %[a11] offset:32768\n\t" \
    "v_mfma_f32_16x16x128_f8f6f4 v[24:27], v[64:71], v[112:119], v[24:27]\n\t" \
    "ds_read_b64_tr_b8 v[130:131], %[w01] offset:33792\n\t" \
    "ds_read_b128 v[104:107], %[a01] offset:38912\n\t" \
    "ds_read_b128 v[108:111], %[a11] offset:38912\n\t" \
    "v_mfma_f32_16x16x128_f8f6f4 v[28:31], v[64:71], v[120:127], v[28:31]\n\t" \
    "ds_read_b64_tr_b8 v[138:139], %[w11] offset:33792\n\t" \
    "ds_read_b128 v[64:67], %[a01] offset:34816\n\t" \
    "ds_read_b128 v[68:71], %[a11] offset:34816\n\t" \
    "v_mfma_f32_16x16x128_f8f6f4 v[32:35], v[80:87], v[112:119], v[32:35]\n\t" \
    "ds_read_b64_tr_b8 v[132:133], %[w01] offset:40960\n\t" \
    "ds_read_b64_tr_b8 v[140:141], %[w11] offset:40960\n\t" \
    "ds_read_b64_tr_b8 v[134:135], %[w01] offset:41984\n\t" \
    "v_mfma_f32_16x16x128_f8f6f4 v[36:39], v[80:87], v[120:127], v[36:39]\n\t" \
    "ds_read_b64_tr_b8 v[142:143], %[w11] offset:41984\n\t" \
    "v_mfma_f32_16x16x128_f8f6f4 v[40:43], v[88:95], v[112:119], v[40:43]\n\t" \
    "s_waitcnt lgkmcnt(0)\n\t" \
    "s_barrier\n\t" \
    "v_mfma_f32_16x16x128_f8f6f4 v[44:47], v[88:95], v[120:127], v[44:47]\n\t" \
      \
    "s_mov_b32 %[n], 0\n\t" \
    "s_lshl_b32 %[t], %[rb], 4\n\t" \
    "s_sub_u32 %[t], 96, %[t]\n\t" \
    "s_mov_b32 %[t0], 3\n\t" \
    "s_add_u32 %[n], %[n], 32768\n\t" \
    "s_bitset0_b32 %[n], 17\n\t" \
    "v_add_u32 v80, %[n], %[w00]\n\t" \
    "v_add_u32 v81, %[n], %[w10]\n\t" \
    "v_mfma_f32_16x16x128_f8f6f4 v[16:19], v[56:63], v[128:135], v[16:19]\n\t" \
    "ds_read_b64_tr_b8 v[112:113], %[w00] offset:0\n\t" \
    "ds_read_b64_tr_b8 v[114:115], %[w10] offset:0\n\t" \
    "v_mfma_f32_16x16x128_f8f6f4 v[20:23], v[56:63], v[136:143], v[20:23]\n\t" \
    "ds_read_b64_tr_b8 v[116:117], %[w00] offset:1024\n\t" \
    "ds_read_b64_tr_b8 v[118:119], %[w10] offset:1024\n\t" \
    "v_mfma_f32_16x16x128_f8f6f4 v[24:27], v[64:71], v[128:135], v[24:27]\n\t" \
    "ds_read_b64_tr_b8 v[120:121], %[w00] offset:8192\n\t" \
    "ds_read_b64_tr_b8 v[122:123], %[w10] offset:8192\n\t" \
    "v_mfma_f32_16x16x128_f8f6f4 v[28:31], v[64:71], v[136:143], v[28:31]\n\t" \
    "ds_read_b64_tr_b8 v[124:125], %[w00] offset:9216\n\t" \
    "ds_read_b64_tr_b8 v[126:127], %[w10] offset:9216\n\t" \
    "v_mfma_f32_16x16x128_f8f6f4 v[32:35], v[96:103], v[128:135], v[32:35]\n\t" \
    "v_mfma_f32_16x16x128_f8f6f4 v[36:39], v[96:103], v[136:143], v[36:39]\n\t" \
    "v_mfma_f32_16x16x128_f8f6f4 v[40:43], v[104:111], v[128:135], v[40:43]\n\t" \
    "s_waitcnt lgkmcnt(0)\n\t" \
    "s_barrier\n\t" \
    "v_mfma_f32_16x16x128_f8f6f4 v[44:47], v[104:111], v[136:143], v[44:47]\n\t" \
    "3:\n\t" \
    "ds_read_b64_tr_b8 v[128:129], v80 offset:0\n\t" \
    "ds_read_b64_tr_b8 v[130:131], v81 offset:0\n\t" \
    "ds_read_b64_tr_b8 v[132:133], v80 offset:1024\n\t" \
    "ds_read_b64_tr_b8 v[134:135], v81 offset:1024\n\t" \
    "ds_read_b64_tr_b8 v[136:137], v80 offset:8192\n\t" \
    "ds_read_b64_tr_b8 v[138:139], v81 offset:8192\n\t" \
    "ds_read_b64_tr_b8 v[140:141], v80 offset:9216\n\t" \
    "ds_read_b64_tr_b8 v[142:143], v81 offset:9216\n\t" \
    "s_set_gpr_idx_on %[t], gpr_idx(DST)\n\t" \
    "v_mov_b64 v[144:145], v[112:113]\n\t" \
    "v_mov_b64 v[146:147], v[114:115]\n\t" \
    "v_mov_b64 v[148:149], v[116:117]\n\t" \
    "v_mov_b64 v[150:151], v[118:119]\n\t" \
    "v_mov_b64 v[152:153], v[120:121]\n\t" \
    "v_mov_b64 v[154:155], v[122:123]\n\t" \
    "v_mov_b64 v[156:157], v[124:125]\n\t" \
    "v_mov_b64 v[158:159], v[126:127]\n\t" \
    "s_set_gpr_idx_off\n\t" \
    "s_add_u32 %[t], %[t], 16\n\t" \
    "s_cmp_eq_u32 %[t], 112\n\t" \
    "s_cselect_b32 %[t], 0, %[t]\n\t" \
    "s_add_u32 %[n], %[n], 32768\n\t" \
    "s_bitset0_b32 %[n], 17\n\t" \
    "v_add_u32 v80, %[n], %[w00]\n\t" \
    "v_add_u32 v81, %[n], %[w10]\n\t" \
    "s_waitcnt lgkmcnt(0)\n\t" \
    "s_barrier\n\t" \
    "ds_read_b64_tr_b8 v[112:113], v80 offset:0\n\t" \
    "ds_read_b64_tr_b8 v[114:115], v81 offset:0\n\t" \
    "ds_read_b64_tr_b8 v[116:117], v80 offset:1024\n\t" \
    "ds_read_b64_tr_b8 v[118:119], v81 offset:1024\n\t" \
    "ds_read_b64_tr_b8 v[120:121], v80 offset:8192\n\t" \
    "ds_read_b64_tr_b8 v[122:123], v81 offset:8192\n\t" \
    "ds_read_b64_tr_b8 v[124:125], v80 offset:9216\n\t" \
    "ds_read_b64_tr_b8 v[126:127], v81 offset:9216\n\t" \
    "s_set_gpr_idx_on %[t], gpr_idx(DST)\n\t" \
    "v_mov_b64 v[144:145], v[128:129]\n\t" \
    "v_mov_b64 v[146:147], v[130:131]\n\t" \
    "v_mov_b64 v[148:149], v[132:133]\n\t" \
    "v_mov_b64 v[150:151], v[134:135]\n\t" \
    "v_mov_b64 v[152:153], v[136:137]\n\t" \
    "v_mov_b64 v[154:155], v[138:139]\n\t" \
    "v_mov_b64 v[156:157], v[140:141]\n\t" \
    "v_mov_b64 v[158:159], v[142:143]\n\t" \
    "s_set_gpr_idx_off\n\t" \
    "s_add_u32 %[t], %[t], 16\n\t" \
    "s_cmp_eq_u32 %[t], 112\n\t" \
    "s_cselect_b32 %[t], 0, %[t]\n\t" \
    "s_add_u32 %[n], %[n], 32768\n\t" \
    "s_bitset0_b32 %[n], 17\n\t" \
    "v_add_u32 v80, %[n], %[w00]\n\t" \
    "v_add_u32 v81, %[n], %[w10]\n\t" \
    "s_sub_u32 %[t0], %[t0], 1\n\t" \
    "s_cmp_lg_u32 %[t0], 0\n\t" \
    "s_waitcnt lgkmcnt(0)\n\t" \
    "s_barrier\n\t" \
    "s_cbranch_scc1 3b\n\t" \
    "s_mov_b32 %[n], 7\n\t" \
      \
    "ds_read_b32 %[vd], %[z1] offset:59392\n\t" \
    "ds_read_b128 v[56:59], %[a01] offset:32768\n\t" \
    "ds_read_b128 v[60:63], %[a11] offset:32768\n\t" \
    "ds_read_b128 v[64:67], %[a01] offset:34816\n\t" \
    "ds_read_b128 v[68:71], %[a11] offset:34816\n\t" \
    "ds_read_b128 v[80:83], %[a01] offset:36864\n\t" \
    "ds_read_b128 v[84:87], %[a11] offset:36864\n\t" \
    "ds_read_b128 v[88:91], %[a01] offset:38912\n\t" \
    "ds_read_b128 v[92:95], %[a11] offset:38912\n\t" \
    "s_set_gpr_idx_on %[t], gpr_idx(DST)\n\t" \
    "v_mov_b64 v[144:145], v[112:113]\n\t" \
    "v_mov_b64 v[146:147], v[114:115]\n\t" \
    "v_mov_b64 v[148:149], v[116:117]\n\t" \
    "v_mov_b64 v[150:151], v[118:119]\n\t" \
    "v_mov_b64 v[152:153], v[120:121]\n\t" \
    "v_mov_b64 v[154:155], v[122:123]\n\t" \
    "v_mov_b64 v[156:157], v[124:125]\n\t" \
    "v_mov_b64 v[158:159], v[126:127]\n\t" \
    "s_set_gpr_idx_off\n\t" \
    "s_add_u32 %[t], %[t], 16\n\t" \
    "s_cmp_eq_u32 %[t], 112\n\t" \
    "s_cselect_b32 %[t], 0, %[t]\n\t" \
    "s_waitcnt lgkmcnt(8)\n\t" \
    "v_readfirstlane_b32 %[t], %[vd]\n\t" \
    "ds_read_b64_tr_b8 v[112:113], %[w01] offset:32768\n\t" \
    "ds_read_b64_tr_b8 v[120:121], %[w11] offset:32768\n\t" \
    "ds_read_b64_tr_b8 v[114:115], %[w01] offset:33792\n\t" \
    "ds_read_b64_tr_b8 v[122:123], %[w11] offset:33792\n\t" \
    "ds_read_b64_tr_b8 v[116:117], %[w01] offset:40960\n\t" \
    "ds_read_b64_tr_b8 v[124:125], %[w11] offset:40960\n\t" \
    "s_set_gpr_idx_on %[t], gpr_idx(SRC0)\n\t" \
    "v_mov_b64 v[118:119], v[144:145]\n\t" \
    "v_mov_b64 v[126:127], v[146:147]\n\t" \
    "s_set_gpr_idx_off\n\t" \
    "s_waitcnt lgkmcnt(0)\n\t" \
    "s_barrier\n\t" \
    "ds_read_b32 %[vd], %[z0] offset:26624\n\t" \
    "1:\n\t" \
    "v_mfma_f32_16x16x128_f8f6f4 v[16:19], v[56:63], v[112:119], v[16:19]\n\t" \
    "ds_read_b64_tr_b8 v[128:129], %[w00] offset:0\n\t" \
    "ds_read_b128 v[96:99], %[a00] offset:4096\n\t" \
    "ds_read_b128 v[100:103], %[a10] offset:4096\n\t" \
    "v_mfma_f32_16x16x128_f8f6f4 v[20:23], v[56:63], v[120:127], v[20:23]\n\t" \
    "ds_read_b64_tr_b8 v[136:137], %[w10] offset:0\n\t" \
    "ds_read_b128 v[56:59], %[a00] offset:0\n\t" \
    "ds_read_b128 v[60:63], %[a10] offset:0\n\t" \
    "v_mfma_f32_16x16x128_f8f6f4 v[24:27], v[64:71], v[112:119], v[24:27]\n\t" \
    "ds_read_b64_tr_b8 v[130:131], %[w00] offset:1024\n\t" \
    "ds_read_b128 v[104:107], %[a00] offset:6144\n\t" \
    "ds_read_b128 v[108:111], %[a10] offset:6144\n\t" \
    "v_mfma_f32_16x16x128_f8f6f4 v[28:31], v[64:71], v[120:127], v[28:31]\n\t" \
    "ds_read_b64_tr_b8 v[138:139], %[w10] offset:1024\n\t" \
    "ds_read_b128 v[64:67], %[a00] offset:2048\n\t" \
    "ds_read_b128 v[68:71], %[a10] offset:2048\n\t" \
    "v_mfma_f32_16x16x128_f8f6f4 v[32:35], v[80:87], v[112:119], v[32:35]\n\t" \
    "s_waitcnt lgkmcnt(12)\n\t" \
    "v_readfirstlane_b32 %[t], %[vd]\n\t" \
    "ds_read_b64_tr_b8 v[132:133], %[w00] offset:8192\n\t" \
    "ds_read_b64_tr_b8 v[140:141], %[w10] offset:8192\n\t" \
    "s_set_gpr_idx_on %[t], gpr_idx(SRC0)\n\t" \
    "v_mov_b64 v[134:135], v[144:145]\n\t" \
    "v_mov_b64 v[142:143], v[146:147]\n\t" \
    "s_set_gpr_idx_off\n\t" \
    "v_mfma_f32_16x16x128_f8f6f4 v[36:39], v[80:87], v[120:127], v[36:39]\n\t" \
    "v_mfma_f32_16x16x128_f8f6f4 v[40:43], v[88:95], v[112:119], v[40:43]\n\t" \
    "s_waitcnt lgkmcnt(0)\n\t" \
    "s_barrier\n\t" \
    "ds_read_b32 %[vd], %[z0] offset:59392\n\t" \
    "v_mfma_f32_16x16x128_f8f6f4 v[44:47], v[88:95], v[120:127], v[44:47]\n\t" \
    "v_mfma_f32_16x16x128_f8f6f4 v[16:19], v[56:63], v[128:135], v[16:19]\n\t" \
    "ds_read_b64_tr_b8 v[112:113], %[w00] offset:32768\n\t" \
    "ds_read_b128 v[80:83], %[a00] offset:36864\n\t" \
    "ds_read_b128 v[84:87], %[a10] offset:36864\n\t" \
    "v_mfma_f32_16x16x128_f8f6f4 v[20:23], v[56:63], v[136:143], v[20:23]\n\t" \
    "ds_read_b64_tr_b8 v[120:121], %[w10] offset:32768\n\t" \
    "ds_read_b128 v[56:59], %[a00] offset:32768\n\t" \
    "ds_read_b128 v[60:63], %[a10] offset:32768\n\t" \
    "v_mfma_f32_16x16x128_f8f6f4 v[24:27], v[64:71], v[128:135], v[24:27]\n\t" \
    "ds_read_b64_tr_b8 v[114:115], %[w00] offset:33792\n\t" \
    "ds_read_b128 v[88:91], %[a00] offset:38912\n\t" \
    "ds_read_b128 v[92:95], %[a10] offset:38912\n\t" \
    "v_mfma_f32_16x16x128_f8f6f4 v[28:31], v[64:71], v[136:143], v[28:31]\n\t" \
    "ds_read_b64_tr_b8 v[122:123], %[w10] offset:33792\n\t" \
    "ds_read_b128 v[64:67], %[a00] offset:34816\n\t" \
    "ds_read_b128 v[68:71], %[a10] offset:34816\n\t" \
    "v_mfma_f32_16x16x128_f8f6f4 v[32:35], v[96:103], v[128:135], v[32:35]\n\t" \
    "s_waitcnt lgkmcnt(12)\n\t" \
    "v_readfirstlane_b32 %[t], %[vd]\n\t" \
    "ds_read_b64_tr_b8 v[116:117], %[w00] offset:40960\n\t" \
    "ds_read_b64_tr_b8 v[124:125], %[w10] offset:40960\n\t" \
    "s_set_gpr_idx_on %[t], gpr_idx(SRC0)\n\t" \
    "v_mov_b64 v[118:119], v[144:145]\n\t" \
    "v_mov_b64 v[126:127], v[146:147]\n\t" \
    "s_set_gpr_idx_off\n\t" \
    "v_mfma_f32_16x16x128_f8f6f4 v[36:39], v[96:103], v[136:143], v[36:39]\n\t" \
    "v_mfma_f32_16x16x128_f8f6f4 v[40:43], v[104:111], v[128:135], v[40:43]\n\t" \
    "s_waitcnt lgkmcnt(0)\n\t" \
    "s_barrier\n\t" \
    "ds_read_b32 %[vd], %[z1] offset:26624\n\t" \
    "v_mfma_f32_16x16x128_f8f6f4 v[44:47], v[104:111], v[136:143], v[44:47]\n\t" \
    "v_mfma_f32_16x16x128_f8f6f4 v[16:19], v[56:63], v[112:119], v[16:19]\n\t" \
    "ds_read_b64_tr_b8 v[128:129], %[w01] offset:0\n\t" \
    "ds_read_b128 v[96:99], %[a01] offset:4096\n\t" \
    "ds_read_b128 v[100:103], %[a11] offset:4096\n\t" \
    "v_mfma_f32_16x16x128_f8f6f4 v[20:23], v[56:63], v[120:127], v[20:23]\n\t" \
    "ds_read_b64_tr_b8 v[136:137], %[w11] offset:0\n\t" \
    "ds_read_b128 v[56:59], %[a01] offset:0\n\t" \
    "ds_read_b128 v[60:63], %[a11] offset:0\n\t" \
    "v_mfma_f32_16x16x128_f8f6f4 v[24:27], v[64:71], v[112:119], v[24:27]\n\t" \
    "ds_read_b64_tr_b8 v[130:131], %[w01] offset:1024\n\t" \
    "ds_read_b128 v[104:107], %[a01] offset:6144\n\t" \
    "ds_read_b128 v[108:111], %[a11] offset:6144\n\t" \
    "v_mfma_f32_16x16x128_f8f6f4 v[28:31], v[64:71], v[120:127], v[28:31]\n\t" \
    "ds_read_b64_tr_b8 v[138:139], %[w11] offset:1024\n\t" \
    "ds_read_b128 v[64:67], %[a01] offset:2048\n\t" \
    "ds_read_b128 v[68:71], %[a11] offset:2048\n\t" \
    "v_mfma_f32_16x16x128_f8f6f4 v[32:35], v[80:87], v[112:119], v[32:35]\n\t" \
    "s_waitcnt lgkmcnt(12)\n\t" \
    "v_readfirstlane_b32 %[t], %[vd]\n\t" \
    "ds_read_b64_tr_b8 v[132:133], %[w01] offset:8192\n\t" \
    "ds_read_b64_tr_b8 v[140:141], %[w11] offset:8192\n\t" \
    "s_set_gpr_idx_on %[t], gpr_idx(SRC0)\n\t" \
    "v_mov_b64 v[134:135], v[144:145]\n\t" \
    "v_mov_b64 v[142:143], v[146:147]\n\t" \
    "s_set_gpr_idx_off\n\t" \
    "v_mfma_f32_16x16x128_f8f6f4 v[36:39], v[80:87], v[120:127], v[36:39]\n\t" \
    "v_mfma_f32_16x16x128_f8f6f4 v[40:43], v[88:95], v[112:119], v[40:43]\n\t" \
    "s_sub_u32 %[n], %[n], 1\n\t" \
    "s_cmp_eq_u32 %[n], 0\n\t" \
    "s_waitcnt lgkmcnt(0)\n\t" \
    "s_cbranch_scc1 2f\n\t" \
    "s_barrier\n\t" \
    "ds_read_b32 %[vd], %[z1] offset:59392\n\t" \
    "v_mfma_f32_16x16x128_f8f6f4 v[44:47], v[88:95], v[120:127], v[44:47]\n\t" \
    "v_mfma_f32_16x16x128_f8f6f4 v[16:19], v[56:63], v[128:135], v[16:19]\n\t" \
    "ds_read_b64_tr_b8 v[112:113], %[w01] offset:32768\n\t" \
    "ds_read_b128 v[80:83], %[a01] offset:36864\n\t" \
    "ds_read_b128 v[84:87], %[a11] offset:36864\n\t" \
    "v_mfma_f32_16x16x128_f8f6f4 v[20:23], v[56:63], v[136:143], v[20:23]\n\t" \
    "ds_read_b64_tr_b8 v[120:121], %[w11] offset:32768\n\t" \
    "ds_read_b128 v[56:59], %[a01] offset:32768\n\t" \
    "ds_read_b128 v[60:63], %[a11] offset:32768\n\t" \
    "v_mfma_f32_16x16x128_f8f6f4 v[24:27], v[64:71], v[128:135], v[24:27]\n\t" \
    "ds_read_b64_tr_b8 v[114:115], %[w01] offset:33792\n\t" \
    "ds_read_b128 v[88:91], %[a01] offset:38912\n\t" \
    "ds_read_b128 v[92:95], %[a11] offset:38912\n\t" \
    "v_mfma_f32_16x16x128_f8f6f4 v[28:31], v[64:71], v[136:143], v[28:31]\n\t" \
    "ds_read_b64_tr_b8 v[122:123], %[w11] offset:33792\n\t" \
    "ds_read_b128 v[64:67], %[a01] offset:34816\n\t" \
    "ds_read_b128 v[68:71], %[a11] offset:34816\n\t" \
    "v_mfma_f32_16x16x128_f8f6f4 v[32:35], v[96:103], v[128:135], v[32:35]\n\t" \
    "s_waitcnt lgkmcnt(12)\n\t" \
    "v_readfirstlane_b32 %[t], %[vd]\n\t" \
    "ds_read_b64_tr_b8 v[116:117], %[w01] offset:40960\n\t" \
    "ds_read_b64_tr_b8 v[124:125], %[w11] offset:40960\n\t" \
    "s_set_gpr_idx_on %[t], gpr_idx(SRC0)\n\t" \
    "v_mov_b64 v[118:119], v[144:145]\n\t" \
    "v_mov_b64 v[126:127], v[146:147]\n\t" \
    "s_set_gpr_idx_off\n\t" \
    "v_mfma_f32_16x16x128_f8f6f4 v[36:39], v[96:103], v[136:143], v[36:39]\n\t" \
    "v_mfma_f32_16x16x128_f8f6f4 v[40:43], v[104:111], v[128:135], v[40:43]\n\t" \
    "s_waitcnt lgkmcnt(0)\n\t" \
    "s_barrier\n\t" \
    "ds_read_b32 %[vd], %[z0] offset:26624\n\t" \
    "v_mfma_f32_16x16x128_f8f6f4 v[44:47], v[104:111], v[136:143], v[44:47]\n\t" \
    "s_branch 1b\n\t" \
      \
    "2:\n\t" \
    "ds_read_b32 %[vd], %[z1] offset:65520\n\t" \
    "v_mfma_f32_16x16x128_f8f6f4 v[44:47], v[88:95], v[120:127], v[44:47]\n\t" \
    "v_mfma_f32_16x16x128_f8f6f4 v[16:19], v[56:63], v[128:135], v[16:19]\n\t" \
    "v_mfma_f32_16x16x128_f8f6f4 v[20:23], v[56:63], v[136:143], v[20:23]\n\t" \
    "s_waitcnt lgkmcnt(0)\n\t" \
    "v_cmp_ne_u32 vcc, 0, %[vd]\n\t" \
    "v_mov_b32 %[vd], 0x7fc00000\n\t" \
    "s_nop 0\n\t" \
    "v_cndmask_b32 %[sc], %[sc], %[vd], vcc\n\t" \
    "v_mfma_f32_16x16x128_f8f6f4 v[24:27], v[64:71], v[128:135], v[24:27]\n\t" \
    "v_mfma_f32_16x16x128_f8f6f4 v[28:31], v[64:71], v[136:143], v[28:31]\n\t" \
    "s_nop 4\n\t" \
    "v_mul_f32 v16, v16, %[sc]\n\t" \
    "v_mul_f32 v17, v17, %[sc]\n\t" \
    "v_mul_f32 v18, v18, %[sc]\n\t" \
    "v_mul_f32 v19, v19, %[sc]\n\t" \
    "v_mul_f32 v20, v20, %[sc]\n\t" \
    "v_mul_f32 v21, v21, %[sc]\n\t" \
    "v_mul_f32 v22, v22, %[sc]\n\t" \
    "v_mul_f32 v23, v23, %[sc]\n\t" \
    "v_maximum3_f32 v16, v16, 0, 0\n\t" \
    "v_maximum3_f32 v20, v20, 0, 0\n\t" \
    "v_maximum3_f32 v17, v17, 0, 0\n\t" \
    "v_maximum3_f32 v21, v21, 0, 0\n\t" \
    "v_maximum3_f32 v18, v18, 0, 0\n\t" \
    "v_maximum3_f32 v22, v22, 0, 0\n\t" \
    "v_maximum3_f32 v19, v19, 0, 0\n\t" \
    "v_maximum3_f32 v23, v23, 0, 0\n\t" \
    "v_permlane16_swap_b32 v16, v20\n\t" \
    "v_permlane16_swap_b32 v17, v21\n\t" \
    "v_permlane16_swap_b32 v18, v22\n\t" \
    "v_permlane16_swap_b32 v19, v23\n\t" \
    "buffer_store_dword v16, %[vo], %[ro], 0 offen sc0 sc1\n\t" \
    "buffer_store_dword v17, %[vo], %[ro], s70 offen sc0 sc1\n\t" \
    "buffer_store_dword v18, %[vo], %[ro], s71 offen sc0 sc1\n\t" \
    "buffer_store_dword v19, %[vo], %[ro], s72 offen sc0 sc1\n\t" \
    "buffer_store_dword v20, %[vo], %[ro], s73 offen sc0 sc1\n\t" \
    "buffer_store_dword v21, %[vo], %[ro], s74 offen sc0 sc1\n\t" \
    "buffer_store_dword v22, %[vo], %[ro], s75 offen sc0 sc1\n\t" \
    "buffer_store_dword v23, %[vo], %[ro], s76 offen sc0 sc1\n\t" \
    "v_add_u32 %[vo], s77, %[vo]\n\t" \
    "v_mfma_f32_16x16x128_f8f6f4 v[32:35], v[96:103], v[128:135], v[32:35]\n\t" \
    "v_mfma_f32_16x16x128_f8f6f4 v[36:39], v[96:103], v[136:143], v[36:39]\n\t" \
    "v_mul_f32 v24, v24, %[sc]\n\t" \
    "v_mul_f32 v25, v25, %[sc]\n\t" \
    "v_mul_f32 v26, v26, %[sc]\n\t" \
    "v_mul_f32 v27, v27, %[sc]\n\t" \
    "v_mul_f32 v28, v28, %[sc]\n\t" \
    "v_mul_f32 v29, v29, %[sc]\n\t" \
    "v_mul_f32 v30, v30, %[sc]\n\t" \
    "v_mul_f32 v31, v31, %[sc]\n\t" \
    "v_maximum3_f32 v24, v24, 0, 0\n\t" \
    "v_maximum3_f32 v28, v28, 0, 0\n\t" \
    "v_maximum3_f32 v25, v25, 0, 0\n\t" \
    "v_maximum3_f32 v29, v29, 0, 0\n\t" \
    "v_maximum3_f32 v26, v26, 0, 0\n\t" \
    "v_maximum3_f32 v30, v30, 0, 0\n\t" \
    "v_maximum3_f32 v27, v27, 0, 0\n\t" \
    "v_maximum3_f32 v31, v31, 0, 0\n\t" \
    "v_permlane16_swap_b32 v24, v28\n\t" \
    "v_permlane16_swap_b32 v25, v29\n\t" \
    "v_permlane16_swap_b32 v26, v30\n\t" \
    "v_permlane16_swap_b32 v27, v31\n\t" \
    "buffer_store_dword v24, %[vo], %[ro], 0 offen sc0 sc1\n\t" \
    "buffer_store_dword v25, %[vo], %[ro], s70 offen sc0 sc1\n\t" \
    "buffer_store_dword v26, %[vo], %[ro], s71 offen sc0 sc1\n\t" \
    "buffer_store_dword v27, %[vo], %[ro], s72 offen sc0 sc1\n\t" \
    "buffer_store_dword v28, %[vo], %[ro], s73 offen sc0 sc1\n\t" \
    "buffer_store_dword v29, %[vo], %[ro], s74 offen sc0 sc1\n\t" \
    "buffer_store_dword v30, %[vo], %[ro], s75 offen sc0 sc1\n\t" \
    "buffer_store_dword v31, %[vo], %[ro], s76 offen sc0 sc1\n\t" \
    "v_add_u32 %[vo], s77, %[vo]\n\t" \
    "v_mfma_f32_16x16x128_f8f6f4 v[40:43], v[104:111], v[128:135], v[40:43]\n\t" \
    "v_mfma_f32_16x16x128_f8f6f4 v[44:47], v[104:111], v[136:143], v[44:47]\n\t" \
    "v_mul_f32 v32, v32, %[sc]\n\t" \
    "v_mul_f32 v33, v33, %[sc]\n\t" \
    "v_mul_f32 v34, v34, %[sc]\n\t" \
    "v_mul_f32 v35, v35, %[sc]\n\t" \
    "v_mul_f32 v36, v36, %[sc]\n\t" \
    "v_mul_f32 v37, v37, %[sc]\n\t" \
    "v_mul_f32 v38, v38, %[sc]\n\t" \
    "v_mul_f32 v39, v39, %[sc]\n\t" \
    "v_maximum3_f32 v32, v32, 0, 0\n\t" \
    "v_maximum3_f32 v36, v36, 0, 0\n\t" \
    "v_maximum3_f32 v33, v33, 0, 0\n\t" \
    "v_maximum3_f32 v37, v37, 0, 0\n\t" \
    "v_maximum3_f32 v34, v34, 0, 0\n\t" \
    "v_maximum3_f32 v38, v38, 0, 0\n\t" \
    "v_maximum3_f32 v35, v35, 0, 0\n\t" \
    "v_maximum3_f32 v39, v39, 0, 0\n\t" \
    "v_permlane16_swap_b32 v32, v36\n\t" \
    "v_permlane16_swap_b32 v33, v37\n\t" \
    "v_permlane16_swap_b32 v34, v38\n\t" \
    "v_permlane16_swap_b32 v35, v39\n\t" \
    "buffer_store_dword v32, %[vo], %[ro], 0 offen sc0 sc1\n\t" \
    "buffer_store_dword v33, %[vo], %[ro], s70 offen sc0 sc1\n\t" \
    "buffer_store_dword v34, %[vo], %[ro], s71 offen sc0 sc1\n\t" \
    "buffer_store_dword v35, %[vo], %[ro], s72 offen sc0 sc1\n\t" \
    "buffer_store_dword v36, %[vo], %[ro], s73 offen sc0 sc1\n\t" \
    "buffer_store_dword v37, %[vo], %[ro], s74 offen sc0 sc1\n\t" \
    "buffer_store_dword v38, %[vo], %[ro], s75 offen sc0 sc1\n\t" \
    "buffer_store_dword v39, %[vo], %[ro], s76 offen sc0 sc1\n\t" \
    "v_add_u32 %[vo], s77, %[vo]\n\t" \
    "v_mul_f32 v40, v40, %[sc]\n\t" \
    "v_mul_f32 v41, v41, %[sc]\n\t" \
    "v_mul_f32 v42, v42, %[sc]\n\t" \
    "v_mul_f32 v43, v43, %[sc]\n\t" \
    "v_mul_f32 v44, v44, %[sc]\n\t" \
    "v_mul_f32 v45, v45, %[sc]\n\t" \
    "v_mul_f32 v46, v46, %[sc]\n\t" \
    "v_mul_f32 v47, v47, %[sc]\n\t" \
    "v_maximum3_f32 v40, v40, 0, 0\n\t" \
    "v_maximum3_f32 v44, v44, 0, 0\n\t" \
    "v_maximum3_f32 v41, v41, 0, 0\n\t" \
    "v_maximum3_f32 v45, v45, 0, 0\n\t" \
    "v_maximum3_f32 v42, v42, 0, 0\n\t" \
    "v_maximum3_f32 v46, v46, 0, 0\n\t" \
    "v_maximum3_f32 v43, v43, 0, 0\n\t" \
    "v_maximum3_f32 v47, v47, 0, 0\n\t" \
    "v_permlane16_swap_b32 v40, v44\n\t" \
    "v_permlane16_swap_b32 v41, v45\n\t" \
    "v_permlane16_swap_b32 v42, v46\n\t" \
    "v_permlane16_swap_b32 v43, v47\n\t" \
    "buffer_store_dword v40, %[vo], %[ro], 0 offen sc0 sc1\n\t" \
    "buffer_store_dword v41, %[vo], %[ro], s70 offen sc0 sc1\n\t" \
    "buffer_store_dword v42, %[vo], %[ro], s71 offen sc0 sc1\n\t" \
    "buffer_store_dword v43, %[vo], %[ro], s72 offen sc0 sc1\n\t" \
    "buffer_store_dword v44, %[vo], %[ro], s73 offen sc0 sc1\n\t" \
    "buffer_store_dword v45, %[vo], %[ro], s74 offen sc0 sc1\n\t" \
    "buffer_store_dword v46, %[vo], %[ro], s75 offen sc0 sc1\n\t" \
    "buffer_store_dword v47, %[vo], %[ro], s76 offen sc0 sc1\n\t"

#define CONS_ZERO \
    "v_mov_b64 v[16:17], 0\n\t" \
    "v_mov_b64 v[18:19], 0\n\t" \
    "v_mov_b64 v[20:21], 0\n\t" \
    "v_mov_b64 v[22:23], 0\n\t" \
    "v_mov_b64 v[24:25], 0\n\t" \
    "v_mov_b64 v[26:27], 0\n\t" \
    "v_mov_b64 v[28:29], 0\n\t" \
    "v_mov_b64 v[30:31], 0\n\t" \
    "v_mov_b64 v[32:33], 0\n\t" \
    "v_mov_b64 v[34:35], 0\n\t" \
    "v_mov_b64 v[36:37], 0\n\t" \
    "v_mov_b64 v[38:39], 0\n\t" \
    "v_mov_b64 v[40:41], 0\n\t" \
    "v_mov_b64 v[42:43], 0\n\t" \
    "v_mov_b64 v[44:45], 0\n\t" \
    "v_mov_b64 v[46:47], 0\n\t" \
    "v_mov_b64 v[48:49], 0\n\t" \
    "v_mov_b64 v[50:51], 0\n\t" \
    "v_mov_b64 v[52:53], 0\n\t" \
    "v_mov_b64 v[54:55], 0\n\t" \
    "s_mov_b32 s70, 0x4000\n\t" \
    "s_mov_b32 s71, 0x8000\n\t" \
    "s_mov_b32 s72, 0xc000\n\t" \
    "s_mov_b32 s73, 0x10000\n\t" \
    "s_mov_b32 s74, 0x14000\n\t" \
    "s_mov_b32 s75, 0x18000\n\t" \
    "s_mov_b32 s76, 0x1c000\n\t" \
    "s_mov_b32 s77, 0x40000\n\t"
#define CONS_CLOBBERS \
    "s70", "s71", "s72", "s73", "s74", "s75", "s76", "s77", "v16", "v17", "v18", "v19", "v20", "v21", "v22", "v23", \
    "v24", "v25", "v26", "v27", "v28", "v29", "v30", "v31", "v32", "v33", "v34", "v35", "v36", "v37", "v38", "v39", \
    "v40", "v41", "v42", "v43", "v44", "v45", "v46", "v47", "v48", "v49", "v50", "v51", "v52", "v53", "v54", "v55", \
    "v56", "v57", "v58", "v59", "v60", "v61", "v62", "v63", "v64", "v65", "v66", "v67", "v68", "v69", "v70", "v71", \
    "v72", "v73", "v74", "v75", "v76", "v77", "v78", "v79", "v80", "v81", "v82", "v83", "v84", "v85", "v86", "v87", \
    "v88", "v89", "v90", "v91", "v92", "v93", "v94", "v95", "v96", "v97", "v98", "v99", "v100", "v101", "v102", "v103", \
    "v104", "v105", "v106", "v107", "v108", "v109", "v110", "v111", "v112", "v113", "v114", "v115", "v116", "v117", "v118", "v119", \
    "v120", "v121", "v122", "v123", "v124", "v125", "v126", "v127", "v128", "v129", "v130", "v131", "v132", "v133", "v134", "v135", \
    "v136", "v137", "v138", "v139", "v140", "v141", "v142", "v143", "v144", "v145", "v146", "v147", "v148", "v149", "v150", "v151", \
    "v152", "v153", "v154", "v155", "v156", "v157", "v158", "v159", "v160", "v161", "v162", "v163", "v164", "v165", "v166", "v167", \
    "v168", "v169", "v170", "v171", "v172", "v173", "v174", "v175", "v176", "v177", "v178", "v179", "v180", "v181", "v182", "v183", \
    "v184", "v185", "v186", "v187", "v188", "v189", "v190", "v191", "v192", "v193", "v194", "v195", "v196", "v197", "v198", "v199", \
    "v200", "v201", "v202", "v203", "v204", "v205", "v206", "v207", "v208", "v209", "v210", "v211", "v212", "v213", "v214", "v215", \
    "v216", "v217", "v218", "v219", "v220", "v221", "v222", "v223", "v224", "v225", "v226", "v227", "v228", "v229", "v230", "v231", \
    "v232", "v233", "v234", "v235", "v236", "v237", "v238", "v239", "v240", "v241", "v242", "v243", "v244", "v245", "v246", "v247", \
    "v248", "v249", "v250", "v251", "v252", "v253", "v254", "v255"
#define LOADER5_ASM \
    "s_mov_b32 %[u], 1\n\t" \
    "s_mov_b32 %[dst], %[slot]\n\t" \
    "s_mov_b32 %[xp], 1\n\t" \
    "s_mov_b32 m0, %[slot]\n\t" \
    "s_mov_b32 %[wm], 0\n\t" \
      \
    "1:\n\t" \
    "global_load_dword %[f], %[foff], %[ws] sc0 sc1\n\t" \
    "buffer_load_dwordx4 %[a0], %[rx], %[oa] offen offset:0 lds\n\t" \
    "s_lshl_b32 %[ow], %[oa], 12\n\t" \
    "s_add_u32 %[ow], %[ow], %[own]\n\t" \
    "buffer_load_dwordx4 %[a1], %[rx], %[oa] offen offset:1024 lds\n\t" \
    "buffer_load_dwordx4 %[a2], %[rx], %[oa] offen offset:2048 lds\n\t" \
    "buffer_load_dwordx4 %[a3], %[rx], %[oa] offen offset:3072 lds\n\t" \
    "s_addk_i32 m0, 0x1000\n\t" \
    "s_add_u32 %[dst], %[dst], %[slot]\n\t" \
    "s_bitset0_b32 %[dst], 17\n\t" \
    "buffer_load_dwordx4 %[a4], %[rx], %[oa] offen offset:0 lds\n\t" \
    "buffer_load_dwordx4 %[a5], %[rx], %[oa] offen offset:1024 lds\n\t" \
    "buffer_load_dwordx4 %[a6], %[rx], %[oa] offen offset:2048 lds\n\t" \
    "buffer_load_dwordx4 %[a7], %[rx], %[oa] offen offset:3072 lds\n\t" \
    "s_addk_i32 m0, 0x1000\n\t" \
    "s_nop 0\n\t" \
    "buffer_load_dwordx4 %[a8], %[rx], %[oa] offen offset:0 lds\n\t" \
    "buffer_load_dwordx4 %[a9], %[rx], %[oa] offen offset:1024 lds\n\t" \
    "buffer_load_dwordx4 %[w0], %[rw], %[ow] offen offset:2048 lds\n\t" \
    "buffer_load_dwordx4 %[w1], %[rw], %[ow] offen offset:3072 lds\n\t" \
    "s_addk_i32 m0, 0x1000\n\t" \
    "s_addk_i32 %[oa], 128\n\t" \
    "s_bitset0_b32 %[oa], 9\n\t" \
    "buffer_load_dwordx4 %[w2], %[rw], %[ow] offen offset:0 lds\n\t" \
    "buffer_load_dwordx4 %[w3], %[rw], %[ow] offen offset:1024 lds\n\t" \
    "buffer_load_dwordx4 %[w4], %[rw], %[ow] offen offset:2048 lds\n\t" \
    "buffer_load_dwordx4 %[w5], %[rw], %[ow] offen offset:3072 lds\n\t" \
    "s_addk_i32 m0, 0x1000\n\t" \
    "s_add_u32 %[u], %[u], 1\n\t" \
    "buffer_load_dwordx4 %[w6], %[rw], %[ow] offen offset:0 lds\n\t" \
    "buffer_load_dwordx4 %[w7], %[rw], %[ow] offen offset:1024 lds\n\t" \
    "buffer_load_dwordx4 %[w8], %[rw], %[ow] offen offset:2048 lds\n\t" \
    "buffer_load_dwordx4 %[w9], %[rw], %[ow] offen offset:3072 lds\n\t" \
    "s_addk_i32 m0, 0x1000\n\t" \
    "s_add_u32 %[xp], %[xp], 1\n\t" \
    "buffer_load_dwordx4 %[w10], %[rw], %[ow] offen offset:0 lds\n\t" \
    "buffer_load_dwordx4 %[w11], %[rw], %[ow] offen offset:1024 lds\n\t" \
    "buffer_load_dwordx4 %[w12], %[rw], %[ow] offen offset:2048 lds\n\t" \
    "buffer_load_dwordx4 %[w13], %[rw], %[ow] offen offset:3072 lds\n\t" \
    "s_addk_i32 m0, 0x1000\n\t" \
    "s_nop 0\n\t" \
    "buffer_load_dwordx4 %[w14], %[rw], %[ow] offen offset:0 lds\n\t" \
    "buffer_load_dwordx4 %[w15], %[rw], %[ow] offen offset:1024 lds\n\t" \
    "s_mov_b32 m0, %[dst]\n\t" \
    "s_cmp_le_u32 %[u], 2\n\t" \
    "s_cbranch_scc1 1b\n\t" \
    "s_waitcnt vmcnt(52)\n\t" \
    "s_barrier\n\t" \
    "s_mov_b32 %[u], 2\n\t" \
    "s_cmp_lt_u32 %[xp], 4\n\t" \
    "s_cbranch_scc1 1b\n\t" \
      \
    "s_mov_b32 %[xp], 0\n\t" \
    "s_lshl_b32 %[ow], %[s0], 1\n\t" \
    "v_readlane_b32 %[call], %[f], %[lc]\n\t" \
    "s_addk_i32 m0, 0x2000\n\t" \
    "s_movk_i32 %[sw], 0x100\n\t" \
    "s_add_u32 %[call], %[call], 1\n\t" \
    "v_mov_b32 %[vcall], " VCALL "\n\t" \
    "2:\n\t" \
    "buffer_load_dwordx4 %[k0], %[rw], %[ow] offen offset:2048 lds\n\t" \
    "buffer_load_dwordx4 %[k1], %[rw], %[ow] offen offset:3072 lds\n\t" \
    "s_addk_i32 m0, 0x1000\n\t" \
    "s_add_u32 %[dst], %[dst], %[slot]\n\t" \
    "s_bitset0_b32 %[dst], 17\n\t" \
    "buffer_load_dwordx4 %[k2], %[rw], %[ow] offen offset:0 lds\n\t" \
    "buffer_load_dwordx4 %[k3], %[rw], %[ow] offen offset:1024 lds\n\t" \
    "buffer_load_dwordx4 %[k4], %[rw], %[ow] offen offset:2048 lds\n\t" \
    "buffer_load_dwordx4 %[k5], %[rw], %[ow] offen offset:3072 lds\n\t" \
    "s_addk_i32 m0, 0x1000\n\t" \
    "s_add_u32 %[xp], %[xp], 1\n\t" \
    "s_add_u32 %[t], %[ow], 0x200000\n\t" \
    "s_bitset0_b32 %[t], 24\n\t" \
    "buffer_load_dwordx4 %[k6], %[rw], %[ow] offen offset:0 lds\n\t" \
    "buffer_load_dwordx4 %[k7], %[rw], %[ow] offen offset:1024 lds\n\t" \
    "buffer_load_dwordx4 %[k8], %[rw], %[ow] offen offset:2048 lds\n\t" \
    "buffer_load_dwordx4 %[k9], %[rw], %[ow] offen offset:3072 lds\n\t" \
    "s_addk_i32 m0, 0x1000\n\t" \
    "s_cmp_eq_u32 %[t], %[own]\n\t" \
    "s_cselect_b32 %[od], 0x200000, 0\n\t" \
    "buffer_load_dwordx4 %[k10], %[rw], %[ow] offen offset:0 lds\n\t" \
    "buffer_load_dwordx4 %[k11], %[rw], %[ow] offen offset:1024 lds\n\t" \
    "buffer_load_dwordx4 %[k12], %[rw], %[ow] offen offset:2048 lds\n\t" \
    "buffer_load_dwordx4 %[k13], %[rw], %[ow] offen offset:3072 lds\n\t" \
    "s_addk_i32 m0, 0x1000\n\t" \
    "s_add_u32 %[t], %[t], %[od]\n\t" \
    "s_bitset0_b32 %[t], 24\n\t" \
    "buffer_load_dwordx4 %[k14], %[rw], %[ow] offen offset:0 lds\n\t" \
    "buffer_load_dwordx4 %[k15], %[rw], %[ow] offen offset:1024 lds\n\t" \
    "s_mov_b32 %[ow], %[t]\n\t" \
    "s_add_u32 m0, %[dst], 0x2000\n\t" \
    "s_bitcmp1_b32 232, %[xp]\n\t" \
    "s_cbranch_scc1 7f\n\t" \
    "3:\n\t" \
    "s_cmp_lt_u32 %[xp], 7\n\t" \
    "s_waitcnt vmcnt(32)\n\t" \
    "s_barrier\n\t" \
    "s_cbranch_scc1 2b\n\t" \
    "s_mov_b32 m0, %[dst]\n\t" \
    "s_mov_b32 %[rdy], 0\n\t" \
    "s_and_b32 %[t], %[call], 1\n\t" \
    "s_lshl_b32 %[t], %[t], 16\n\t" \
    "v_add_u32 %[l0], %[t], %[l0]\n\t" \
    "v_add_u32 %[l1], %[t], %[l1]\n\t" \
    "v_add_u32 %[l2], %[t], %[l2]\n\t" \
    "s_mov_b32 %[xp], 0\n\t" \
    "s_mov_b32 %[np], 0\n\t" \
    "s_mov_b32 %[tp], 0\n\t" \
    "s_movk_i32 %[msk], 39\n\t" \
    "s_waitcnt vmcnt(16)\n\t" \
    "s_barrier\n\t" \
    "s_mov_b32 %[u], 1\n\t" \
    "s_branch 8f\n\t" \
      \
    ".p2align 6\n\t" \
    "40:\n\t" \
    "buffer_load_dwordx4 %[l0], %[ra], %[oa] offen offset:0 sc0 sc1 lds\n\t" \
    "buffer_load_dwordx4 %[l0], %[ra], %[oa] offen offset:1024 sc0 sc1 lds\n\t" \
    "buffer_load_dwordx4 %[l0], %[ra], %[oa] offen offset:2048 sc0 sc1 lds\n\t" \
    "buffer_load_dwordx4 %[l0], %[ra], %[oa] offen offset:3072 sc0 sc1 lds\n\t" \
    "s_addk_i32 m0, 0x1000\n\t" \
    "s_lshl_b32 %[ow], %[oa], 1\n\t" \
    "s_lshr_b32 %[od], %[oa], 16\n\t" \
    "buffer_load_dwordx4 %[l1], %[ra], %[oa] offen offset:0 sc0 sc1 lds\n\t" \
    "buffer_load_dwordx4 %[l1], %[ra], %[oa] offen offset:1024 sc0 sc1 lds\n\t" \
    "buffer_load_dwordx4 %[l1], %[ra], %[oa] offen offset:2048 sc0 sc1 lds\n\t" \
    "buffer_load_dwordx4 %[l1], %[ra], %[oa] offen offset:3072 sc0 sc1 lds\n\t" \
    "s_addk_i32 m0, 0x1000\n\t" \
    "s_bcnt1_i32_b32 %[t], %[rdy]\n\t" \
    "buffer_load_dwordx4 %[l2], %[ra], %[oa] offen offset:0 sc0 sc1 lds\n\t" \
    "buffer_load_dwordx4 %[l2], %[ra], %[oa] offen offset:1024 sc0 sc1 lds\n\t" \
    "buffer_load_dwordx4 %[w0], %[rw], %[ow] offen offset:2048 lds\n\t" \
    "buffer_load_dwordx4 %[w1], %[rw], %[ow] offen offset:3072 lds\n\t" \
    "s_addk_i32 m0, 0x1000\n\t" \
    "s_bitcmp1_b32 %[msk], %[t]\n\t" \
    "s_cbranch_scc1 60f\n\t" \
    "50:\n\t" \
    "buffer_load_dwordx4 %[w2], %[rw], %[ow] offen offset:0 lds\n\t" \
    "buffer_load_dwordx4 %[w3], %[rw], %[ow] offen offset:1024 lds\n\t" \
    "buffer_load_dwordx4 %[w4], %[rw], %[ow] offen offset:2048 lds\n\t" \
    "s_waitcnt vmcnt(38)\n\t" \
    "s_barrier\n\t" \
    "buffer_load_dwordx4 %[w5], %[rw], %[ow] offen offset:3072 lds\n\t" \
    "s_addk_i32 m0, 0x1000\n\t" \
    "s_ff1_i32_b32 %[id], %[rdy]\n\t" \
    "s_bitset0_b32 %[rdy], %[id]\n\t" \
    "buffer_load_dwordx4 %[w6], %[rw], %[ow] offen offset:0 lds\n\t" \
    "buffer_load_dwordx4 %[w7], %[rw], %[ow] offen offset:1024 lds\n\t" \
    "buffer_load_dwordx4 %[w8], %[rw], %[ow] offen offset:2048 lds\n\t" \
    "s_addk_i32 m0, 0x1000\n\t" \
    "s_mul_i32 %[t], %[id], %[mul]\n\t" \
    "s_add_u32 %[t], %[t], %[s0]\n\t" \
    "buffer_load_dwordx4 %[w10], %[rw], %[ow] offen offset:0 lds\n\t" \
    "buffer_load_dwordx4 %[w12], %[rw], %[ow] offen offset:2048 lds\n\t" \
    "s_addk_i32 m0, 0x1000\n\t" \
    "s_and_b32 %[oa], %[t], %[cut]\n\t" \
    "buffer_load_dwordx4 %[w14], %[rw], %[ow] offen offset:0 lds\n\t" \
    "buffer_load_dword %[zero], %[rd], %[od] offen offset:2048 lds\n\t" \
    "s_addk_i32 m0, 0x2000\n\t" \
    "s_cmp_lt_i32 %[id], 0\n\t" \
    "s_cbranch_scc1 20f\n\t" \
    "41:\n\t" \
    "buffer_load_dwordx4 %[l0], %[ra], %[oa] offen offset:0 sc0 sc1 lds\n\t" \
    "buffer_load_dwordx4 %[l0], %[ra], %[oa] offen offset:1024 sc0 sc1 lds\n\t" \
    "buffer_load_dwordx4 %[l0], %[ra], %[oa] offen offset:2048 sc0 sc1 lds\n\t" \
    "buffer_load_dwordx4 %[l0], %[ra], %[oa] offen offset:3072 sc0 sc1 lds\n\t" \
    "s_addk_i32 m0, 0x1000\n\t" \
    "s_lshl_b32 %[ow], %[oa], 1\n\t" \
    "s_lshr_b32 %[od], %[oa], 16\n\t" \
    "buffer_load_dwordx4 %[l1], %[ra], %[oa] offen offset:0 sc0 sc1 lds\n\t" \
    "buffer_load_dwordx4 %[l1], %[ra], %[oa] offen offset:1024 sc0 sc1 lds\n\t" \
    "buffer_load_dwordx4 %[l1], %[ra], %[oa] offen offset:2048 sc0 sc1 lds\n\t" \
    "buffer_load_dwordx4 %[l1], %[ra], %[oa] offen offset:3072 sc0 sc1 lds\n\t" \
    "s_addk_i32 m0, 0x1000\n\t" \
    "s_bcnt1_i32_b32 %[t], %[rdy]\n\t" \
    "buffer_load_dwordx4 %[l2], %[ra], %[oa] offen offset:0 sc0 sc1 lds\n\t" \
    "buffer_load_dwordx4 %[l2], %[ra], %[oa] offen offset:1024 sc0 sc1 lds\n\t" \
    "buffer_load_dwordx4 %[w0], %[rw], %[ow] offen offset:2048 lds\n\t" \
    "buffer_load_dwordx4 %[w1], %[rw], %[ow] offen offset:3072 lds\n\t" \
    "s_addk_i32 m0, 0x1000\n\t" \
    "s_bitcmp1_b32 %[msk], %[t]\n\t" \
    "s_cbranch_scc1 61f\n\t" \
    "51:\n\t" \
    "buffer_load_dwordx4 %[w2], %[rw], %[ow] offen offset:0 lds\n\t" \
    "buffer_load_dwordx4 %[w3], %[rw], %[ow] offen offset:1024 lds\n\t" \
    "buffer_load_dwordx4 %[w4], %[rw], %[ow] offen offset:2048 lds\n\t" \
    "s_waitcnt vmcnt(38)\n\t" \
    "s_barrier\n\t" \
    "buffer_load_dwordx4 %[w5], %[rw], %[ow] offen offset:3072 lds\n\t" \
    "s_addk_i32 m0, 0x1000\n\t" \
    "s_ff1_i32_b32 %[id], %[rdy]\n\t" \
    "s_bitset0_b32 %[rdy], %[id]\n\t" \
    "buffer_load_dwordx4 %[w6], %[rw], %[ow] offen offset:0 lds\n\t" \
    "buffer_load_dwordx4 %[w7], %[rw], %[ow] offen offset:1024 lds\n\t" \
    "buffer_load_dwordx4 %[w8], %[rw], %[ow] offen offset:2048 lds\n\t" \
    "s_addk_i32 m0, 0x1000\n\t" \
    "s_mul_i32 %[t], %[id], %[mul]\n\t" \
    "s_add_u32 %[t], %[t], %[s0]\n\t" \
    "buffer_load_dwordx4 %[w10], %[rw], %[ow] offen offset:0 lds\n\t" \
    "buffer_load_dwordx4 %[w12], %[rw], %[ow] offen offset:2048 lds\n\t" \
    "s_addk_i32 m0, 0x1000\n\t" \
    "s_and_b32 %[oa], %[t], %[cut]\n\t" \
    "buffer_load_dwordx4 %[w14], %[rw], %[ow] offen offset:0 lds\n\t" \
    "buffer_load_dword %[zero], %[rd], %[od] offen offset:2048 lds\n\t" \
    "s_addk_i32 m0, 0x2000\n\t" \
    "s_cmp_lt_i32 %[id], 0\n\t" \
    "s_cbranch_scc1 20f\n\t" \
    "42:\n\t" \
    "buffer_load_dwordx4 %[l0], %[ra], %[oa] offen offset:0 sc0 sc1 lds\n\t" \
    "buffer_load_dwordx4 %[l0], %[ra], %[oa] offen offset:1024 sc0 sc1 lds\n\t" \
    "buffer_load_dwordx4 %[l0], %[ra], %[oa] offen offset:2048 sc0 sc1 lds\n\t" \
    "buffer_load_dwordx4 %[l0], %[ra], %[oa] offen offset:3072 sc0 sc1 lds\n\t" \
    "s_addk_i32 m0, 0x1000\n\t" \
    "s_lshl_b32 %[ow], %[oa], 1\n\t" \
    "s_lshr_b32 %[od], %[oa], 16\n\t" \
    "buffer_load_dwordx4 %[l1], %[ra], %[oa] offen offset:0 sc0 sc1 lds\n\t" \
    "buffer_load_dwordx4 %[l1], %[ra], %[oa] offen offset:1024 sc0 sc1 lds\n\t" \
    "buffer_load_dwordx4 %[l1], %[ra], %[oa] offen offset:2048 sc0 sc1 lds\n\t" \
    "buffer_load_dwordx4 %[l1], %[ra], %[oa] offen offset:3072 sc0 sc1 lds\n\t" \
    "s_addk_i32 m0, 0x1000\n\t" \
    "s_bcnt1_i32_b32 %[t], %[rdy]\n\t" \
    "buffer_load_dwordx4 %[l2], %[ra], %[oa] offen offset:0 sc0 sc1 lds\n\t" \
    "buffer_load_dwordx4 %[l2], %[ra], %[oa] offen offset:1024 sc0 sc1 lds\n\t" \
    "buffer_load_dwordx4 %[w0], %[rw], %[ow] offen offset:2048 lds\n\t" \
    "buffer_load_dwordx4 %[w1], %[rw], %[ow] offen offset:3072 lds\n\t" \
    "s_addk_i32 m0, 0x1000\n\t" \
    "s_bitcmp1_b32 %[msk], %[t]\n\t" \
    "s_cbranch_scc1 62f\n\t" \
    "52:\n\t" \
    "buffer_load_dwordx4 %[w2], %[rw], %[ow] offen offset:0 lds\n\t" \
    "buffer_load_dwordx4 %[w3], %[rw], %[ow] offen offset:1024 lds\n\t" \
    "buffer_load_dwordx4 %[w4], %[rw], %[ow] offen offset:2048 lds\n\t" \
    "s_waitcnt vmcnt(38)\n\t" \
    "s_barrier\n\t" \
    "buffer_load_dwordx4 %[w5], %[rw], %[ow] offen offset:3072 lds\n\t" \
    "s_addk_i32 m0, 0x1000\n\t" \
    "s_ff1_i32_b32 %[id], %[rdy]\n\t" \
    "s_bitset0_b32 %[rdy], %[id]\n\t" \
    "buffer_load_dwordx4 %[w6], %[rw], %[ow] offen offset:0 lds\n\t" \
    "buffer_load_dwordx4 %[w7], %[rw], %[ow] offen offset:1024 lds\n\t" \
    "buffer_load_dwordx4 %[w8], %[rw], %[ow] offen offset:2048 lds\n\t" \
    "s_addk_i32 m0, 0x1000\n\t" \
    "s_mul_i32 %[t], %[id], %[mul]\n\t" \
    "s_add_u32 %[t], %[t], %[s0]\n\t" \
    "buffer_load_dwordx4 %[w10], %[rw], %[ow] offen offset:0 lds\n\t" \
    "buffer_load_dwordx4 %[w12], %[rw], %[ow] offen offset:2048 lds\n\t" \
    "s_addk_i32 m0, 0x1000\n\t" \
    "s_and_b32 %[oa], %[t], %[cut]\n\t" \
    "buffer_load_dwordx4 %[w14], %[rw], %[ow] offen offset:0 lds\n\t" \
    "buffer_load_dword %[zero], %[rd], %[od] offen offset:2048 lds\n\t" \
    "s_addk_i32 m0, 0x2000\n\t" \
    "s_cmp_lt_i32 %[id], 0\n\t" \
    "s_cbranch_scc1 20f\n\t" \
    "43:\n\t" \
    "buffer_load_dwordx4 %[l0], %[ra], %[oa] offen offset:0 sc0 sc1 lds\n\t" \
    "buffer_load_dwordx4 %[l0], %[ra], %[oa] offen offset:1024 sc0 sc1 lds\n\t" \
    "buffer_load_dwordx4 %[l0], %[ra], %[oa] offen offset:2048 sc0 sc1 lds\n\t" \
    "buffer_load_dwordx4 %[l0], %[ra], %[oa] offen offset:3072 sc0 sc1 lds\n\t" \
    "s_addk_i32 m0, 0x1000\n\t" \
    "s_lshl_b32 %[ow], %[oa], 1\n\t" \
    "s_lshr_b32 %[od], %[oa], 16\n\t" \
    "buffer_load_dwordx4 %[l1], %[ra], %[oa] offen offset:0 sc0 sc1 lds\n\t" \
    "buffer_load_dwordx4 %[l1], %[ra], %[oa] offen offset:1024 sc0 sc1 lds\n\t" \
    "buffer_load_dwordx4 %[l1], %[ra], %[oa] offen offset:2048 sc0 sc1 lds\n\t" \
    "buffer_load_dwordx4 %[l1], %[ra], %[oa] offen offset:3072 sc0 sc1 lds\n\t" \
    "s_addk_i32 m0, 0x1000\n\t" \
    "s_bcnt1_i32_b32 %[t], %[rdy]\n\t" \
    "buffer_load_dwordx4 %[l2], %[ra], %[oa] offen offset:0 sc0 sc1 lds\n\t" \
    "buffer_load_dwordx4 %[l2], %[ra], %[oa] offen offset:1024 sc0 sc1 lds\n\t" \
    "buffer_load_dwordx4 %[w0], %[rw], %[ow] offen offset:2048 lds\n\t" \
    "buffer_load_dwordx4 %[w1], %[rw], %[ow] offen offset:3072 lds\n\t" \
    "s_addk_i32 m0, 0x1000\n\t" \
    "s_bitcmp1_b32 %[msk], %[t]\n\t" \
    "s_cbranch_scc1 63f\n\t" \
    "53:\n\t" \
    "buffer_load_dwordx4 %[w2], %[rw], %[ow] offen offset:0 lds\n\t" \
    "buffer_load_dwordx4 %[w3], %[rw], %[ow] offen offset:1024 lds\n\t" \
    "buffer_load_dwordx4 %[w4], %[rw], %[ow] offen offset:2048 lds\n\t" \
    "s_waitcnt vmcnt(38)\n\t" \
    "s_barrier\n\t" \
    "buffer_load_dwordx4 %[w5], %[rw], %[ow] offen offset:3072 lds\n\t" \
    "s_addk_i32 m0, 0x1000\n\t" \
    "s_ff1_i32_b32 %[id], %[rdy]\n\t" \
    "s_bitset0_b32 %[rdy], %[id]\n\t" \
    "buffer_load_dwordx4 %[w6], %[rw], %[ow] offen offset:0 lds\n\t" \
    "buffer_load_dwordx4 %[w7], %[rw], %[ow] offen offset:1024 lds\n\t" \
    "buffer_load_dwordx4 %[w8], %[rw], %[ow] offen offset:2048 lds\n\t" \
    "s_addk_i32 m0, 0x1000\n\t" \
    "s_mul_i32 %[t], %[id], %[mul]\n\t" \
    "s_add_u32 %[t], %[t], %[s0]\n\t" \
    "buffer_load_dwordx4 %[w10], %[rw], %[ow] offen offset:0 lds\n\t" \
    "buffer_load_dwordx4 %[w12], %[rw], %[ow] offen offset:2048 lds\n\t" \
    "s_addk_i32 m0, 0x1000\n\t" \
    "s_and_b32 %[oa], %[t], %[cut]\n\t" \
    "buffer_load_dwordx4 %[w14], %[rw], %[ow] offen offset:0 lds\n\t" \
    "buffer_load_dword %[zero], %[rd], %[od] offen offset:2048 lds\n\t" \
    "s_mov_b32 m0, 0\n\t" \
    "s_cmp_ge_i32 %[id], 0\n\t" \
    "s_cbranch_scc1 40b\n\t" \
    "20:\n\t" \
    "s_mov_b32 %[dst], m0\n\t" \
    "s_branch 9f\n\t" \
      \
    "4:\n\t" \
    "s_cmp_lt_u32 %[u], 2\n\t" \
    "s_cbranch_scc1 6f\n\t" \
    "s_bitcmp1_b32 %[dst], 16\n\t" \
    "s_cbranch_scc1 5f\n\t" \
    "s_bitcmp1_b32 %[dst], 15\n\t" \
    "s_cbranch_scc1 41b\n\t" \
    "s_branch 40b\n\t" \
    "5:\n\t" \
    "s_bitcmp1_b32 %[dst], 15\n\t" \
    "s_cbranch_scc1 43b\n\t" \
    "s_branch 42b\n\t" \
    "6:\n\t" \
    "buffer_load_dwordx4 %[l0], %[ra], %[oa] offen offset:0 sc0 sc1 lds\n\t" \
    "s_add_u32 %[dst], %[dst], %[slot]\n\t" \
    "s_bitset0_b32 %[dst], 17\n\t" \
    "s_add_u32 %[u], %[u], 1\n\t" \
    "buffer_load_dwordx4 %[l0], %[ra], %[oa] offen offset:1024 sc0 sc1 lds\n\t" \
    "buffer_load_dwordx4 %[l0], %[ra], %[oa] offen offset:2048 sc0 sc1 lds\n\t" \
    "buffer_load_dwordx4 %[l0], %[ra], %[oa] offen offset:3072 sc0 sc1 lds\n\t" \
    "s_addk_i32 m0, 0x1000\n\t" \
    "s_lshl_b32 %[ow], %[oa], 1\n\t" \
    "s_lshr_b32 %[od], %[oa], 16\n\t" \
    "buffer_load_dwordx4 %[l1], %[ra], %[oa] offen offset:0 sc0 sc1 lds\n\t" \
    "buffer_load_dwordx4 %[l1], %[ra], %[oa] offen offset:1024 sc0 sc1 lds\n\t" \
    "buffer_load_dwordx4 %[l1], %[ra], %[oa] offen offset:2048 sc0 sc1 lds\n\t" \
    "buffer_load_dwordx4 %[l1], %[ra], %[oa] offen offset:3072 sc0 sc1 lds\n\t" \
    "s_addk_i32 m0, 0x1000\n\t" \
    "s_bcnt1_i32_b32 %[t], %[rdy]\n\t" \
    "buffer_load_dwordx4 %[l2], %[ra], %[oa] offen offset:0 sc0 sc1 lds\n\t" \
    "buffer_load_dwordx4 %[l2], %[ra], %[oa] offen offset:1024 sc0 sc1 lds\n\t" \
    "buffer_load_dwordx4 %[w0], %[rw], %[ow] offen offset:2048 lds\n\t" \
    "buffer_load_dwordx4 %[w1], %[rw], %[ow] offen offset:3072 lds\n\t" \
    "s_addk_i32 m0, 0x1000\n\t" \
    "s_bitcmp1_b32 %[msk], %[t]\n\t" \
    "s_cbranch_scc1 64f\n\t" \
    "54:\n\t" \
    "buffer_load_dwordx4 %[w2], %[rw], %[ow] offen offset:0 lds\n\t" \
    "buffer_load_dwordx4 %[w3], %[rw], %[ow] offen offset:1024 lds\n\t" \
    "buffer_load_dwordx4 %[w4], %[rw], %[ow] offen offset:2048 lds\n\t" \
    "buffer_load_dwordx4 %[w5], %[rw], %[ow] offen offset:3072 lds\n\t" \
    "s_addk_i32 m0, 0x1000\n\t" \
    "s_ff1_i32_b32 %[id], %[rdy]\n\t" \
    "s_bitset0_b32 %[rdy], %[id]\n\t" \
    "buffer_load_dwordx4 %[w6], %[rw], %[ow] offen offset:0 lds\n\t" \
    "buffer_load_dwordx4 %[w7], %[rw], %[ow] offen offset:1024 lds\n\t" \
    "buffer_load_dwordx4 %[w8], %[rw], %[ow] offen offset:2048 lds\n\t" \
    "s_addk_i32 m0, 0x1000\n\t" \
    "s_mul_i32 %[t], %[id], %[mul]\n\t" \
    "s_add_u32 %[t], %[t], %[s0]\n\t" \
    "buffer_load_dwordx4 %[w10], %[rw], %[ow] offen offset:0 lds\n\t" \
    "buffer_load_dwordx4 %[w12], %[rw], %[ow] offen offset:2048 lds\n\t" \
    "s_addk_i32 m0, 0x1000\n\t" \
    "s_and_b32 %[oa], %[t], %[cut]\n\t" \
    "buffer_load_dwordx4 %[w14], %[rw], %[ow] offen offset:0 lds\n\t" \
    "buffer_load_dword %[zero], %[rd], %[od] offen offset:2048 lds\n\t" \
    "s_mov_b32 m0, %[dst]\n\t" \
    "s_cmp_ge_i32 %[id], 0\n\t" \
    "s_cbranch_scc1 4b\n\t" \
    "9:\n\t" \
    "s_mov_b32 %[xp], 0\n\t" \
    "s_mov_b32 %[np], 0\n\t" \
    "s_cmp_eq_u32 %[un], 0\n\t" \
    "s_cbranch_scc1 24f\n\t" \
      \
    "10:\n\t" \
    "s_getreg_b32 s68, hwreg(HW_REG_IB_STS)\n\t" \
    "s_lshr_b32 s69, s68, 18\n\t" \
    "s_and_b32 s69, s69, 0x30\n\t" \
    "s_and_b32 s68, s68, 15\n\t" \
    "s_or_b32 s68, s68, s69\n\t" \
    "s_cmp_eq_u32 %[u], 0\n\t" \
    "s_cbranch_scc1 12f\n\t" \
    "s_cmp_eq_u32 %[u], 2\n\t" \
    "s_cselect_b32 %[t], 23, 0\n\t" \
    "s_add_u32 %[t], %[t], %[xp]\n\t" \
    "s_min_u32 %[t], %[t], 63\n\t" \
    "s_cmp_gt_u32 s68, %[t]\n\t" \
    "s_cbranch_scc1 12f\n\t" \
    "s_getpc_b64 s[68:69]\n\t" \
    "s_lshl_b32 %[t], %[t], 3\n\t" \
    "s_add_u32 %[t], %[t], 20\n\t" \
    "s_add_u32 s68, s68, %[t]\n\t" \
    "s_addc_u32 s69, s69, 0\n\t" \
    "s_setpc_b64 s[68:69]\n\t" \
    "s_waitcnt vmcnt(0)\n\t" \
    "s_branch 13f\n\t" \
    "s_waitcnt vmcnt(1)\n\t" \
    "s_branch 13f\n\t" \
    "s_waitcnt vmcnt(2)\n\t" \
    "s_branch 13f\n\t" \
    "s_waitcnt vmcnt(3)\n\t" \
    "s_branch 13f\n\t" \
    "s_waitcnt vmcnt(4)\n\t" \
    "s_branch 13f\n\t" \
    "s_waitcnt vmcnt(5)\n\t" \
    "s_branch 13f\n\t" \
    "s_waitcnt vmcnt(6)\n\t" \
    "s_branch 13f\n\t" \
    "s_waitcnt vmcnt(7)\n\t" \
    "s_branch 13f\n\t" \
    "s_waitcnt vmcnt(8)\n\t" \
    "s_branch 13f\n\t" \
    "s_waitcnt vmcnt(9)\n\t" \
    "s_branch 13f\n\t" \
    "s_waitcnt vmcnt(10)\n\t" \
    "s_branch 13f\n\t" \
    "s_waitcnt vmcnt(11)\n\t" \
    "s_branch 13f\n\t" \
    "s_waitcnt vmcnt(12)\n\t" \
    "s_branch 13f\n\t" \
    "s_waitcnt vmcnt(13)\n\t" \
    "s_branch 13f\n\t" \
    "s_waitcnt vmcnt(14)\n\t" \
    "s_branch 13f\n\t" \
    "s_waitcnt vmcnt(15)\n\t" \
    "s_branch 13f\n\t" \
    "s_waitcnt vmcnt(16)\n\t" \
    "s_branch 13f\n\t" \
    "s_waitcnt vmcnt(17)\n\t" \
    "s_branch 13f\n\t" \
    "s_waitcnt vmcnt(18)\n\t" \
    "s_branch 13f\n\t" \
    "s_waitcnt vmcnt(19)\n\t" \
    "s_branch 13f\n\t" \
    "s_waitcnt vmcnt(20)\n\t" \
    "s_branch 13f\n\t" \
    "s_waitcnt vmcnt(21)\n\t" \
    "s_branch 13f\n\t" \
    "s_waitcnt vmcnt(22)\n\t" \
    "s_branch 13f\n\t" \
    "s_waitcnt vmcnt(23)\n\t" \
    "s_branch 13f\n\t" \
    "s_waitcnt vmcnt(24)\n\t" \
    "s_branch 13f\n\t" \
    "s_waitcnt vmcnt(25)\n\t" \
    "s_branch 13f\n\t" \
    "s_waitcnt vmcnt(26)\n\t" \
    "s_branch 13f\n\t" \
    "s_waitcnt vmcnt(27)\n\t" \
    "s_branch 13f\n\t" \
    "s_waitcnt vmcnt(28)\n\t" \
    "s_branch 13f\n\t" \
    "s_waitcnt vmcnt(29)\n\t" \
    "s_branch 13f\n\t" \
    "s_waitcnt vmcnt(30)\n\t" \
    "s_branch 13f\n\t" \
    "s_waitcnt vmcnt(31)\n\t" \
    "s_branch 13f\n\t" \
    "s_waitcnt vmcnt(32)\n\t" \
    "s_branch 13f\n\t" \
    "s_waitcnt vmcnt(33)\n\t" \
    "s_branch 13f\n\t" \
    "s_waitcnt vmcnt(34)\n\t" \
    "s_branch 13f\n\t" \
    "s_waitcnt vmcnt(35)\n\t" \
    "s_branch 13f\n\t" \
    "s_waitcnt vmcnt(36)\n\t" \
    "s_branch 13f\n\t" \
    "s_waitcnt vmcnt(37)\n\t" \
    "s_branch 13f\n\t" \
    "s_waitcnt vmcnt(38)\n\t" \
    "s_branch 13f\n\t" \
    "s_waitcnt vmcnt(39)\n\t" \
    "s_branch 13f\n\t" \
    "s_waitcnt vmcnt(40)\n\t" \
    "s_branch 13f\n\t" \
    "s_waitcnt vmcnt(41)\n\t" \
    "s_branch 13f\n\t" \
    "s_waitcnt vmcnt(42)\n\t" \
    "s_branch 13f\n\t" \
    "s_waitcnt vmcnt(43)\n\t" \
    "s_branch 13f\n\t" \
    "s_waitcnt vmcnt(44)\n\t" \
    "s_branch 13f\n\t" \
    "s_waitcnt vmcnt(45)\n\t" \
    "s_branch 13f\n\t" \
    "s_waitcnt vmcnt(46)\n\t" \
    "s_branch 13f\n\t" \
    "s_waitcnt vmcnt(47)\n\t" \
    "s_branch 13f\n\t" \
    "s_waitcnt vmcnt(48)\n\t" \
    "s_branch 13f\n\t" \
    "s_waitcnt vmcnt(49)\n\t" \
    "s_branch 13f\n\t" \
    "s_waitcnt vmcnt(50)\n\t" \
    "s_branch 13f\n\t" \
    "s_waitcnt vmcnt(51)\n\t" \
    "s_branch 13f\n\t" \
    "s_waitcnt vmcnt(52)\n\t" \
    "s_branch 13f\n\t" \
    "s_waitcnt vmcnt(53)\n\t" \
    "s_branch 13f\n\t" \
    "s_waitcnt vmcnt(54)\n\t" \
    "s_branch 13f\n\t" \
    "s_waitcnt vmcnt(55)\n\t" \
    "s_branch 13f\n\t" \
    "s_waitcnt vmcnt(56)\n\t" \
    "s_branch 13f\n\t" \
    "s_waitcnt vmcnt(57)\n\t" \
    "s_branch 13f\n\t" \
    "s_waitcnt vmcnt(58)\n\t" \
    "s_branch 13f\n\t" \
    "s_waitcnt vmcnt(59)\n\t" \
    "s_branch 13f\n\t" \
    "s_waitcnt vmcnt(60)\n\t" \
    "s_branch 13f\n\t" \
    "s_waitcnt vmcnt(61)\n\t" \
    "s_branch 13f\n\t" \
    "s_waitcnt vmcnt(62)\n\t" \
    "s_branch 13f\n\t" \
    "s_waitcnt vmcnt(63)\n\t" \
    "s_branch 13f\n\t" \
    "13:\n\t" \
    "s_barrier\n\t" \
    "s_sub_u32 %[u], %[u], 1\n\t" \
    "s_branch 8f\n\t" \
    "12:\n\t" \
    "s_cmp_ge_u32 %[wm], 1048576\n\t" \
    "s_cbranch_scc1 15f\n\t" \
    "s_add_u32 %[t], s68, 8\n\t" \
    "s_cmp_gt_i32 %[t], %[sw]\n\t" \
    "s_cbranch_scc1 15f\n\t" \
    "buffer_load_dword %[dump], %[c0], %[rt], %[wm] offen\n\t" \
    "buffer_load_dword %[dump], %[c1], %[rt], %[wm] offen\n\t" \
    "buffer_load_dword %[dump], %[c2], %[rt], %[wm] offen\n\t" \
    "s_add_u32 %[wm], %[wm], 0x100000\n\t" \
    "s_add_u32 %[xp], %[xp], 3\n\t" \
    "s_mov_b32 %[sw], 0\n\t" \
    "s_mov_b32 %[np], 0\n\t" \
    "15:\n\t" \
    "s_min_u32 %[t], s68, %[np]\n\t" \
    "s_cmp_ge_u32 %[t], 2\n\t" \
    "s_cbranch_scc1 17f\n\t" \
    "s_mov_b32 exec_hi, 0\n\t" \
    "global_load_dword %[f], %[foff], %[ws] sc0 sc1\n\t" \
    "s_mov_b32 exec_hi, -1\n\t" \
    "s_add_u32 %[xp], %[xp], 1\n\t" \
    "s_add_u32 %[sw], %[sw], 1\n\t" \
    "s_add_u32 %[np], %[np], 1\n\t" \
    "17:\n\t" \
    "s_memrealtime s[68:69]\n\t" \
    "v_readlane_b32 %[t], %[f], %[lt]\n\t" \
    "s_waitcnt lgkmcnt(0)\n\t" \
    "s_cmp_eq_u32 %[t], %[call]\n\t" \
    "s_cbranch_scc1 18f\n\t" \
    "s_sub_u32 s68, s68, %[t0]\n\t" \
    "s_cmp_gt_u32 s68, %[dl]\n\t" \
    "s_cbranch_scc1 18f\n\t" \
    "8:\n\t" \
    "v_cmp_ge_u32 vcc, %[f], %[vcall]\n\t" \
    "s_and_b32 %[t], vcc_lo, %[un]\n\t" \
    "s_or_b32 %[rdy], %[rdy], %[t]\n\t" \
    "s_andn2_b32 %[un], %[un], vcc_lo\n\t" \
    "s_cselect_b32 %[msk], %[msk], 0\n\t" \
    "s_ff1_i32_b32 %[id], %[rdy]\n\t" \
    "s_bitset0_b32 %[rdy], %[id]\n\t" \
    "s_mul_i32 %[t], %[id], %[mul]\n\t" \
    "s_add_u32 %[t], %[t], %[s0]\n\t" \
    "s_and_b32 %[oa], %[t], %[cut]\n\t" \
    "s_cmp_lt_i32 %[id], 0\n\t" \
    "s_cbranch_scc1 10b\n\t" \
    "s_branch 4b\n\t" \
      \
    "60:\n\t" \
    "v_cmp_ge_u32 vcc, %[f], %[vcall]\n\t" \
    "s_and_b32 %[t], vcc_lo, %[un]\n\t" \
    "s_or_b32 %[rdy], %[rdy], %[t]\n\t" \
    "s_andn2_b32 %[un], %[un], vcc_lo\n\t" \
    "s_cselect_b32 %[msk], %[msk], 0\n\t" \
    "s_cbranch_scc0 50b\n\t" \
    "s_bcnt1_i32_b32 %[t], %[rdy]\n\t" \
    "s_cmp_gt_u32 %[t], 5\n\t" \
    "s_cbranch_scc1 50b\n\t" \
    "s_mov_b32 exec_hi, 0\n\t" \
    "global_load_dword %[f], %[foff], %[ws] sc0 sc1\n\t" \
    "s_mov_b32 exec_hi, -1\n\t" \
    "s_branch 50b\n\t" \
    "61:\n\t" \
    "v_cmp_ge_u32 vcc, %[f], %[vcall]\n\t" \
    "s_and_b32 %[t], vcc_lo, %[un]\n\t" \
    "s_or_b32 %[rdy], %[rdy], %[t]\n\t" \
    "s_andn2_b32 %[un], %[un], vcc_lo\n\t" \
    "s_cselect_b32 %[msk], %[msk], 0\n\t" \
    "s_cbranch_scc0 51b\n\t" \
    "s_bcnt1_i32_b32 %[t], %[rdy]\n\t" \
    "s_cmp_gt_u32 %[t], 5\n\t" \
    "s_cbranch_scc1 51b\n\t" \
    "s_mov_b32 exec_hi, 0\n\t" \
    "global_load_dword %[f], %[foff], %[ws] sc0 sc1\n\t" \
    "s_mov_b32 exec_hi, -1\n\t" \
    "s_branch 51b\n\t" \
    "62:\n\t" \
    "v_cmp_ge_u32 vcc, %[f], %[vcall]\n\t" \
    "s_and_b32 %[t], vcc_lo, %[un]\n\t" \
    "s_or_b32 %[rdy], %[rdy], %[t]\n\t" \
    "s_andn2_b32 %[un], %[un], vcc_lo\n\t" \
    "s_cselect_b32 %[msk], %[msk], 0\n\t" \
    "s_cbranch_scc0 52b\n\t" \
    "s_bcnt1_i32_b32 %[t], %[rdy]\n\t" \
    "s_cmp_gt_u32 %[t], 5\n\t" \
    "s_cbranch_scc1 52b\n\t" \
    "s_mov_b32 exec_hi, 0\n\t" \
    "global_load_dword %[f], %[foff], %[ws] sc0 sc1\n\t" \
    "s_mov_b32 exec_hi, -1\n\t" \
    "s_branch 52b\n\t" \
    "63:\n\t" \
    "v_cmp_ge_u32 vcc, %[f], %[vcall]\n\t" \
    "s_and_b32 %[t], vcc_lo, %[un]\n\t" \
    "s_or_b32 %[rdy], %[rdy], %[t]\n\t" \
    "s_andn2_b32 %[un], %[un], vcc_lo\n\t" \
    "s_cselect_b32 %[msk], %[msk], 0\n\t" \
    "s_cbranch_scc0 53b\n\t" \
    "s_bcnt1_i32_b32 %[t], %[rdy]\n\t" \
    "s_cmp_gt_u32 %[t], 5\n\t" \
    "s_cbranch_scc1 53b\n\t" \
    "s_mov_b32 exec_hi, 0\n\t" \
    "global_load_dword %[f], %[foff], %[ws] sc0 sc1\n\t" \
    "s_mov_b32 exec_hi, -1\n\t" \
    "s_branch 53b\n\t" \
    "64:\n\t" \
    "v_cmp_ge_u32 vcc, %[f], %[vcall]\n\t" \
    "s_and_b32 %[t], vcc_lo, %[un]\n\t" \
    "s_or_b32 %[rdy], %[rdy], %[t]\n\t" \
    "s_andn2_b32 %[un], %[un], vcc_lo\n\t" \
    "s_cselect_b32 %[msk], %[msk], 0\n\t" \
    "s_cbranch_scc0 54b\n\t" \
    "s_bcnt1_i32_b32 %[t], %[rdy]\n\t" \
    "s_cmp_gt_u32 %[t], 5\n\t" \
    "s_cbranch_scc1 54b\n\t" \
    "s_mov_b32 exec_hi, 0\n\t" \
    "global_load_dword %[f], %[foff], %[ws] sc0 sc1\n\t" \
    "s_mov_b32 exec_hi, -1\n\t" \
    "s_branch 54b\n\t" \
      \
    "7:\n\t" \
    "s_cmp_eq_u32 %[xp], 5\n\t" \
    "s_cbranch_scc1 11f\n\t" \
    "s_mov_b32 exec_hi, 0\n\t" \
    "global_load_dword %[f], %[foff], %[ws] sc0 sc1\n\t" \
    "s_mov_b32 exec_hi, -1\n\t" \
    "s_branch 3b\n\t" \
    "11:\n\t" \
    "v_cmp_ge_u32 vcc, %[f], %[vcall]\n\t" \
    "s_and_b32 %[t], vcc_lo, %[un]\n\t" \
    "s_bcnt1_i32_b32 %[t], %[t]\n\t" \
    "s_cmp_gt_u32 %[t], 1\n\t" \
    "s_cbranch_scc1 3b\n\t" \
    "buffer_load_dword %[dump], %[c0], %[rt], %[wm] offen\n\t" \
    "buffer_load_dword %[dump], %[c1], %[rt], %[wm] offen\n\t" \
    "buffer_load_dword %[dump], %[c2], %[rt], %[wm] offen\n\t" \
    "s_add_u32 %[wm], %[wm], 0x100000\n\t" \
    "s_mov_b32 %[sw], 0\n\t" \
    "s_branch 3b\n\t" \
      \
    "24:\n\t" \
    "v_readlane_b32 %[t], %[f], %[lt]\n\t" \
    "s_cmp_eq_u32 %[t], %[call]\n\t" \
    "s_cbranch_scc1 18f\n\t" \
    "s_cmp_eq_u32 %[u], 1\n\t" \
    "s_cbranch_scc1 25f\n\t" \
    "s_waitcnt vmcnt(23)\n\t" \
    "s_barrier\n\t" \
    "25:\n\t" \
    "s_waitcnt vmcnt(0)\n\t" \
    "s_barrier\n\t" \
    "s_mov_b32 %[u], 0\n\t" \
    "s_branch 19f\n\t" \
    "18:\n\t" \
    "s_mov_b32 %[u], 1\n\t" \
    "s_waitcnt vmcnt(0)\n\t" \
    "19:\n\t"

#define LOADER4_ASM \
    "s_mov_b32 %[u], 1\n\t" \
    "s_mov_b32 %[dst], %[slot]\n\t" \
    "s_mov_b32 %[xp], 1\n\t" \
    "s_mov_b32 m0, %[slot]\n\t" \
    "s_mov_b32 %[wm], 0\n\t" \
      \
    "1:\n\t" \
    "global_load_dword %[f], %[foff], %[ws] sc0 sc1\n\t" \
    "buffer_load_dwordx4 %[a0], %[rx], %[oa] offen offset:0 lds\n\t" \
    "s_lshl_b32 %[ow], %[oa], 12\n\t" \
    "s_add_u32 %[ow], %[ow], %[own]\n\t" \
    "buffer_load_dwordx4 %[a1], %[rx], %[oa] offen offset:1024 lds\n\t" \
    "buffer_load_dwordx4 %[a2], %[rx], %[oa] offen offset:2048 lds\n\t" \
    "buffer_load_dwordx4 %[a3], %[rx], %[oa] offen offset:3072 lds\n\t" \
    "s_addk_i32 m0, 0x1000\n\t" \
    "s_add_u32 %[dst], %[dst], %[slot]\n\t" \
    "s_bitset0_b32 %[dst], 17\n\t" \
    "buffer_load_dwordx4 %[a4], %[rx], %[oa] offen offset:0 lds\n\t" \
    "buffer_load_dwordx4 %[a5], %[rx], %[oa] offen offset:1024 lds\n\t" \
    "buffer_load_dwordx4 %[a6], %[rx], %[oa] offen offset:2048 lds\n\t" \
    "buffer_load_dwordx4 %[a7], %[rx], %[oa] offen offset:3072 lds\n\t" \
    "s_addk_i32 m0, 0x1000\n\t" \
    "s_nop 0\n\t" \
    "buffer_load_dwordx4 %[w0], %[rw], %[ow] offen offset:2048 lds\n\t" \
    "buffer_load_dwordx4 %[w1], %[rw], %[ow] offen offset:3072 lds\n\t" \
    "s_addk_i32 m0, 0x1000\n\t" \
    "s_addk_i32 %[oa], 128\n\t" \
    "s_bitset0_b32 %[oa], 9\n\t" \
    "buffer_load_dwordx4 %[w2], %[rw], %[ow] offen offset:0 lds\n\t" \
    "buffer_load_dwordx4 %[w3], %[rw], %[ow] offen offset:1024 lds\n\t" \
    "buffer_load_dwordx4 %[w4], %[rw], %[ow] offen offset:2048 lds\n\t" \
    "buffer_load_dwordx4 %[w5], %[rw], %[ow] offen offset:3072 lds\n\t" \
    "s_addk_i32 m0, 0x1000\n\t" \
    "s_add_u32 %[u], %[u], 1\n\t" \
    "buffer_load_dwordx4 %[w6], %[rw], %[ow] offen offset:0 lds\n\t" \
    "buffer_load_dwordx4 %[w7], %[rw], %[ow] offen offset:1024 lds\n\t" \
    "buffer_load_dwordx4 %[w8], %[rw], %[ow] offen offset:2048 lds\n\t" \
    "buffer_load_dwordx4 %[w9], %[rw], %[ow] offen offset:3072 lds\n\t" \
    "s_addk_i32 m0, 0x1000\n\t" \
    "s_add_u32 %[xp], %[xp], 1\n\t" \
    "buffer_load_dwordx4 %[w10], %[rw], %[ow] offen offset:0 lds\n\t" \
    "buffer_load_dwordx4 %[w11], %[rw], %[ow] offen offset:1024 lds\n\t" \
    "buffer_load_dwordx4 %[w12], %[rw], %[ow] offen offset:2048 lds\n\t" \
    "buffer_load_dwordx4 %[w13], %[rw], %[ow] offen offset:3072 lds\n\t" \
    "s_addk_i32 m0, 0x1000\n\t" \
    "s_nop 0\n\t" \
    "buffer_load_dwordx4 %[w14], %[rw], %[ow] offen offset:0 lds\n\t" \
    "buffer_load_dwordx4 %[w15], %[rw], %[ow] offen offset:1024 lds\n\t" \
    "s_mov_b32 m0, %[dst]\n\t" \
    "s_cmp_le_u32 %[u], 2\n\t" \
    "s_cbranch_scc1 1b\n\t" \
    "s_waitcnt vmcnt(48)\n\t" \
    "s_barrier\n\t" \
    "s_mov_b32 %[u], 2\n\t" \
    "s_cmp_lt_u32 %[xp], 4\n\t" \
    "s_cbranch_scc1 1b\n\t" \
      \
    "s_mov_b32 %[xp], 0\n\t" \
    "s_lshl_b32 %[ow], %[s0], 1\n\t" \
    "v_readlane_b32 %[call], %[f], %[lc]\n\t" \
    "s_addk_i32 m0, 0x2000\n\t" \
    "s_movk_i32 %[sw], 0x100\n\t" \
    "s_add_u32 %[call], %[call], 1\n\t" \
    "v_mov_b32 %[vcall], " VCALL "\n\t" \
    "2:\n\t" \
    "buffer_load_dwordx4 %[k0], %[rw], %[ow] offen offset:2048 lds\n\t" \
    "buffer_load_dwordx4 %[k1], %[rw], %[ow] offen offset:3072 lds\n\t" \
    "s_addk_i32 m0, 0x1000\n\t" \
    "s_add_u32 %[dst], %[dst], %[slot]\n\t" \
    "s_bitset0_b32 %[dst], 17\n\t" \
    "buffer_load_dwordx4 %[k2], %[rw], %[ow] offen offset:0 lds\n\t" \
    "buffer_load_dwordx4 %[k3], %[rw], %[ow] offen offset:1024 lds\n\t" \
    "buffer_load_dwordx4 %[k4], %[rw], %[ow] offen offset:2048 lds\n\t" \
    "buffer_load_dwordx4 %[k5], %[rw], %[ow] offen offset:3072 lds\n\t" \
    "s_addk_i32 m0, 0x1000\n\t" \
    "s_add_u32 %[xp], %[xp], 1\n\t" \
    "s_add_u32 %[t], %[ow], 0x200000\n\t" \
    "s_bitset0_b32 %[t], 24\n\t" \
    "buffer_load_dwordx4 %[k6], %[rw], %[ow] offen offset:0 lds\n\t" \
    "buffer_load_dwordx4 %[k7], %[rw], %[ow] offen offset:1024 lds\n\t" \
    "buffer_load_dwordx4 %[k8], %[rw], %[ow] offen offset:2048 lds\n\t" \
    "buffer_load_dwordx4 %[k9], %[rw], %[ow] offen offset:3072 lds\n\t" \
    "s_addk_i32 m0, 0x1000\n\t" \
    "s_cmp_eq_u32 %[t], %[own]\n\t" \
    "s_cselect_b32 %[od], 0x200000, 0\n\t" \
    "buffer_load_dwordx4 %[k10], %[rw], %[ow] offen offset:0 lds\n\t" \
    "buffer_load_dwordx4 %[k11], %[rw], %[ow] offen offset:1024 lds\n\t" \
    "buffer_load_dwordx4 %[k12], %[rw], %[ow] offen offset:2048 lds\n\t" \
    "buffer_load_dwordx4 %[k13], %[rw], %[ow] offen offset:3072 lds\n\t" \
    "s_addk_i32 m0, 0x1000\n\t" \
    "s_add_u32 %[t], %[t], %[od]\n\t" \
    "s_bitset0_b32 %[t], 24\n\t" \
    "buffer_load_dwordx4 %[k14], %[rw], %[ow] offen offset:0 lds\n\t" \
    "buffer_load_dwordx4 %[k15], %[rw], %[ow] offen offset:1024 lds\n\t" \
    "s_mov_b32 %[ow], %[t]\n\t" \
    "s_add_u32 m0, %[dst], 0x2000\n\t" \
    "s_bitcmp1_b32 232, %[xp]\n\t" \
    "s_cbranch_scc1 7f\n\t" \
    "3:\n\t" \
    "s_cmp_lt_u32 %[xp], 7\n\t" \
    "s_waitcnt vmcnt(32)\n\t" \
    "s_barrier\n\t" \
    "s_cbranch_scc1 2b\n\t" \
    "s_mov_b32 m0, %[dst]\n\t" \
    "s_mov_b32 %[rdy], 0\n\t" \
    "s_and_b32 %[t], %[call], 1\n\t" \
    "s_lshl_b32 %[t], %[t], 16\n\t" \
    "v_add_u32 %[l0], %[t], %[l0]\n\t" \
    "v_add_u32 %[l1], %[t], %[l1]\n\t" \
    "v_add_u32 %[l2], %[t], %[l2]\n\t" \
    "s_mov_b32 %[xp], 0\n\t" \
    "s_mov_b32 %[np], 0\n\t" \
    "s_mov_b32 %[tp], 0\n\t" \
    "s_movk_i32 %[msk], 39\n\t" \
    "s_waitcnt vmcnt(16)\n\t" \
    "s_barrier\n\t" \
    "s_mov_b32 %[u], 1\n\t" \
    "s_branch 8f\n\t" \
      \
    ".p2align 6\n\t" \
    "40:\n\t" \
    "buffer_load_dwordx4 %[l0], %[ra], %[oa] offen offset:0 sc0 sc1 lds\n\t" \
    "buffer_load_dwordx4 %[l0], %[ra], %[oa] offen offset:1024 sc0 sc1 lds\n\t" \
    "buffer_load_dwordx4 %[l0], %[ra], %[oa] offen offset:2048 sc0 sc1 lds\n\t" \
    "buffer_load_dwordx4 %[l0], %[ra], %[oa] offen offset:3072 sc0 sc1 lds\n\t" \
    "s_addk_i32 m0, 0x1000\n\t" \
    "s_lshl_b32 %[ow], %[oa], 1\n\t" \
    "s_lshr_b32 %[od], %[oa], 16\n\t" \
    "buffer_load_dwordx4 %[l1], %[ra], %[oa] offen offset:0 sc0 sc1 lds\n\t" \
    "buffer_load_dwordx4 %[l1], %[ra], %[oa] offen offset:1024 sc0 sc1 lds\n\t" \
    "buffer_load_dwordx4 %[l1], %[ra], %[oa] offen offset:2048 sc0 sc1 lds\n\t" \
    "buffer_load_dwordx4 %[l1], %[ra], %[oa] offen offset:3072 sc0 sc1 lds\n\t" \
    "s_addk_i32 m0, 0x1000\n\t" \
    "s_bcnt1_i32_b32 %[t], %[rdy]\n\t" \
    "buffer_load_dwordx4 %[w0], %[rw], %[ow] offen offset:2048 lds\n\t" \
    "buffer_load_dwordx4 %[w1], %[rw], %[ow] offen offset:3072 lds\n\t" \
    "s_addk_i32 m0, 0x1000\n\t" \
    "s_bitcmp1_b32 %[msk], %[t]\n\t" \
    "s_cbranch_scc1 60f\n\t" \
    "50:\n\t" \
    "buffer_load_dwordx4 %[w2], %[rw], %[ow] offen offset:0 lds\n\t" \
    "buffer_load_dwordx4 %[w3], %[rw], %[ow] offen offset:1024 lds\n\t" \
    "buffer_load_dwordx4 %[w4], %[rw], %[ow] offen offset:2048 lds\n\t" \
    "s_waitcnt vmcnt(34)\n\t" \
    "s_barrier\n\t" \
    "buffer_load_dwordx4 %[w5], %[rw], %[ow] offen offset:3072 lds\n\t" \
    "s_addk_i32 m0, 0x1000\n\t" \
    "s_ff1_i32_b32 %[id], %[rdy]\n\t" \
    "s_bitset0_b32 %[rdy], %[id]\n\t" \
    "buffer_load_dwordx4 %[w6], %[rw], %[ow] offen offset:0 lds\n\t" \
    "buffer_load_dwordx4 %[w7], %[rw], %[ow] offen offset:1024 lds\n\t" \
    "buffer_load_dwordx4 %[w8], %[rw], %[ow] offen offset:2048 lds\n\t" \
    "s_addk_i32 m0, 0x1000\n\t" \
    "s_mul_i32 %[t], %[id], %[mul]\n\t" \
    "s_add_u32 %[t], %[t], %[s0]\n\t" \
    "buffer_load_dwordx4 %[w10], %[rw], %[ow] offen offset:0 lds\n\t" \
    "buffer_load_dwordx4 %[w12], %[rw], %[ow] offen offset:2048 lds\n\t" \
    "s_addk_i32 m0, 0x1000\n\t" \
    "s_and_b32 %[oa], %[t], %[cut]\n\t" \
    "buffer_load_dwordx4 %[w14], %[rw], %[ow] offen offset:0 lds\n\t" \
    "buffer_load_dword %[zero], %[rd], %[od] offen offset:2048 lds\n\t" \
    "s_addk_i32 m0, 0x2000\n\t" \
    "s_cmp_lt_i32 %[id], 0\n\t" \
    "s_cbranch_scc1 20f\n\t" \
    "41:\n\t" \
    "buffer_load_dwordx4 %[l0], %[ra], %[oa] offen offset:0 sc0 sc1 lds\n\t" \
    "buffer_load_dwordx4 %[l0], %[ra], %[oa] offen offset:1024 sc0 sc1 lds\n\t" \
    "buffer_load_dwordx4 %[l0], %[ra], %[oa] offen offset:2048 sc0 sc1 lds\n\t" \
    "buffer_load_dwordx4 %[l0], %[ra], %[oa] offen offset:3072 sc0 sc1 lds\n\t" \
    "s_addk_i32 m0, 0x1000\n\t" \
    "s_lshl_b32 %[ow], %[oa], 1\n\t" \
    "s_lshr_b32 %[od], %[oa], 16\n\t" \
    "buffer_load_dwordx4 %[l1], %[ra], %[oa] offen offset:0 sc0 sc1 lds\n\t" \
    "buffer_load_dwordx4 %[l1], %[ra], %[oa] offen offset:1024 sc0 sc1 lds\n\t" \
    "buffer_load_dwordx4 %[l1], %[ra], %[oa] offen offset:2048 sc0 sc1 lds\n\t" \
    "buffer_load_dwordx4 %[l1], %[ra], %[oa] offen offset:3072 sc0 sc1 lds\n\t" \
    "s_addk_i32 m0, 0x1000\n\t" \
    "s_bcnt1_i32_b32 %[t], %[rdy]\n\t" \
    "buffer_load_dwordx4 %[w0], %[rw], %[ow] offen offset:2048 lds\n\t" \
    "buffer_load_dwordx4 %[w1], %[rw], %[ow] offen offset:3072 lds\n\t" \
    "s_addk_i32 m0, 0x1000\n\t" \
    "s_bitcmp1_b32 %[msk], %[t]\n\t" \
    "s_cbranch_scc1 61f\n\t" \
    "51:\n\t" \
    "buffer_load_dwordx4 %[w2], %[rw], %[ow] offen offset:0 lds\n\t" \
    "buffer_load_dwordx4 %[w3], %[rw], %[ow] offen offset:1024 lds\n\t" \
    "buffer_load_dwordx4 %[w4], %[rw], %[ow] offen offset:2048 lds\n\t" \
    "s_waitcnt vmcnt(34)\n\t" \
    "s_barrier\n\t" \
    "buffer_load_dwordx4 %[w5], %[rw], %[ow] offen offset:3072 lds\n\t" \
    "s_addk_i32 m0, 0x1000\n\t" \
    "s_ff1_i32_b32 %[id], %[rdy]\n\t" \
    "s_bitset0_b32 %[rdy], %[id]\n\t" \
    "buffer_load_dwordx4 %[w6], %[rw], %[ow] offen offset:0 lds\n\t" \
    "buffer_load_dwordx4 %[w7], %[rw], %[ow] offen offset:1024 lds\n\t" \
    "buffer_load_dwordx4 %[w8], %[rw], %[ow] offen offset:2048 lds\n\t" \
    "s_addk_i32 m0, 0x1000\n\t" \
    "s_mul_i32 %[t], %[id], %[mul]\n\t" \
    "s_add_u32 %[t], %[t], %[s0]\n\t" \
    "buffer_load_dwordx4 %[w10], %[rw], %[ow] offen offset:0 lds\n\t" \
    "buffer_load_dwordx4 %[w12], %[rw], %[ow] offen offset:2048 lds\n\t" \
    "s_addk_i32 m0, 0x1000\n\t" \
    "s_and_b32 %[oa], %[t], %[cut]\n\t" \
    "buffer_load_dwordx4 %[w14], %[rw], %[ow] offen offset:0 lds\n\t" \
    "buffer_load_dword %[zero], %[rd], %[od] offen offset:2048 lds\n\t" \
    "s_addk_i32 m0, 0x2000\n\t" \
    "s_cmp_lt_i32 %[id], 0\n\t" \
    "s_cbranch_scc1 20f\n\t" \
    "42:\n\t" \
    "buffer_load_dwordx4 %[l0], %[ra], %[oa] offen offset:0 sc0 sc1 lds\n\t" \
    "buffer_load_dwordx4 %[l0], %[ra], %[oa] offen offset:1024 sc0 sc1 lds\n\t" \
    "buffer_load_dwordx4 %[l0], %[ra], %[oa] offen offset:2048 sc0 sc1 lds\n\t" \
    "buffer_load_dwordx4 %[l0], %[ra], %[oa] offen offset:3072 sc0 sc1 lds\n\t" \
    "s_addk_i32 m0, 0x1000\n\t" \
    "s_lshl_b32 %[ow], %[oa], 1\n\t" \
    "s_lshr_b32 %[od], %[oa], 16\n\t" \
    "buffer_load_dwordx4 %[l1], %[ra], %[oa] offen offset:0 sc0 sc1 lds\n\t" \
    "buffer_load_dwordx4 %[l1], %[ra], %[oa] offen offset:1024 sc0 sc1 lds\n\t" \
    "buffer_load_dwordx4 %[l1], %[ra], %[oa] offen offset:2048 sc0 sc1 lds\n\t" \
    "buffer_load_dwordx4 %[l1], %[ra], %[oa] offen offset:3072 sc0 sc1 lds\n\t" \
    "s_addk_i32 m0, 0x1000\n\t" \
    "s_bcnt1_i32_b32 %[t], %[rdy]\n\t" \
    "buffer_load_dwordx4 %[w0], %[rw], %[ow] offen offset:2048 lds\n\t" \
    "buffer_load_dwordx4 %[w1], %[rw], %[ow] offen offset:3072 lds\n\t" \
    "s_addk_i32 m0, 0x1000\n\t" \
    "s_bitcmp1_b32 %[msk], %[t]\n\t" \
    "s_cbranch_scc1 62f\n\t" \
    "52:\n\t" \
    "buffer_load_dwordx4 %[w2], %[rw], %[ow] offen offset:0 lds\n\t" \
    "buffer_load_dwordx4 %[w3], %[rw], %[ow] offen offset:1024 lds\n\t" \
    "buffer_load_dwordx4 %[w4], %[rw], %[ow] offen offset:2048 lds\n\t" \
    "s_waitcnt vmcnt(34)\n\t" \
    "s_barrier\n\t" \
    "buffer_load_dwordx4 %[w5], %[rw], %[ow] offen offset:3072 lds\n\t" \
    "s_addk_i32 m0, 0x1000\n\t" \
    "s_ff1_i32_b32 %[id], %[rdy]\n\t" \
    "s_bitset0_b32 %[rdy], %[id]\n\t" \
    "buffer_load_dwordx4 %[w6], %[rw], %[ow] offen offset:0 lds\n\t" \
    "buffer_load_dwordx4 %[w7], %[rw], %[ow] offen offset:1024 lds\n\t" \
    "buffer_load_dwordx4 %[w8], %[rw], %[ow] offen offset:2048 lds\n\t" \
    "s_addk_i32 m0, 0x1000\n\t" \
    "s_mul_i32 %[t], %[id], %[mul]\n\t" \
    "s_add_u32 %[t], %[t], %[s0]\n\t" \
    "buffer_load_dwordx4 %[w10], %[rw], %[ow] offen offset:0 lds\n\t" \
    "buffer_load_dwordx4 %[w12], %[rw], %[ow] offen offset:2048 lds\n\t" \
    "s_addk_i32 m0, 0x1000\n\t" \
    "s_and_b32 %[oa], %[t], %[cut]\n\t" \
    "buffer_load_dwordx4 %[w14], %[rw], %[ow] offen offset:0 lds\n\t" \
    "buffer_load_dword %[zero], %[rd], %[od] offen offset:2048 lds\n\t" \
    "s_addk_i32 m0, 0x2000\n\t" \
    "s_cmp_lt_i32 %[id], 0\n\t" \
    "s_cbranch_scc1 20f\n\t" \
    "43:\n\t" \
    "buffer_load_dwordx4 %[l0], %[ra], %[oa] offen offset:0 sc0 sc1 lds\n\t" \
    "buffer_load_dwordx4 %[l0], %[ra], %[oa] offen offset:1024 sc0 sc1 lds\n\t" \
    "buffer_load_dwordx4 %[l0], %[ra], %[oa] offen offset:2048 sc0 sc1 lds\n\t" \
    "buffer_load_dwordx4 %[l0], %[ra], %[oa] offen offset:3072 sc0 sc1 lds\n\t" \
    "s_addk_i32 m0, 0x1000\n\t" \
    "s_lshl_b32 %[ow], %[oa], 1\n\t" \
    "s_lshr_b32 %[od], %[oa], 16\n\t" \
    "buffer_load_dwordx4 %[l1], %[ra], %[oa] offen offset:0 sc0 sc1 lds\n\t" \
    "buffer_load_dwordx4 %[l1], %[ra], %[oa] offen offset:1024 sc0 sc1 lds\n\t" \
    "buffer_load_dwordx4 %[l1], %[ra], %[oa] offen offset:2048 sc0 sc1 lds\n\t" \
    "buffer_load_dwordx4 %[l1], %[ra], %[oa] offen offset:3072 sc0 sc1 lds\n\t" \
    "s_addk_i32 m0, 0x1000\n\t" \
    "s_bcnt1_i32_b32 %[t], %[rdy]\n\t" \
    "buffer_load_dwordx4 %[w0], %[rw], %[ow] offen offset:2048 lds\n\t" \
    "buffer_load_dwordx4 %[w1], %[rw], %[ow] offen offset:3072 lds\n\t" \
    "s_addk_i32 m0, 0x1000\n\t" \
    "s_bitcmp1_b32 %[msk], %[t]\n\t" \
    "s_cbranch_scc1 63f\n\t" \
    "53:\n\t" \
    "buffer_load_dwordx4 %[w2], %[rw], %[ow] offen offset:0 lds\n\t" \
    "buffer_load_dwordx4 %[w3], %[rw], %[ow] offen offset:1024 lds\n\t" \
    "buffer_load_dwordx4 %[w4], %[rw], %[ow] offen offset:2048 lds\n\t" \
    "s_waitcnt vmcnt(34)\n\t" \
    "s_barrier\n\t" \
    "buffer_load_dwordx4 %[w5], %[rw], %[ow] offen offset:3072 lds\n\t" \
    "s_addk_i32 m0, 0x1000\n\t" \
    "s_ff1_i32_b32 %[id], %[rdy]\n\t" \
    "s_bitset0_b32 %[rdy], %[id]\n\t" \
    "buffer_load_dwordx4 %[w6], %[rw], %[ow] offen offset:0 lds\n\t" \
    "buffer_load_dwordx4 %[w7], %[rw], %[ow] offen offset:1024 lds\n\t" \
    "buffer_load_dwordx4 %[w8], %[rw], %[ow] offen offset:2048 lds\n\t" \
    "s_addk_i32 m0, 0x1000\n\t" \
    "s_mul_i32 %[t], %[id], %[mul]\n\t" \
    "s_add_u32 %[t], %[t], %[s0]\n\t" \
    "buffer_load_dwordx4 %[w10], %[rw], %[ow] offen offset:0 lds\n\t" \
    "buffer_load_dwordx4 %[w12], %[rw], %[ow] offen offset:2048 lds\n\t" \
    "s_addk_i32 m0, 0x1000\n\t" \
    "s_and_b32 %[oa], %[t], %[cut]\n\t" \
    "buffer_load_dwordx4 %[w14], %[rw], %[ow] offen offset:0 lds\n\t" \
    "buffer_load_dword %[zero], %[rd], %[od] offen offset:2048 lds\n\t" \
    "s_mov_b32 m0, 0\n\t" \
    "s_cmp_ge_i32 %[id], 0\n\t" \
    "s_cbranch_scc1 40b\n\t" \
    "20:\n\t" \
    "s_mov_b32 %[dst], m0\n\t" \
    "s_branch 9f\n\t" \
      \
    "4:\n\t" \
    "s_cmp_lt_u32 %[u], 2\n\t" \
    "s_cbranch_scc1 6f\n\t" \
    "s_bitcmp1_b32 %[dst], 16\n\t" \
    "s_cbranch_scc1 5f\n\t" \
    "s_bitcmp1_b32 %[dst], 15\n\t" \
    "s_cbranch_scc1 41b\n\t" \
    "s_branch 40b\n\t" \
    "5:\n\t" \
    "s_bitcmp1_b32 %[dst], 15\n\t" \
    "s_cbranch_scc1 43b\n\t" \
    "s_branch 42b\n\t" \
    "6:\n\t" \
    "buffer_load_dwordx4 %[l0], %[ra], %[oa] offen offset:0 sc0 sc1 lds\n\t" \
    "s_add_u32 %[dst], %[dst], %[slot]\n\t" \
    "s_bitset0_b32 %[dst], 17\n\t" \
    "s_add_u32 %[u], %[u], 1\n\t" \
    "buffer_load_dwordx4 %[l0], %[ra], %[oa] offen offset:1024 sc0 sc1 lds\n\t" \
    "buffer_load_dwordx4 %[l0], %[ra], %[oa] offen offset:2048 sc0 sc1 lds\n\t" \
    "buffer_load_dwordx4 %[l0], %[ra], %[oa] offen offset:3072 sc0 sc1 lds\n\t" \
    "s_addk_i32 m0, 0x1000\n\t" \
    "s_lshl_b32 %[ow], %[oa], 1\n\t" \
    "s_lshr_b32 %[od], %[oa], 16\n\t" \
    "buffer_load_dwordx4 %[l1], %[ra], %[oa] offen offset:0 sc0 sc1 lds\n\t" \
    "buffer_load_dwordx4 %[l1], %[ra], %[oa] offen offset:1024 sc0 sc1 lds\n\t" \
    "buffer_load_dwordx4 %[l1], %[ra], %[oa] offen offset:2048 sc0 sc1 lds\n\t" \
    "buffer_load_dwordx4 %[l1], %[ra], %[oa] offen offset:3072 sc0 sc1 lds\n\t" \
    "s_addk_i32 m0, 0x1000\n\t" \
    "s_bcnt1_i32_b32 %[t], %[rdy]\n\t" \
    "buffer_load_dwordx4 %[w0], %[rw], %[ow] offen offset:2048 lds\n\t" \
    "buffer_load_dwordx4 %[w1], %[rw], %[ow] offen offset:3072 lds\n\t" \
    "s_addk_i32 m0, 0x1000\n\t" \
    "s_bitcmp1_b32 %[msk], %[t]\n\t" \
    "s_cbranch_scc1 64f\n\t" \
    "54:\n\t" \
    "buffer_load_dwordx4 %[w2], %[rw], %[ow] offen offset:0 lds\n\t" \
    "buffer_load_dwordx4 %[w3], %[rw], %[ow] offen offset:1024 lds\n\t" \
    "buffer_load_dwordx4 %[w4], %[rw], %[ow] offen offset:2048 lds\n\t" \
    "buffer_load_dwordx4 %[w5], %[rw], %[ow] offen offset:3072 lds\n\t" \
    "s_addk_i32 m0, 0x1000\n\t" \
    "s_ff1_i32_b32 %[id], %[rdy]\n\t" \
    "s_bitset0_b32 %[rdy], %[id]\n\t" \
    "buffer_load_dwordx4 %[w6], %[rw], %[ow] offen offset:0 lds\n\t" \
    "buffer_load_dwordx4 %[w7], %[rw], %[ow] offen offset:1024 lds\n\t" \
    "buffer_load_dwordx4 %[w8], %[rw], %[ow] offen offset:2048 lds\n\t" \
    "s_addk_i32 m0, 0x1000\n\t" \
    "s_mul_i32 %[t], %[id], %[mul]\n\t" \
    "s_add_u32 %[t], %[t], %[s0]\n\t" \
    "buffer_load_dwordx4 %[w10], %[rw], %[ow] offen offset:0 lds\n\t" \
    "buffer_load_dwordx4 %[w12], %[rw], %[ow] offen offset:2048 lds\n\t" \
    "s_addk_i32 m0, 0x1000\n\t" \
    "s_and_b32 %[oa], %[t], %[cut]\n\t" \
    "buffer_load_dwordx4 %[w14], %[rw], %[ow] offen offset:0 lds\n\t" \
    "buffer_load_dword %[zero], %[rd], %[od] offen offset:2048 lds\n\t" \
    "s_mov_b32 m0, %[dst]\n\t" \
    "s_cmp_ge_i32 %[id], 0\n\t" \
    "s_cbranch_scc1 4b\n\t" \
    "9:\n\t" \
    "s_mov_b32 %[xp], 0\n\t" \
    "s_mov_b32 %[np], 0\n\t" \
    "s_cmp_eq_u32 %[un], 0\n\t" \
    "s_cbranch_scc1 24f\n\t" \
      \
    "10:\n\t" \
    "s_getreg_b32 s68, hwreg(HW_REG_IB_STS)\n\t" \
    "s_lshr_b32 s69, s68, 18\n\t" \
    "s_and_b32 s69, s69, 0x30\n\t" \
    "s_and_b32 s68, s68, 15\n\t" \
    "s_or_b32 s68, s68, s69\n\t" \
    "s_cmp_eq_u32 %[u], 0\n\t" \
    "s_cbranch_scc1 12f\n\t" \
    "s_cmp_eq_u32 %[u], 2\n\t" \
    "s_cselect_b32 %[t], 21, 0\n\t" \
    "s_add_u32 %[t], %[t], %[xp]\n\t" \
    "s_min_u32 %[t], %[t], 63\n\t" \
    "s_cmp_gt_u32 s68, %[t]\n\t" \
    "s_cbranch_scc1 12f\n\t" \
    "s_getpc_b64 s[68:69]\n\t" \
    "s_lshl_b32 %[t], %[t], 3\n\t" \
    "s_add_u32 %[t], %[t], 20\n\t" \
    "s_add_u32 s68, s68, %[t]\n\t" \
    "s_addc_u32 s69, s69, 0\n\t" \
    "s_setpc_b64 s[68:69]\n\t" \
    "s_waitcnt vmcnt(0)\n\t" \
    "s_branch 13f\n\t" \
    "s_waitcnt vmcnt(1)\n\t" \
    "s_branch 13f\n\t" \
    "s_waitcnt vmcnt(2)\n\t" \
    "s_branch 13f\n\t" \
    "s_waitcnt vmcnt(3)\n\t" \
    "s_branch 13f\n\t" \
    "s_waitcnt vmcnt(4)\n\t" \
    "s_branch 13f\n\t" \
    "s_waitcnt vmcnt(5)\n\t" \
    "s_branch 13f\n\t" \
    "s_waitcnt vmcnt(6)\n\t" \
    "s_branch 13f\n\t" \
    "s_waitcnt vmcnt(7)\n\t" \
    "s_branch 13f\n\t" \
    "s_waitcnt vmcnt(8)\n\t" \
    "s_branch 13f\n\t" \
    "s_waitcnt vmcnt(9)\n\t" \
    "s_branch 13f\n\t" \
    "s_waitcnt vmcnt(10)\n\t" \
    "s_branch 13f\n\t" \
    "s_waitcnt vmcnt(11)\n\t" \
    "s_branch 13f\n\t" \
    "s_waitcnt vmcnt(12)\n\t" \
    "s_branch 13f\n\t" \
    "s_waitcnt vmcnt(13)\n\t" \
    "s_branch 13f\n\t" \
    "s_waitcnt vmcnt(14)\n\t" \
    "s_branch 13f\n\t" \
    "s_waitcnt vmcnt(15)\n\t" \
    "s_branch 13f\n\t" \
    "s_waitcnt vmcnt(16)\n\t" \
    "s_branch 13f\n\t" \
    "s_waitcnt vmcnt(17)\n\t" \
    "s_branch 13f\n\t" \
    "s_waitcnt vmcnt(18)\n\t" \
    "s_branch 13f\n\t" \
    "s_waitcnt vmcnt(19)\n\t" \
    "s_branch 13f\n\t" \
    "s_waitcnt vmcnt(20)\n\t" \
    "s_branch 13f\n\t" \
    "s_waitcnt vmcnt(21)\n\t" \
    "s_branch 13f\n\t" \
    "s_waitcnt vmcnt(22)\n\t" \
    "s_branch 13f\n\t" \
    "s_waitcnt vmcnt(23)\n\t" \
    "s_branch 13f\n\t" \
    "s_waitcnt vmcnt(24)\n\t" \
    "s_branch 13f\n\t" \
    "s_waitcnt vmcnt(25)\n\t" \
    "s_branch 13f\n\t" \
    "s_waitcnt vmcnt(26)\n\t" \
    "s_branch 13f\n\t" \
    "s_waitcnt vmcnt(27)\n\t" \
    "s_branch 13f\n\t" \
    "s_waitcnt vmcnt(28)\n\t" \
    "s_branch 13f\n\t" \
    "s_waitcnt vmcnt(29)\n\t" \
    "s_branch 13f\n\t" \
    "s_waitcnt vmcnt(30)\n\t" \
    "s_branch 13f\n\t" \
    "s_waitcnt vmcnt(31)\n\t" \
    "s_branch 13f\n\t" \
    "s_waitcnt vmcnt(32)\n\t" \
    "s_branch 13f\n\t" \
    "s_waitcnt vmcnt(33)\n\t" \
    "s_branch 13f\n\t" \
    "s_waitcnt vmcnt(34)\n\t" \
    "s_branch 13f\n\t" \
    "s_waitcnt vmcnt(35)\n\t" \
    "s_branch 13f\n\t" \
    "s_waitcnt vmcnt(36)\n\t" \
    "s_branch 13f\n\t" \
    "s_waitcnt vmcnt(37)\n\t" \
    "s_branch 13f\n\t" \
    "s_waitcnt vmcnt(38)\n\t" \
    "s_branch 13f\n\t" \
    "s_waitcnt vmcnt(39)\n\t" \
    "s_branch 13f\n\t" \
    "s_waitcnt vmcnt(40)\n\t" \
    "s_branch 13f\n\t" \
    "s_waitcnt vmcnt(41)\n\t" \
    "s_branch 13f\n\t" \
    "s_waitcnt vmcnt(42)\n\t" \
    "s_branch 13f\n\t" \
    "s_waitcnt vmcnt(43)\n\t" \
    "s_branch 13f\n\t" \
    "s_waitcnt vmcnt(44)\n\t" \
    "s_branch 13f\n\t" \
    "s_waitcnt vmcnt(45)\n\t" \
    "s_branch 13f\n\t" \
    "s_waitcnt vmcnt(46)\n\t" \
    "s_branch 13f\n\t" \
    "s_waitcnt vmcnt(47)\n\t" \
    "s_branch 13f\n\t" \
    "s_waitcnt vmcnt(48)\n\t" \
    "s_branch 13f\n\t" \
    "s_waitcnt vmcnt(49)\n\t" \
    "s_branch 13f\n\t" \
    "s_waitcnt vmcnt(50)\n\t" \
    "s_branch 13f\n\t" \
    "s_waitcnt vmcnt(51)\n\t" \
    "s_branch 13f\n\t" \
    "s_waitcnt vmcnt(52)\n\t" \
    "s_branch 13f\n\t" \
    "s_waitcnt vmcnt(53)\n\t" \
    "s_branch 13f\n\t" \
    "s_waitcnt vmcnt(54)\n\t" \
    "s_branch 13f\n\t" \
    "s_waitcnt vmcnt(55)\n\t" \
    "s_branch 13f\n\t" \
    "s_waitcnt vmcnt(56)\n\t" \
    "s_branch 13f\n\t" \
    "s_waitcnt vmcnt(57)\n\t" \
    "s_branch 13f\n\t" \
    "s_waitcnt vmcnt(58)\n\t" \
    "s_branch 13f\n\t" \
    "s_waitcnt vmcnt(59)\n\t" \
    "s_branch 13f\n\t" \
    "s_waitcnt vmcnt(60)\n\t" \
    "s_branch 13f\n\t" \
    "s_waitcnt vmcnt(61)\n\t" \
    "s_branch 13f\n\t" \
    "s_waitcnt vmcnt(62)\n\t" \
    "s_branch 13f\n\t" \
    "s_waitcnt vmcnt(63)\n\t" \
    "s_branch 13f\n\t" \
    "13:\n\t" \
    "s_barrier\n\t" \
    "s_sub_u32 %[u], %[u], 1\n\t" \
    "s_branch 8f\n\t" \
    "12:\n\t" \
    "s_cmp_ge_u32 %[wm], 1048576\n\t" \
    "s_cbranch_scc1 15f\n\t" \
    "s_add_u32 %[t], s68, 8\n\t" \
    "s_cmp_gt_i32 %[t], %[sw]\n\t" \
    "s_cbranch_scc1 15f\n\t" \
    "buffer_load_dword %[dump], %[c0], %[rt], %[wm] offen\n\t" \
    "buffer_load_dword %[dump], %[c1], %[rt], %[wm] offen\n\t" \
    "buffer_load_dword %[dump], %[c2], %[rt], %[wm] offen\n\t" \
    "s_add_u32 %[wm], %[wm], 0x100000\n\t" \
    "s_add_u32 %[xp], %[xp], 3\n\t" \
    "s_mov_b32 %[sw], 0\n\t" \
    "s_mov_b32 %[np], 0\n\t" \
    "15:\n\t" \
    "s_min_u32 %[t], s68, %[np]\n\t" \
    "s_cmp_ge_u32 %[t], 2\n\t" \
    "s_cbranch_scc1 17f\n\t" \
    "s_mov_b32 exec_hi, 0\n\t" \
    "global_load_dword %[f], %[foff], %[ws] sc0 sc1\n\t" \
    "s_mov_b32 exec_hi, -1\n\t" \
    "s_add_u32 %[xp], %[xp], 1\n\t" \
    "s_add_u32 %[sw], %[sw], 1\n\t" \
    "s_add_u32 %[np], %[np], 1\n\t" \
    "17:\n\t" \
    "s_memrealtime s[68:69]\n\t" \
    "v_readlane_b32 %[t], %[f], %[lt]\n\t" \
    "s_waitcnt lgkmcnt(0)\n\t" \
    "s_cmp_eq_u32 %[t], %[call]\n\t" \
    "s_cbranch_scc1 18f\n\t" \
    "s_sub_u32 s68, s68, %[t0]\n\t" \
    "s_cmp_gt_u32 s68, %[dl]\n\t" \
    "s_cbranch_scc1 18f\n\t" \
    "8:\n\t" \
    "v_cmp_ge_u32 vcc, %[f], %[vcall]\n\t" \
    "s_and_b32 %[t], vcc_lo, %[un]\n\t" \
    "s_or_b32 %[rdy], %[rdy], %[t]\n\t" \
    "s_andn2_b32 %[un], %[un], vcc_lo\n\t" \
    "s_cselect_b32 %[msk], %[msk], 0\n\t" \
    "s_ff1_i32_b32 %[id], %[rdy]\n\t" \
    "s_bitset0_b32 %[rdy], %[id]\n\t" \
    "s_mul_i32 %[t], %[id], %[mul]\n\t" \
    "s_add_u32 %[t], %[t], %[s0]\n\t" \
    "s_and_b32 %[oa], %[t], %[cut]\n\t" \
    "s_cmp_lt_i32 %[id], 0\n\t" \
    "s_cbranch_scc1 10b\n\t" \
    "s_branch 4b\n\t" \
      \
    "60:\n\t" \
    "v_cmp_ge_u32 vcc, %[f], %[vcall]\n\t" \
    "s_and_b32 %[t], vcc_lo, %[un]\n\t" \
    "s_or_b32 %[rdy], %[rdy], %[t]\n\t" \
    "s_andn2_b32 %[un], %[un], vcc_lo\n\t" \
    "s_cselect_b32 %[msk], %[msk], 0\n\t" \
    "s_cbranch_scc0 50b\n\t" \
    "s_bcnt1_i32_b32 %[t], %[rdy]\n\t" \
    "s_cmp_gt_u32 %[t], 5\n\t" \
    "s_cbranch_scc1 50b\n\t" \
    "s_mov_b32 exec_hi, 0\n\t" \
    "global_load_dword %[f], %[foff], %[ws] sc0 sc1\n\t" \
    "s_mov_b32 exec_hi, -1\n\t" \
    "s_branch 50b\n\t" \
    "61:\n\t" \
    "v_cmp_ge_u32 vcc, %[f], %[vcall]\n\t" \
    "s_and_b32 %[t], vcc_lo, %[un]\n\t" \
    "s_or_b32 %[rdy], %[rdy], %[t]\n\t" \
    "s_andn2_b32 %[un], %[un], vcc_lo\n\t" \
    "s_cselect_b32 %[msk], %[msk], 0\n\t" \
    "s_cbranch_scc0 51b\n\t" \
    "s_bcnt1_i32_b32 %[t], %[rdy]\n\t" \
    "s_cmp_gt_u32 %[t], 5\n\t" \
    "s_cbranch_scc1 51b\n\t" \
    "s_mov_b32 exec_hi, 0\n\t" \
    "global_load_dword %[f], %[foff], %[ws] sc0 sc1\n\t" \
    "s_mov_b32 exec_hi, -1\n\t" \
    "s_branch 51b\n\t" \
    "62:\n\t" \
    "v_cmp_ge_u32 vcc, %[f], %[vcall]\n\t" \
    "s_and_b32 %[t], vcc_lo, %[un]\n\t" \
    "s_or_b32 %[rdy], %[rdy], %[t]\n\t" \
    "s_andn2_b32 %[un], %[un], vcc_lo\n\t" \
    "s_cselect_b32 %[msk], %[msk], 0\n\t" \
    "s_cbranch_scc0 52b\n\t" \
    "s_bcnt1_i32_b32 %[t], %[rdy]\n\t" \
    "s_cmp_gt_u32 %[t], 5\n\t" \
    "s_cbranch_scc1 52b\n\t" \
    "s_mov_b32 exec_hi, 0\n\t" \
    "global_load_dword %[f], %[foff], %[ws] sc0 sc1\n\t" \
    "s_mov_b32 exec_hi, -1\n\t" \
    "s_branch 52b\n\t" \
    "63:\n\t" \
    "v_cmp_ge_u32 vcc, %[f], %[vcall]\n\t" \
    "s_and_b32 %[t], vcc_lo, %[un]\n\t" \
    "s_or_b32 %[rdy], %[rdy], %[t]\n\t" \
    "s_andn2_b32 %[un], %[un], vcc_lo\n\t" \
    "s_cselect_b32 %[msk], %[msk], 0\n\t" \
    "s_cbranch_scc0 53b\n\t" \
    "s_bcnt1_i32_b32 %[t], %[rdy]\n\t" \
    "s_cmp_gt_u32 %[t], 5\n\t" \
    "s_cbranch_scc1 53b\n\t" \
    "s_mov_b32 exec_hi, 0\n\t" \
    "global_load_dword %[f], %[foff], %[ws] sc0 sc1\n\t" \
    "s_mov_b32 exec_hi, -1\n\t" \
    "s_branch 53b\n\t" \
    "64:\n\t" \
    "v_cmp_ge_u32 vcc, %[f], %[vcall]\n\t" \
    "s_and_b32 %[t], vcc_lo, %[un]\n\t" \
    "s_or_b32 %[rdy], %[rdy], %[t]\n\t" \
    "s_andn2_b32 %[un], %[un], vcc_lo\n\t" \
    "s_cselect_b32 %[msk], %[msk], 0\n\t" \
    "s_cbranch_scc0 54b\n\t" \
    "s_bcnt1_i32_b32 %[t], %[rdy]\n\t" \
    "s_cmp_gt_u32 %[t], 5\n\t" \
    "s_cbranch_scc1 54b\n\t" \
    "s_mov_b32 exec_hi, 0\n\t" \
    "global_load_dword %[f], %[foff], %[ws] sc0 sc1\n\t" \
    "s_mov_b32 exec_hi, -1\n\t" \
    "s_branch 54b\n\t" \
      \
    "7:\n\t" \
    "s_cmp_eq_u32 %[xp], 5\n\t" \
    "s_cbranch_scc1 11f\n\t" \
    "s_mov_b32 exec_hi, 0\n\t" \
    "global_load_dword %[f], %[foff], %[ws] sc0 sc1\n\t" \
    "s_mov_b32 exec_hi, -1\n\t" \
    "s_branch 3b\n\t" \
    "11:\n\t" \
    "v_cmp_ge_u32 vcc, %[f], %[vcall]\n\t" \
    "s_and_b32 %[t], vcc_lo, %[un]\n\t" \
    "s_bcnt1_i32_b32 %[t], %[t]\n\t" \
    "s_cmp_gt_u32 %[t], 1\n\t" \
    "s_cbranch_scc1 3b\n\t" \
    "buffer_load_dword %[dump], %[c0], %[rt], %[wm] offen\n\t" \
    "buffer_load_dword %[dump], %[c1], %[rt], %[wm] offen\n\t" \
    "buffer_load_dword %[dump], %[c2], %[rt], %[wm] offen\n\t" \
    "s_add_u32 %[wm], %[wm], 0x100000\n\t" \
    "s_mov_b32 %[sw], 0\n\t" \
    "s_branch 3b\n\t" \
      \
    "24:\n\t" \
    "v_readlane_b32 %[t], %[f], %[lt]\n\t" \
    "s_cmp_eq_u32 %[t], %[call]\n\t" \
    "s_cbranch_scc1 18f\n\t" \
    "s_cmp_eq_u32 %[u], 1\n\t" \
    "s_cbranch_scc1 25f\n\t" \
    "s_waitcnt vmcnt(21)\n\t" \
    "s_barrier\n\t" \
    "25:\n\t" \
    "s_waitcnt vmcnt(0)\n\t" \
    "s_barrier\n\t" \
    "s_mov_b32 %[u], 0\n\t" \
    "s_branch 19f\n\t" \
    "18:\n\t" \
    "s_mov_b32 %[u], 1\n\t" \
    "s_waitcnt vmcnt(0)\n\t" \
    "19:\n\t"

#define LOADER_HEAD_ASM \
    "s_mov_b32 m0, 0\n\t" \
    "s_lshl_b32 %[ow], %[oa], 12\n\t" \
    "buffer_load_dwordx4 %[a0], %[rx], %[oa] offen offset:0 lds\n\t" \
    "s_add_u32 %[ow], %[ow], %[own]\n\t" \
    "buffer_load_dwordx4 %[a1], %[rx], %[oa] offen offset:1024 lds\n\t" \
    "buffer_load_dwordx4 %[a2], %[rx], %[oa] offen offset:2048 lds\n\t" \
    "buffer_load_dwordx4 %[a3], %[rx], %[oa] offen offset:3072 lds\n\t" \
    "s_addk_i32 m0, 0x1000\n\t" \
    "s_nop 0\n\t" \
    "buffer_load_dwordx4 %[a4], %[rx], %[oa] offen offset:0 lds\n\t" \
    "buffer_load_dwordx4 %[a5], %[rx], %[oa] offen offset:1024 lds\n\t" \
    "buffer_load_dwordx4 %[a6], %[rx], %[oa] offen offset:2048 lds\n\t" \
    "buffer_load_dwordx4 %[a7], %[rx], %[oa] offen offset:3072 lds\n\t" \
    "s_addk_i32 m0, 0x1000\n\t" \
    "s_cmp_lt_u32 %[rb], 4\n\t" \
    "s_cbranch_scc0 1f\n\t" \
    "buffer_load_dwordx4 %[a8], %[rx], %[oa] offen offset:0 lds\n\t" \
    "buffer_load_dwordx4 %[a9], %[rx], %[oa] offen offset:1024 lds\n\t" \
    "1:\n\t" \
    "buffer_load_dwordx4 %[w0], %[rw], %[ow] offen offset:2048 lds\n\t" \
    "buffer_load_dwordx4 %[w1], %[rw], %[ow] offen offset:3072 lds\n\t" \
    "s_addk_i32 m0, 0x1000\n\t" \
    "s_addk_i32 %[oa], 128\n\t" \
    "s_bitset0_b32 %[oa], 9\n\t" \
    "buffer_load_dwordx4 %[w2], %[rw], %[ow] offen offset:0 lds\n\t" \
    "buffer_load_dwordx4 %[w3], %[rw], %[ow] offen offset:1024 lds\n\t" \
    "buffer_load_dwordx4 %[w4], %[rw], %[ow] offen offset:2048 lds\n\t" \
    "buffer_load_dwordx4 %[w5], %[rw], %[ow] offen offset:3072 lds\n\t" \
    "s_addk_i32 m0, 0x1000\n\t" \
    "s_nop 0\n\t" \
    "buffer_load_dwordx4 %[w6], %[rw], %[ow] offen offset:0 lds\n\t" \
    "buffer_load_dwordx4 %[w7], %[rw], %[ow] offen offset:1024 lds\n\t" \
    "buffer_load_dwordx4 %[w8], %[rw], %[ow] offen offset:2048 lds\n\t" \
    "buffer_load_dwordx4 %[w9], %[rw], %[ow] offen offset:3072 lds\n\t" \
    "s_addk_i32 m0, 0x1000\n\t" \
    "s_nop 0\n\t" \
    "buffer_load_dwordx4 %[w10], %[rw], %[ow] offen offset:0 lds\n\t" \
    "buffer_load_dwordx4 %[w11], %[rw], %[ow] offen offset:1024 lds\n\t" \
    "buffer_load_dwordx4 %[w12], %[rw], %[ow] offen offset:2048 lds\n\t" \
    "buffer_load_dwordx4 %[w13], %[rw], %[ow] offen offset:3072 lds\n\t" \
    "s_addk_i32 m0, 0x1000\n\t" \
    "s_nop 0\n\t" \
    "buffer_load_dwordx4 %[w14], %[rw], %[ow] offen offset:0 lds\n\t" \
    "buffer_load_dwordx4 %[w15], %[rw], %[ow] offen offset:1024 lds\n\t"

__device__ __forceinline__ u32x4 make_rsrc(const void* base, unsigned bytes) {
    return u32x4{(unsigned)(size_t)base, (unsigned)((size_t)base >> 32) & 0xffffu, bytes, 0x00020000u};
}
__device__ __forceinline__ void loader(const Head& p, unsigned lds0, int b, int rb, int row0, int cb) {
    int lane = lane_id();
    const unsigned r = lane >> 3, c = lane & 7;
    const unsigned sw0 = (c ^ ((r >> 1) & 3)) << 4;
    unsigned va[10], vw[16];
#pragma unroll
    for (int i = 0; i < 10; ++i) va[i] = (8 * i + r) * KC + ((kperm(sw0 >> 4) ^ kperm((i & 1) << 2)) << 4) + 3072 - (i & 3) * 1024;
#pragma unroll
    for (int i = 0; i < 16; ++i) vw[i] = (16 * kperm(i >> 1) + 8 * (i & 1) + r) * NTOT + (sw0 ^ ((i & 2) << 5)) + 3072 - ((i + 10) & 3) * 1024;
    asm volatile("" : "+v"(va[0]), "+v"(va[1]), "+v"(va[2]), "+v"(va[3]), "+v"(va[4]), "+v"(va[5]), "+v"(va[6]), "+v"(va[7]),
                      "+v"(va[8]), "+v"(va[9]), "+v"(vw[0]), "+v"(vw[1]), "+v"(vw[2]), "+v"(vw[3]), "+v"(vw[4]), "+v"(vw[5]),
                      "+v"(vw[6]), "+v"(vw[7]), "+v"(vw[8]), "+v"(vw[9]), "+v"(vw[10]), "+v"(vw[11]), "+v"(vw[12]),
                      "+v"(vw[13]), "+v"(vw[14]), "+v"(vw[15]));
    const int rank = p.rank;
    const u32x4 rx = make_rsrc(p.x + row0 * KC - 3072, 3072 + (512 - row0) * KC);
    const u32x4 rw = make_rsrc(p.w + cb * BN - 3072, 3072 + 4096 * NTOT - cb * BN);
    unsigned oa = ((rb + cb) & 3) * BK, ow;
#define LOADER_XW                                                                                                 \
          [a0] "v"(va[0]), [a1] "v"(va[1]), [a2] "v"(va[2]), [a3] "v"(va[3]), [a4] "v"(va[4]),                    \
          [a5] "v"(va[5]), [a6] "v"(va[6]), [a7] "v"(va[7]), [a8] "v"(va[8]), [a9] "v"(va[9]),                    \
          [w0] "v"(vw[0]), [w1] "v"(vw[1]), [w2] "v"(vw[2]), [w3] "v"(vw[3]), [w4] "v"(vw[4]), [w5] "v"(vw[5]),   \
          [w6] "v"(vw[6]), [w7] "v"(vw[7]), [w8] "v"(vw[8]), [w9] "v"(vw[9]), [w10] "v"(vw[10]),                  \
          [w11] "v"(vw[11]), [w12] "v"(vw[12]), [w13] "v"(vw[13]), [w14] "v"(vw[14]), [w15] "v"(vw[15]),          \
          [rx] "s"(rx), [rw] "s"(rw), [own] "s"(rank * (KC * NTOT)), [rb] "s"(rb)
    asm volatile(LOADER_HEAD_ASM : [ow] "=&s"(ow), [oa] "+&s"(oa) : LOADER_XW : "memory", "scc");
    asm volatile("" : "+v"(lane));
    const unsigned s0 = (rank - 1 - rb) & 7, qo = (rank - s0) & 7;
    const unsigned ks = (lane >> 3) & 3, src = (lane + s0) & 7;
    const unsigned cnt_off = (unsigned)WS_CNT + b * 256;
    unsigned foff = src * (unsigned)WS_SRC + (unsigned)WS_FLAG + (rb * 8 + src) * 128 + ks * 4;
    if (src == (unsigned)rank) foff = ks == 1 ? cnt_off : (unsigned)WS_TMO;
    unsigned l0 = lane * 16, l1 = l0 + 4096, l2 = l0 + 8192, un = ~(0x01010101u << qo);
    const u32x4 ra = make_rsrc(p.myws + row0 * BK, WS_END - row0 * BK);
    const u32x4 rt = make_rsrc(p.w + (size_t)s0 * (KC * NTOT) + cb * BN, KC * NTOT - cb * BN);
    unsigned vc[3], vk[16];
#pragma unroll
    for (int i = 0; i < 3; ++i) {
        const unsigned n = i * 64 + lane, hi = n >= 96, m = n - 96 * hi;
        const unsigned lr = m < 64 ? m : m + ((m - 64) & ~7u);
        vc[i] = (hi * 128 + 16 * kperm(lr >> 4) + (lr & 15)) * NTOT;
    }
#pragma unroll
    for (int u = 0; u < 16; ++u) {
        const int uw = 9 + (u & 6);
        vk[u] = vw[uw] + (((u >> 3) * 2 + (u & 1)) * BK * NTOT + ((uw + 10) & 3) * 1024 - ((u + 10) & 3) * 1024);
    }
    const unsigned char* tab;
    asm volatile("s_getpc_b64 %0\n\ts_branch 1f\n\t"
                 ".long 0, 4, 8, 12, 16, 20, 24, 28, 32, 36, 40, 44, 48, 52, 56, 60\n\t"
                 ".long 64, 68, 72, 76, 80, 84, 88, 92, 96, 100, 104, 108, 112, 116, 120, 124\n\t"
                 ".long 0, 4, 8, 12, 16, 20, 24, 28, 32, 36, 40, 44, 48, 52, 56, 60\n\t"
                 ".long 64, 68, 72, 76, 80, 84, 88, 92, 96, 100, 104, 108, 112, 116, 120, 124\n\t"
                 "1:" : "=s"(tab));
    const u32x4 rd = make_rsrc(tab + 4 + 16 * (7 - rank) - 2048, 2048 + 128);
    unsigned od;
    unsigned f, vcall, call, u, dst, id, t, rdy, msk, tp, np, xp, wm, sw, dump;
    static_assert(WS_PARITY == (1u << 16) && 2 * WS_KS == BK * NTOT && D * SLOT == (1 << 17), "shifts in the loader");
    asm volatile("s_cmp_lt_u32 %[rb], 4\n\ts_cbranch_scc0 30f\n\t" LOADER5_ASM "s_branch 31f\n\t30:\n\t" LOADER4_ASM "31:"
        : [f] "=&v"(f), [vcall] "=&v"(vcall), [call] "=&s"(call), [u] "=&s"(u), [dst] "=&s"(dst), [id] "=&s"(id),
          [t] "=&s"(t), [rdy] "=&s"(rdy), [msk] "=&s"(msk), [tp] "=&s"(tp), [np] "=&s"(np), [xp] "=&s"(xp),
          [ow] "=&s"(ow), [od] "=&s"(od), [wm] "=&s"(wm), [sw] "=&s"(sw), [dump] "=&v"(dump), [oa] "+&s"(oa),
          [un] "+&s"(un), [l0] "+&v"(l0), [l1] "+&v"(l1), [l2] "+&v"(l2)
        : LOADER_XW, [foff] "v"(foff), [s0] "s"(s0 << 20), [mul] "s"(0x108000), [cut] "s"(0x7c0000), [lt] "s"(qo), [lc] "s"(qo + 8),
          [c0] "v"(vc[0]), [c1] "v"(vc[1]), [c2] "v"(vc[2]),
          [k0] "v"(vk[0]), [k1] "v"(vk[1]), [k2] "v"(vk[2]), [k3] "v"(vk[3]), [k4] "v"(vk[4]), [k5] "v"(vk[5]),
          [k6] "v"(vk[6]), [k7] "v"(vk[7]), [k8] "v"(vk[8]), [k9] "v"(vk[9]), [k10] "v"(vk[10]), [k11] "v"(vk[11]),
          [k12] "v"(vk[12]), [k13] "v"(vk[13]), [k14] "v"(vk[14]), [k15] "v"(vk[15]), [zero] "v"(0u), [rd] "s"(rd),
          [ws] "s"(p.myws), [ra] "s"(ra), [rt] "s"(rt), [t0] "s"(now32()), [slot] "s"(SLOT), [dl] "s"(DEADLINE_TICKS)
        : "memory", "scc", "vcc", "s68", "s69");
    if (u) {
        __hip_atomic_store((unsigned*)(p.myws + WS_TMO), call, __ATOMIC_RELAXED, __HIP_MEMORY_SCOPE_AGENT);
        lds_wr32(lds0 + LDS_ABORT, 1);
    }
    if (lane == 0) __hip_atomic_store((unsigned*)(p.myws + cnt_off), call, __ATOMIC_RELAXED, __HIP_MEMORY_SCOPE_AGENT);
}

__device__ __forceinline__ void consumer(const Head& p, unsigned lds0, int lane, int cw, int rb, int row0, int cb) {
    if (cw == 0) lds_wr32(lds0 + LDS_ABORT, 0);
    float sx, sw;
    asm volatile("s_load_dword %0, %2, 0x0\n\ts_load_dword %1, %3, 0x0\n\ts_waitcnt lgkmcnt(0)"
                 : "=&s"(sx), "=&s"(sw) : "s"(p.sx), "s"(p.sw) : "memory");
    float sc = sx * sw;
    unsigned wb0, wb1, ab0, ab1;
    {
        const unsigned q = lane >> 4, i = (lane & 15) >> 1, pp = lane & 1;
        const unsigned fw = ((i >> 1) & 3) | ((q & 1) << 2);
        wb0 = lds0 + SLOT_W + (16 * q + i) * 128 + 8 * pp + (((cw * 2) ^ fw) << 4);
        wb1 = lds0 + SLOT_W + (16 * q + i) * 128 + 8 * pp + (((cw * 2 + 1) ^ fw) << 4);
        const unsigned row = lane & 15, fa = (row >> 1) & 7;
        ab0 = lds0 + row * 128 + ((q ^ fa) << 4);
        ab1 = lds0 + row * 128 + (((q + 4) ^ fa) << 4);
    }
    unsigned vo = (lane >> 5) * (8 * NTOT * 4) + (lane & 31) * 4;
    const unsigned ooff = (row0 * NTOT + cb * BN + cw * 32) * 4;
    const u32x4 ro = make_rsrc((const unsigned char*)p.out + ooff, 512 * NTOT * 4 - ooff);
    unsigned n, t, t0, vd;
    static_assert(LDS_ABORT == 2 * SLOT + 65520, "offset in the consumer");
    asm volatile(CONS_ZERO "s_cmp_lt_u32 %[rb], 4\n\ts_cbranch_scc0 20f\n\t" CONS5_ASM "s_branch 21f\n\t20:\n\t" CONS4_ASM "21:"
        : [n] "=&s"(n), [t] "=&s"(t), [t0] "=&s"(t0), [sc] "+&v"(sc), [vo] "+&v"(vo), [vd] "=&v"(vd)
        : [w00] "v"(wb0), [w10] "v"(wb1), [a00] "v"(ab0), [a10] "v"(ab1),
          [w01] "v"(wb0 + 2 * SLOT), [w11] "v"(wb1 + 2 * SLOT), [a01] "v"(ab0 + 2 * SLOT), [a11] "v"(ab1 + 2 * SLOT),
          [z0] "v"(lds0), [z1] "v"(lds0 + 2 * SLOT), [ro] "s"(ro), [rb] "s"(rb)
        : "memory", "scc", "vcc", CONS_CLOBBERS);
}

__device__ __forceinline__ void touch_w(const Head& p, int lane, int b, int ks, int j0, int dj) {
    unsigned dump = 0;
    const unsigned rr = lane >> 2;
    for (int j = j0; j < 42; j += dj) {
        const int q = (j * 43) >> 8, i = j - 6 * q;
        const unsigned lr = i < 4 ? 16 * i + rr : 32 * i - 64 + rr + (rr & 8);
        asm volatile("global_load_dword %0, %1, %2"
                     : "+v"(dump) : "v"((16 * kperm(lr >> 4) + (lr & 15)) * NTOT + (lane & 3) * 128),
                       "s"(p.w + (size_t)(((p.rank + 1 + q) & 7) * KC + ks * BK) * NTOT + (b & 7) * 512) : "memory");
    }
    asm volatile("s_waitcnt vmcnt(0)" : "+v"(dump) :: "memory");
}

template <bool FIRST>
__device__ __forceinline__ void push_piece(const Head& p, const Link& lk, int lane, int wave, int b, int e, int slot,
                                           unsigned& t0, unsigned& call) {
    const int ks = (slot * 37) >> 8;
    const int g = slot + e - 7 * (((slot + e) * 37) >> 8);
    const int row0 = rb_row0(g);
    const unsigned r = lane >> 3, c = lane & 7;
    const unsigned vo0 = (row0 + r) * KC + ks * BK + (kperm(c ^ ((r >> 1) & 3)) << 4);
    const unsigned vo1 = (vo0 ^ (kperm(4) << 4)) + 8 * KC;
    const unsigned cnt_off = (unsigned)WS_CNT + (b * 8 + wave) * 32;
    const int delay = (slot - 1) * K_TAU;
    if (!FIRST || wave != 0) {
        if (!FIRST || wave >= 3) {
            if (FIRST) { BARRIER(); t0 = now32(); }
            while ((int)(now32() - t0) + K_LEAD < delay) __builtin_amdgcn_s_sleep(1);
        } else {
            __builtin_amdgcn_s_sleep(4);
            if (wave == 2) __builtin_amdgcn_s_sleep(4);
        }
    }
    u32x4 dd[10];
#pragma unroll
    for (int u = 0; u < 10; ++u)
        asm volatile("global_load_dwordx4 %0, %1, %2"
                     : "=v"(dd[u]) : "v"(((u & 1) ? vo1 : vo0) + (u < 8 || g < 4 ? (u >> 1) * 16 * KC : 0)), "s"(lk.src), "s"(p.myws) : "memory");
    if (FIRST) {
        asm volatile("global_load_dword %0, %1, %2 sc1" : "=v"(call) : "v"(cnt_off), "s"(p.myws) : "memory");
        if (wave == 1 || wave == 2) { BARRIER(); t0 = now32(); }
    }
    unsigned so0 = ks * (unsigned)WS_KS + row0 * BK + lane * 16;
    asm volatile("s_waitcnt vmcnt(0)"
                 : "+v"(call), "+v"(so0), "+v"(dd[0]), "+v"(dd[1]), "+v"(dd[2]), "+v"(dd[3]), "+v"(dd[4]),
                   "+v"(dd[5]), "+v"(dd[6]), "+v"(dd[7]), "+v"(dd[8]), "+v"(dd[9]) :: "memory");
    if (FIRST) {
        if (wave == 0) { BARRIER(); t0 = now32(); }
        call += 1;
    }
    so0 += (call & 1) * (unsigned)WS_PARITY;
    if (delay > 0)
        while ((int)(now32() - t0) < delay) __builtin_amdgcn_s_sleep(1);
#pragma unroll
    for (int u = 0; u < 10; ++u)
        if (u < 8 || g < 4)
            asm volatile("global_store_dwordx4 %0, %1, %2 offset:%3 sc0 sc1\n\ts_nop 1"
                         :: "v"(so0 + (u >> 2) * 4096), "v"(dd[u]), "s"(lk.dst), "i"((u & 3) * 1024) : "memory");
    unsigned flag_off = (unsigned)WS_FLAG + (g * 8 + p.rank) * 128 + ks * 4;
    asm volatile("s_waitcnt vmcnt(0)" : "+s"(flag_off) :: "memory");
    if (lane == 0) {
        __hip_atomic_store((unsigned*)(lk.dst + flag_off), call, __ATOMIC_RELAXED, __HIP_MEMORY_SCOPE_SYSTEM);
        if (FIRST) __hip_atomic_store((unsigned*)(p.myws + cnt_off), call, __ATOMIC_RELAXED, __HIP_MEMORY_SCOPE_AGENT);
    }
}

__global__ __launch_bounds__(NTHREADS, 2) void a2a_gemm(Params args) {
    asm volatile("s_load_dword s90, %0, 0x0" :: "s"(__builtin_amdgcn_kernarg_segment_ptr()) : "s90");
    const Head& p = args.h;
    const unsigned lds0 = 0;
    const int lane = threadIdx.x & 63;
    const int wave = __builtin_amdgcn_readfirstlane(threadIdx.x >> 6);
    const int b = blockIdx.x;
    if (b >= 32) {
        const int rb = (b >> 5) - 1;
        const int cb = (b & 7) * 4 + ((b >> 3) & 3);
        const int row0 = rb_row0(rb);
        if (wave != 0) {
            if (wave <= 4) consumer(p, lds0, lane, wave - 1, rb, row0, cb);
            return;
        }
        loader(p, lds0, b, rb, row0, cb);
        return;
    }
    unsigned t0 = 0;
    if (b < 21) {
        const int part = (b * 37) >> 8, e = b - 7 * part;
        const Link lk = args.link[e];
        unsigned call;
        push_piece<true>(p, lk, lane, wave, b, e, 3 * wave + part, t0, call);
        if (3 * wave + part < 7) push_piece<false>(p, lk, lane, wave, b, e, 3 * wave + part + 21, t0, call);
    } else {
        t0 = now32() + 100;
    }
    if (b >= 24) {
        touch_w(p, lane, b, 2, wave, 7);
        touch_w(p, lane, b, 3, wave, 7);
    } else {
        while ((int)(now32() - t0) < K_PFT) __builtin_amdgcn_s_sleep(2);
        touch_w(p, lane, b, 3, (b >> 3) * 7 + wave, 21);
    }
}

}

extern "C" void kernel_launch(int rank, int nranks, ncclComm_t comm,
                              hipStream_t stream,
                              void* const* d_in, const int* in_sizes, int n_in,
                              void* d_out, int out_size,
                              const int* in_dtypes, int out_dtype,
                              void* const* d_out_all,
                              void* const* d_ws_all, size_t ws_bytes) {
    if (nranks != NR || (unsigned)rank >= (unsigned)NR || ws_bytes < WS_END || n_in != 4 ||
        in_sizes[0] != 4096 * KC || in_sizes[1] != 4096 * NTOT || out_size != 512 * NTOT) {
        fprintf(stderr, "kernel_launch: unexpected problem shape\n");
        return;
    }
    HIP_CHECK(hipFuncSetAttribute((const void*)a2a_gemm, hipFuncAttributeMaxDynamicSharedMemorySize, LDS_BYTES));
    Params p;
    const unsigned char* x = (const unsigned char*)d_in[0];
    p.h.x = x + (size_t)rank * 512 * KC;
    p.h.w = (const unsigned char*)d_in[1];
    p.h.myws = (unsigned char*)d_ws_all[rank];
    p.h.out = (float*)d_out;
    p.h.sx = (const float*)d_in[2];
    p.h.sw = (const float*)d_in[3];
    p.h.rank = rank;
    p.h.pad[0] = p.h.pad[1] = p.h.pad[2] = 0;
    for (int e = 0; e < 8; ++e) {
        const int peer = (rank + 1 + e) & 7;
        p.link[e].dst = (unsigned char*)d_ws_all[peer] + rank * WS_SRC;
        p.link[e].src = x + (size_t)peer * 512 * KC;
    }
    a2a_gemm<<<NWG, NTHREADS, LDS_BYTES, stream>>>(p);
    HIP_CHECK(hipGetLastError());
}
```
